```python
import numpy as np

kernel_name = "gpu_gemm_rs_m4096_k4096_n16384_w8a8_dequant"


ABSMAX_RTOL = 0.001

COMPUTE_DTYPE = "int8"
EPILOGUE = "dequant"
IN_QUANT = "int8"
OUT_QUANT = None
IN_DTYPES = {"x": "int8", "w_mat": "int8",
             "scale_x": "float32", "scale_w": "float32"}
WORLD_SIZE = 8

def setup_inputs():

    x = np.random.randint(-127, 128, size=(4096, 4096)).astype(np.int8)
    w_mat = np.random.randint(-127, 128, size=(4096, 16384)).astype(np.int8)
    scale_x = (0.02 * (1.0 + np.random.rand(1))).astype(np.float32)
    scale_w = (0.02 * (1.0 + np.random.rand(1))).astype(np.float32)
    return {"x": x, "w_mat": w_mat, "scale_x": scale_x, "scale_w": scale_w}

def reference(x, w_mat, scale_x, scale_w):
    acc = x.astype(np.float64) @ w_mat.astype(np.float64)
    y = (acc.astype(np.float32) * (scale_x * scale_w)).astype(np.float32)
    return y
```

```cpp
#include <hip/hip_runtime.h>
#include <rccl/rccl.h>
#include <cstdio>

#define HIP_CHECK(expr) do { hipError_t _e = (expr); if (_e != hipSuccess) { \
    fprintf(stderr, "HIP error %s at %s:%d\n", hipGetErrorString(_e), __FILE__, __LINE__); \
    return; } } while (0)

typedef unsigned u32;
typedef unsigned long long u64;
typedef u32 u32x4 __attribute__((ext_vector_type(4)));
typedef u32 u32x3 __attribute__((ext_vector_type(3)));
typedef int i32x4 __attribute__((ext_vector_type(4)));
typedef __amdgpu_buffer_rsrc_t rsrc_t;
#define LDS_AS __attribute__((address_space(3)))

namespace {

constexpr int NR = 8;
constexpr int KSL = 512;
constexpr int NCOL = 16384;
constexpr int NWG = 256, NTHR = 512;
constexpr int WPL = 8;
constexpr int NSTR = 7 * WPL;
constexpr int NGEM = NWG - NSTR;
#define FIN_WAVES 5

constexpr u32 KiB = 1024u, MiB = 1024u * 1024u;
constexpr u32 F_CNT = 0;
constexpr u32 F_TMO = 32 * KiB;
constexpr u32 F_S = 64 * KiB;
constexpr u32 F_I = 160 * KiB;
constexpr u32 F_LW = 256 * KiB;
constexpr u32 F_O = 288 * KiB;
constexpr u32 F_P = 320 * KiB;
constexpr u32 XBLK = 16 * KiB, WBLK = 32 * KiB;
constexpr u32 D_IX = 1 * MiB;
constexpr u32 D_IW = D_IX + 16 * MiB;
constexpr u32 D_PAR = D_IW + 8 * MiB;
constexpr u32 P_EX = 0;
constexpr u32 P_EW = 2 * MiB;
constexpr u32 SUB = 2816;
constexpr u32 P_PK = 10 * MiB;
constexpr u32 P_RAW = 32 * MiB;
constexpr u32 PAR_BYTES = 64 * MiB;
constexpr u32 WS_NEED = D_PAR + 2 * PAR_BYTES;
constexpr u32 OUT_BYTES = 512u * NCOL * 4u;

constexpr u32 STAGE = XBLK + WBLK;
constexpr u32 LDS_CTL = 3 * STAGE;
constexpr u32 LDS_BYTES = LDS_CTL + 256;

constexpr u64 DEADLINE = 40000000ull;
constexpr int AUX_SYS = 17, AUX_AGT = 16;

struct Params {
  const signed char* x;
  const signed char* w;
  const float* sx;
  const float* sw;
  float* out;
  char* ws[NR];
  u32 ws_bytes;
  int rank;
};

__device__ __forceinline__ rsrc_t mk_rsrc(const void* p, u32 bytes) {
  return __builtin_amdgcn_make_buffer_rsrc((void*)p, (short)0, (int)bytes, 0x00020000);
}
#define st128(v, r, vo, so, aux) __builtin_amdgcn_raw_buffer_store_b128((v), (r), (vo) + (so), 0, (aux))
#define st96(v, r, vo, so, aux) __builtin_amdgcn_raw_buffer_store_b96((v), (r), (vo) + (so), 0, (aux))
__device__ __forceinline__ void waitv0() { asm volatile("s_waitcnt vmcnt(0)" ::: "memory"); }
__device__ __forceinline__ void bar() {
  asm volatile("s_waitcnt lgkmcnt(0)" ::: "memory");
  __builtin_amdgcn_s_barrier();
  asm volatile("" ::: "memory");
}
__device__ __forceinline__ u32 ld_sys(const char* p) {
  return __hip_atomic_load((const u32*)p, __ATOMIC_RELAXED, __HIP_MEMORY_SCOPE_SYSTEM);
}
__device__ __forceinline__ u32 ld_agt(const char* p) {
  return __hip_atomic_load((const u32*)p, __ATOMIC_RELAXED, __HIP_MEMORY_SCOPE_AGENT);
}
__device__ __forceinline__ void st_sys(char* p, u32 v) {
  __hip_atomic_store((u32*)p, v, __ATOMIC_RELAXED, __HIP_MEMORY_SCOPE_SYSTEM);
}
__device__ __forceinline__ void st_agt(char* p, u32 v) {
  __hip_atomic_store((u32*)p, v, __ATOMIC_RELAXED, __HIP_MEMORY_SCOPE_AGENT);
}

#define STAMP(slot) do {} while (0)

__device__ __forceinline__ bool wave_wait(const char* me, u32 off, bool act, u32 call) {
  bool ok = !act || ld_sys(me + off) >= call;
  if (__all(ok)) return true;
  const u64 t0 = __builtin_amdgcn_s_memrealtime();
  for (;;) {
    __builtin_amdgcn_s_sleep(6);
    if (!ok) ok = ld_sys(me + off) >= call;
    if (__all(ok)) return true;
    if (ld_agt(me + F_TMO) == call || __builtin_amdgcn_s_memrealtime() - t0 > DEADLINE) {
      st_agt((char*)me + F_TMO, call);
      return false;
    }
  }
}

__device__ __forceinline__ bool wg_wait(char* smem, const char* me, u32 off, bool act, u32 call, int wave) {
  volatile LDS_AS u32* word = (volatile LDS_AS u32*)((LDS_AS char*)smem + LDS_CTL);
  if (wave == 0) {
    bool r = wave_wait(me, off, act, call);
    r = r && ld_agt(me + F_TMO) != call;
    __builtin_amdgcn_fence(__ATOMIC_ACQUIRE, "");
    waitv0();
    *word = r ? 1u : 2u;
  }
  bar();
  const u32 r = __builtin_amdgcn_readfirstlane(*word);
  bar();
  return r == 1u;
}

__device__ __forceinline__ u32 perm(u32 hi, u32 lo, u32 sel) { return __builtin_amdgcn_perm(hi, lo, sel); }

__device__ __forceinline__ void gemm_tile(char* smem, rsrc_t rws, int rank, u32 par, u32 g, u32 cg_self, u32 c,
                                          int dskip, int nk, int wave, int lane, i32x4 (&acc)[4][4]) {
  const u32 va = (u32)wave * 2048u + (u32)lane * 16u;
  const u32 vb = (u32)wave * 4096u + (u32)lane * 16u;
  auto issue = [&](int kb, u32 buf) {
    const int qi = kb >> 2;
    const u32 kc = kb & 3;
    const int d = qi + (qi >= dskip ? 1 : 0);
    const u32 q = (u32)(rank + d) & 7u;
    const u32 sa = (d == 0 ? par + P_EX : D_IX + q * (2 * MiB)) + (g * 4 + kc) * XBLK;
    const u32 sb = (d == 0 ? par + P_EW + cg_self * (4 * WBLK) : D_IW + (q * 8 + c) * (4 * WBLK)) + kc * WBLK;
    LDS_AS char* la = (LDS_AS char*)smem + buf * STAGE + wave * 2048;
    LDS_AS char* lb = (LDS_AS char*)smem + buf * STAGE + XBLK + wave * 4096;
    __builtin_amdgcn_raw_ptr_buffer_load_lds(rws, la, 16, va, sa, 0, 0);
    __builtin_amdgcn_raw_ptr_buffer_load_lds(rws, la, 16, va, sa, 1024, 0);
    __builtin_amdgcn_raw_ptr_buffer_load_lds(rws, lb, 16, vb, sb, 0, 0);
    __builtin_amdgcn_raw_ptr_buffer_load_lds(rws, lb, 16, vb, sb, 1024, 0);
    __builtin_amdgcn_raw_ptr_buffer_load_lds(rws, lb, 16, vb, sb, 2048, 0);
    __builtin_amdgcn_raw_ptr_buffer_load_lds(rws, lb, 16, vb, sb, 3072, 0);
  };
#pragma unroll
  for (int m = 0; m < 4; ++m)
#pragma unroll
    for (int t = 0; t < 4; ++t) acc[m][t] = (i32x4){0, 0, 0, 0};
  const u32 ra = (u32)(wave >> 2) * 4096u + (u32)lane * 16u;
  const u32 rb = XBLK + (u32)(wave & 3) * 4096u + (u32)lane * 16u;
  issue(0, 0);
  issue(1, 1);
  u32 buf = 0;
  for (int kb = 0; kb < nk; ++kb) {
    if (kb + 1 < nk) asm volatile("s_waitcnt vmcnt(6)" ::: "memory");
    else asm volatile("s_waitcnt vmcnt(0)" ::: "memory");
    bar();
    const u32 nb = buf == 0 ? 2u : buf - 1;
    if (kb + 2 < nk) issue(kb + 2, nb);
    const char* pa = smem + buf * STAGE + ra;
    const char* pb = smem + buf * STAGE + rb;
#pragma unroll
    for (int ks = 0; ks < 2; ++ks) {
      i32x4 a[4], b[4];
#pragma unroll
      for (int m = 0; m < 4; ++m) a[m] = *(const i32x4*)(pa + ks * 8192 + m * 1024);
#pragma unroll
      for (int t = 0; t < 4; ++t) b[t] = *(const i32x4*)(pb + ks * 16384 + t * 1024);
#pragma unroll
      for (int m = 0; m < 4; ++m)
#pragma unroll
        for (int t = 0; t < 4; ++t)
          acc[m][t] = __builtin_amdgcn_mfma_i32_16x16x64_i8(a[m], b[t], acc[m][t], 0, 0, 0);
    }
    buf = buf == 2 ? 0u : buf + 1;
  }
  bar();
}

__device__ __forceinline__ u32 import_id(u32 o, u32 q) {
  if (o < 4) return 64 + o;
  if (o >= 18 && o < 46) return 64 + 4 + (o - 18);
  u32 s, e;
  if (o < 18) { s = 0; e = o - 4; } else { s = 1 + (o - 46) / 14; e = (o - 46) % 14; }
  u32 p = e >> 1;
  p += (p >= q) ? 1 : 0;
  return (4 * p + s) * 2 + (e & 1);
}

__global__ void __launch_bounds__(NTHR, 2) gemm_rs_kernel(Params P) {
  __shared__ __attribute__((aligned(16))) char smem[LDS_BYTES];
  const int tid = threadIdx.x, lane = tid & 63;
  const int wave = __builtin_amdgcn_readfirstlane(tid >> 6);
  const u32 b = blockIdx.x;
  const u32 rank = (u32)P.rank & 7u;
  char* const me = P.ws[rank];
  const rsrc_t rme = mk_rsrc(me, P.ws_bytes);
  const rsrc_t rout = mk_rsrc(P.out, OUT_BYTES);
  volatile LDS_AS u32* ctl = (volatile LDS_AS u32*)((LDS_AS char*)smem + LDS_CTL);
  STAMP(0);

  const bool hasx = b >= NSTR && b < NSTR + 64;
  u32 call;
  {
    const rsrc_t rw = mk_rsrc(P.w, (u32)KSL * NCOL);
    const rsrc_t rx = mk_rsrc(P.x, 4096u * KSL);
    const u32 cg = b >> 2, kc = b & 3;
    u32 d[16];
    const u32 vw = (kc * 128 + (wave >> 2) * 64 + (lane >> 4) * 16) * NCOL + cg * 256 + (wave & 3) * 64 + (lane & 15) * 4;
#pragma unroll
    for (int r = 0; r < 16; ++r) d[r] = __builtin_amdgcn_raw_buffer_load_b32(rw, vw, r * NCOL, 0);
    u32x4 xv[4];
    const u32 xp = b - NSTR, xg = xp >> 1, xh = xp & 1;
    if (hasx) {
      const u32 vx = (xg * 128 + (wave & 1) * 64 + (lane & 15)) * KSL + (xh * 2 + (wave >> 2)) * 128 + ((wave >> 1) & 1) * 64 + (lane >> 4) * 16;
#pragma unroll
      for (int u = 0; u < 4; ++u) xv[u] = __builtin_amdgcn_raw_buffer_load_b128(rx, vx, u * 16 * KSL, 0);
    }
    if (tid == 0) {
      const u32 c0 = ld_agt(me + F_CNT + b * 128) + 1;
      st_agt(me + F_CNT + b * 128, c0);
      ctl[4] = c0;
    }
    if (tid < 16) ctl[16 + tid] = 0;
    bar();
    call = __builtin_amdgcn_readfirstlane(ctl[4]);
    const u32 par = D_PAR + (call & 1) * PAR_BYTES;
    u32x4 o[4];
#pragma unroll
    for (int r4 = 0; r4 < 4; ++r4) {
      const u32 t0 = perm(d[4 * r4 + 1], d[4 * r4], 0x05010400u), t1 = perm(d[4 * r4 + 1], d[4 * r4], 0x07030602u);
      const u32 t2 = perm(d[4 * r4 + 3], d[4 * r4 + 2], 0x05010400u), t3 = perm(d[4 * r4 + 3], d[4 * r4 + 2], 0x07030602u);
      o[0][r4] = perm(t2, t0, 0x05040100u);
      o[1][r4] = perm(t2, t0, 0x07060302u);
      o[2][r4] = perm(t3, t1, 0x05040100u);
      o[3][r4] = perm(t3, t1, 0x07060302u);
    }
    const u32 vs = wave * 4096 + lane * 16;
#pragma unroll
    for (int t = 0; t < 4; ++t)
      st128(o[t], rme, vs + t * 1024, par + P_EW + b * WBLK, AUX_SYS);
    if (hasx) {
#pragma unroll
      for (int u = 0; u < 4; ++u)
        st128(xv[u], rme, vs + u * 1024, par + P_EX + xp * (2 * XBLK), AUX_SYS);
    }
    waitv0();
    bar();
    if (wave == 0) {
      if (lane == 0) st_sys(P.ws[(cg >> 3) & 7] + F_S + (rank * 96 + 64 + (cg & 7) * 4 + kc) * 128, call);
      if (lane == 1) st_sys(me + F_LW + b * 128, call);
      if (hasx && lane >= 8 && lane < 16) st_sys(P.ws[lane & 7] + F_S + (rank * 96 + xp) * 128, call);
    }
  }
  const u32 par = D_PAR + (call & 1) * PAR_BYTES;
  STAMP(1);

  if (b < NSTR) {
    const u32 q = (rank + 1 + b % 7) & 7u, idx = b / 7;
    const rsrc_t rq = mk_rsrc(P.ws[q], P.ws_bytes);
    bool good = wg_wait(smem, me, F_S + (q * 96 + import_id(idx + WPL * (lane < 11 ? lane : 0), q)) * 128, lane < 11, call, wave);
    STAMP(2);
    if (good) {
      const u32 vs = wave * 4096 + lane * 16;
      u32 prev = 0;
      for (u32 n = 0; n < 12; ++n) {
        u32 id = 0, so = 0, dso = 0;
        if (n < 11) {
          id = __builtin_amdgcn_readfirstlane(import_id(idx + WPL * n, q));
          if (id < 64) { so = par + P_EX + id * (2 * XBLK); dso = D_IX + q * (2 * MiB) + id * (2 * XBLK); }
          else { so = par + P_EW + (rank * 32 + (id - 64)) * WBLK; dso = D_IW + (q * 32 + (id - 64)) * WBLK; }
        }
#pragma unroll
        for (int it = 0; it < 4; ++it) {
          u32x4 v;
          if (n < 11) v = __builtin_amdgcn_raw_buffer_load_b128(rq, vs + it * 1024, so, AUX_SYS);
          waitv0();
          if (it == 0 && n > 0) {
            u32 old = 0;
            if (lane == 0) old = __hip_atomic_fetch_add((u32*)(smem + LDS_CTL) + 16 + (n - 1), 1u, __ATOMIC_RELAXED, __HIP_MEMORY_SCOPE_WORKGROUP);
            old = __builtin_amdgcn_readfirstlane(old);
            if (old == 7 && lane == 0) st_agt(me + F_I + (q * 96 + prev) * 128, call);
          }
          if (n == 11) break;
          st128(v, rme, vs + it * 1024, dso, AUX_AGT);
        }
        prev = id;
      }
    }
    STAMP(3);
    if (wave < FIN_WAVES) {
      constexpr u32 V = WPL * FIN_WAVES;
      const float scale = P.sx[0] * P.sw[0];
      u32 ready = 0, rawm = 0;
      const u32 vo = (u32)(lane >> 4) * 4 * (NCOL * 4) + (lane & 15) * 16;
      for (u32 z = idx * FIN_WAVES + wave; z < 1024; z += V) {
        const u32 i = z >> 5, w8 = (z >> 2) & 7, mi = z & 3;
        if (good && !((ready >> i) & 1)) {
          const u64 t0 = __builtin_amdgcn_s_memrealtime();
          for (;;) {
            const u32 l = lane & 31;
            const u64 of = __hip_atomic_load((const u64*)(me + F_O + (q * 32 + l) * 128), __ATOMIC_RELAXED, __HIP_MEMORY_SCOPE_SYSTEM);
            const u32 pf = ld_agt(me + F_P + (q * 32 + l) * 128);
            rawm = (u32)of;
            ready = (u32)__ballot((u32)(of >> 32) >= call && pf >= call);
            const bool tmo = ld_agt(me + F_TMO) == call;
            if (!tmo && ((ready >> i) & 1)) break;
            if (tmo || __builtin_amdgcn_s_memrealtime() - t0 > DEADLINE) {
              st_agt(me + F_TMO, call);
              good = false;
              break;
            }
            __builtin_amdgcn_s_sleep(6);
          }
          asm volatile("" ::: "memory");
        }
        const u32 s = i >> 3, c = i & 7;
        const u32 oo = ((s * 128 + (w8 >> 2) * 64 + mi * 16) * NCOL + q * 2048 + c * 256 + (w8 & 3) * 64) * 4;
        u32x4 y[4];
        if (good) {
          const u32 sub = ((rank * 4 + s) * 8 + c) * 32 + (z & 31);
          const bool raw = (__builtin_amdgcn_readlane(rawm, i) >> (z & 31)) & 1;
          u32x4 pt[4];
          i32x4 v[4];
          if (!raw) {
            const u32 so = par + P_PK + sub * SUB;
            const u32x4 l0 = __builtin_amdgcn_raw_buffer_load_b128(rq, lane * 16, so, AUX_SYS);
            const u32x4 l1 = __builtin_amdgcn_raw_buffer_load_b128(rq, lane * 16 + 1024, so, AUX_SYS);
            const u32x3 h = __builtin_amdgcn_raw_buffer_load_b96(rq, lane * 12 + 2048, so, AUX_SYS);
#pragma unroll
            for (int r = 0; r < 4; ++r) pt[r] = __builtin_amdgcn_raw_buffer_load_b128(rout, vo + r * (NCOL * 4), oo, AUX_AGT);
            u32 hb[4];
            hb[0] = h[0]; hb[1] = h[1]; hb[2] = h[2];
            hb[3] = ((h[0] >> 6) & 0x03030303u) | ((h[1] >> 4) & 0x0C0C0C0Cu) | ((h[2] >> 2) & 0x30303030u);
#pragma unroll
            for (int r = 0; r < 4; ++r) {
              const u32 lo01 = r < 2 ? l0[2 * r] : l1[2 * r - 4], lo23 = r < 2 ? l0[2 * r + 1] : l1[2 * r - 3];
              v[r][0] = (int)perm((u32)__builtin_amdgcn_sbfe((int)hb[r], 0, 6), lo01, 0x05040100u);
              v[r][1] = (int)perm((u32)__builtin_amdgcn_sbfe((int)hb[r], 8, 6), lo01, 0x05040302u);
              v[r][2] = (int)perm((u32)__builtin_amdgcn_sbfe((int)hb[r], 16, 6), lo23, 0x05040100u);
              v[r][3] = (int)perm((u32)__builtin_amdgcn_sbfe((int)hb[r], 24, 6), lo23, 0x05040302u);
            }
          } else {
            const u32 so = par + P_RAW + sub * 4096;
#pragma unroll
            for (int r = 0; r < 4; ++r) v[r] = __builtin_bit_cast(i32x4, __builtin_amdgcn_raw_buffer_load_b128(rq, lane * 16 + r * 1024, so, AUX_SYS));
#pragma unroll
            for (int r = 0; r < 4; ++r) pt[r] = __builtin_amdgcn_raw_buffer_load_b128(rout, vo + r * (NCOL * 4), oo, AUX_AGT);
          }
#pragma unroll
          for (int r = 0; r < 4; ++r)
#pragma unroll
            for (int t = 0; t < 4; ++t) y[r][t] = __builtin_bit_cast(u32, (float)(v[r][t] + (int)pt[r][t]) * scale);
        } else {
#pragma unroll
          for (int r = 0; r < 4; ++r) y[r] = (u32x4){0x7FC00000u, 0x7FC00000u, 0x7FC00000u, 0x7FC00000u};
        }
#pragma unroll
        for (int r = 0; r < 4; ++r) st128(y[r], rout, vo + r * (NCOL * 4), oo, AUX_AGT);
      }
    }
    STAMP(4);
    return;
  }

  const u32 gi = b - NSTR;
  const u32 vo = ((u32)(wave >> 2) * 64 + (lane >> 4) * 4) * (NCOL * 4) + ((wave & 3) * 64 + (lane & 15) * 4) * 4;
  i32x4 acc[4][4];
  for (int job = 0; job < 4; ++job) {
    const bool partial = job < 2;
    u32 t;
    if (job == 0) t = gi;
    else if (job == 1) { if (gi < NGEM - 24) continue; t = gi + 24; }
    else if (job == 2) { if (gi >= 192) break; t = gi; }
    else { if (gi >= 64) break; t = 192 + gi; }
    u32 g, c, cg_self, foff, s, dst;
    int dskip, nk;
    bool act;
    if (partial) {
      c = t & 7;
      const u32 r = t >> 3;
      s = r / 7;
      dst = (rank + 1 + r % 7) & 7u;
      g = rank * 4 + s;
      cg_self = dst * 8 + c;
      dskip = 8; nk = 4;
      act = lane < 6;
      foff = lane < 4 ? F_LW + (cg_self * 4 + lane) * 128 : F_S + (rank * 96 + g * 2 + (lane & 1)) * 128;
    } else {
      c = t & 7;
      s = t >> 6;
      dst = (rank + 1 + ((t >> 3) & 7)) & 7u;
      g = dst * 4 + s;
      cg_self = rank * 8 + c;
      dskip = dst == rank ? 8 : (int)((dst - rank) & 7u);
      nk = dst == rank ? 32 : 28;
      const u32 q = lane / 6, e = lane % 6;
      act = lane < 48 && (q != dst || dst == rank);
      const u32 id = e < 4 ? 64 + c * 4 + e : g * 2 + (e & 1);
      foff = (q == rank ? F_S : F_I) + ((q & 7) * 96 + id) * 128;
    }
    const bool good = wg_wait(smem, me, foff, act, call, wave);
    STAMP(2 + job * 3);
    if (good) gemm_tile(smem, rme, (int)rank, par, g, cg_self, c, dskip, nk, wave, lane, acc);
    STAMP(3 + job * 3);
    if (partial) {
      if (good) {
        const u32 oo = (s * 128 * NCOL + dst * 2048 + c * 256) * 4;
#pragma unroll
        for (int m = 0; m < 4; ++m)
#pragma unroll
          for (int r = 0; r < 4; ++r) {
            const u32x4 y = {(u32)acc[m][0][r], (u32)acc[m][1][r], (u32)acc[m][2][r], (u32)acc[m][3][r]};
            st128(y, rout, vo + (m * 16 + r) * (NCOL * 4), oo, AUX_AGT);
          }
        waitv0();
        bar();
        if (tid == 0) st_agt(me + F_P + (dst * 32 + s * 8 + c) * 128, call);
      }
    } else if (dst == rank) {
      const float scale = P.sx[0] * P.sw[0];
      const u32 oo = (s * 128 * NCOL + rank * 2048 + c * 256) * 4;
#pragma unroll
      for (int m = 0; m < 4; ++m)
#pragma unroll
        for (int r = 0; r < 4; ++r) {
          u32x4 y;
#pragma unroll
          for (int tt = 0; tt < 4; ++tt) y[tt] = good ? __builtin_bit_cast(u32, (float)acc[m][tt][r] * scale) : 0x7FC00000u;
          st128(y, rout, vo + (m * 16 + r) * (NCOL * 4), oo, AUX_AGT);
        }
    } else if (good) {
      if (tid == 0) ctl[8] = 0;
      bar();
      const u32 sub0 = (g * 8 + c) * 32 + wave * 4;
      u32 rawm = 0;
#pragma unroll
      for (int m = 0; m < 4; ++m) {
        u32 chk = 0;
#pragma unroll
        for (int tt = 0; tt < 4; ++tt)
#pragma unroll
          for (int r = 0; r < 4; ++r) chk |= (u32)acc[m][tt][r] + 0x200000u;
        if (__any(chk >= 0x400000u)) {
          rawm |= 1u << (wave * 4 + m);
#pragma unroll
          for (int r = 0; r < 4; ++r) {
            const u32x4 y = {(u32)acc[m][0][r], (u32)acc[m][1][r], (u32)acc[m][2][r], (u32)acc[m][3][r]};
            st128(y, rme, lane * 16 + r * 1024, par + P_RAW + (sub0 + m) * 4096, AUX_SYS);
          }
        } else {
          u32x4 l0, l1;
          u32 hb[4];
#pragma unroll
          for (int r = 0; r < 4; ++r) {
            const u32 v0 = acc[m][0][r], v1 = acc[m][1][r], v2 = acc[m][2][r], v3 = acc[m][3][r];
            const u32 lo01 = perm(v1, v0, 0x05040100u), lo23 = perm(v3, v2, 0x05040100u);
            if (r < 2) { l0[2 * r] = lo01; l0[2 * r + 1] = lo23; } else { l1[2 * r - 4] = lo01; l1[2 * r - 3] = lo23; }
            hb[r] = perm(perm(v3, v2, 0x0c0c0602u), perm(v1, v0, 0x0c0c0602u), 0x05040100u);
          }
          u32x3 h;
          h[0] = (hb[0] & 0x3F3F3F3Fu) | ((hb[3] << 6) & 0xC0C0C0C0u);
          h[1] = (hb[1] & 0x3F3F3F3Fu) | ((hb[3] << 4) & 0xC0C0C0C0u);
          h[2] = (hb[2] & 0x3F3F3F3Fu) | ((hb[3] << 2) & 0xC0C0C0C0u);
          const u32 so = par + P_PK + (sub0 + m) * SUB;
          st128(l0, rme, lane * 16, so, AUX_SYS);
          st128(l1, rme, lane * 16 + 1024, so, AUX_SYS);
          st96(h, rme, lane * 12 + 2048, so, AUX_SYS);
        }
      }
      if (rawm && lane == 0) __hip_atomic_fetch_or((u32*)(smem + LDS_CTL) + 8, rawm, __ATOMIC_RELAXED, __HIP_MEMORY_SCOPE_WORKGROUP);
      waitv0();
      bar();
      if (tid == 0) {
        const u64 f = ((u64)call << 32) | ctl[8];
        __hip_atomic_store((u64*)(P.ws[dst] + F_O + (rank * 32 + s * 8 + c) * 128), f, __ATOMIC_RELAXED, __HIP_MEMORY_SCOPE_SYSTEM);
      }
    }
    STAMP(4 + job * 3);
  }
}

}

extern "C" void kernel_launch(int rank, int nranks, ncclComm_t comm,
                              hipStream_t stream,
                              void* const* d_in, const int* in_sizes, int n_in,
                              void* d_out, int out_size,
                              const int* in_dtypes, int out_dtype,
                              void* const* d_out_all,
                              void* const* d_ws_all, size_t ws_bytes) {
  if (nranks != NR || (unsigned)rank >= (unsigned)NR || n_in < 4 || ws_bytes < (size_t)WS_NEED + MiB ||
      (size_t)out_size * 4 != OUT_BYTES || in_sizes[0] != 4096 * KSL || in_sizes[1] != KSL * NCOL) {
    fprintf(stderr, "kernel_launch: unexpected shapes\n");
    return;
  }
  Params p;
  p.x = (const signed char*)d_in[0];
  p.w = (const signed char*)d_in[1];
  p.sx = (const float*)d_in[2];
  p.sw = (const float*)d_in[3];
  p.out = (float*)d_out;
  for (int i = 0; i < NR; ++i) p.ws[i] = (char*)d_ws_all[i];
  p.ws_bytes = WS_NEED + MiB;
  p.rank = rank;
  gemm_rs_kernel<<<NWG, NTHR, 0, stream>>>(p);
  HIP_CHECK(hipGetLastError());
}
```

```cpp
#include <hip/hip_runtime.h>
#include <rccl/rccl.h>
#include <cstdio>

#define HIP_CHECK(expr) do { hipError_t _e = (expr); if (_e != hipSuccess) { \
    fprintf(stderr, "HIP error %s at %s:%d\n", hipGetErrorString(_e), __FILE__, __LINE__); \
    return; } } while (0)

typedef unsigned u32;
typedef unsigned long long u64;
typedef u32 u32x4 __attribute__((ext_vector_type(4)));
typedef u32 u32x2 __attribute__((ext_vector_type(2)));
typedef int i32x4 __attribute__((ext_vector_type(4)));
typedef __amdgpu_buffer_rsrc_t rsrc_t;
#define LDS_AS __attribute__((address_space(3)))

namespace {

constexpr int NR = 8;
constexpr int KSL = 512;
constexpr int NCOL = 16384;
constexpr int NWG = 256, NTHR = 512;
constexpr int WPL = 8;
constexpr int NSTR = 7 * WPL;
constexpr int NGEM = NWG - NSTR;
#define EXPORT_DELAY 24
#define P1W 8
#define LWIN 6

constexpr u32 KiB = 1024u, MiB = 1024u * 1024u;
constexpr u32 F_CNT = 0;
constexpr u32 F_TMO = 32 * KiB;
constexpr u32 F_S = 64 * KiB;
constexpr u32 F_I = 160 * KiB;
constexpr u32 F_LW = 256 * KiB;
constexpr u32 F_O = 288 * KiB;
constexpr u32 F_P = 320 * KiB;
constexpr u32 F_T1 = 352 * KiB;
constexpr u32 F_T2 = 353 * KiB;
constexpr u32 XBLK = 16 * KiB, WBLK = 32 * KiB;
constexpr u32 D_IX = 1 * MiB;
constexpr u32 D_PAR = D_IX + 16 * MiB;
constexpr u32 P_EX = 0;
constexpr u32 P_EW = 2 * MiB;
constexpr u32 SUB = 10752;
constexpr u32 P_PK = 10 * MiB;
constexpr u32 P_RAW = 32 * MiB;
constexpr u32 P_IW = 64 * MiB;
constexpr u32 PAR_BYTES = 72 * MiB;
constexpr u32 WS_NEED = D_PAR + 2 * PAR_BYTES;
constexpr u32 OUT_BYTES = 512u * NCOL * 4u;

constexpr u32 STAGE = XBLK + WBLK;
constexpr u32 LDS_CTL = 3 * STAGE;
constexpr u32 LDS_BYTES = LDS_CTL + 512;
constexpr u32 NSLOT = 13, SLOTB = 11264;
constexpr int C_WAIT = 0, C_RAW = 8, C_ELECT = 16, C_ITEM = 17, C_END = 18, C_READY = 19, C_FAIL = 20,
              C_RAWM = 24, C_ASG = 64, C_FULL = 80, C_DONE = 96;

constexpr u64 DEADLINE = 40000000ull;
constexpr int AUX_SYS = 17, AUX_AGT = 16;
constexpr u32 OOB = 0x70000000u;

struct Params {
  const signed char* x;
  const signed char* w;
  const float* sx;
  const float* sw;
  float* out;
  char* me;
  char* ws[NR];
  u32 ws_bytes;
  int rank;
};

__device__ __forceinline__ rsrc_t mk_rsrc(const void* p, u32 bytes) {
  return __builtin_amdgcn_make_buffer_rsrc((void*)p, (short)0, (int)bytes, 0x00020000);
}
#define st128(v, r, vo, so, aux) __builtin_amdgcn_raw_buffer_store_b128((v), (r), (vo) + (so), 0, (aux))
__device__ __forceinline__ void waitv0() { asm volatile("s_waitcnt vmcnt(0)" ::: "memory"); }
__device__ __forceinline__ void bar() {
  asm volatile("s_waitcnt lgkmcnt(0)" ::: "memory");
  __builtin_amdgcn_s_barrier();
  asm volatile("" ::: "memory");
}
__device__ __forceinline__ u32 ld_sys(const char* p) {
  return __hip_atomic_load((const u32*)p, __ATOMIC_RELAXED, __HIP_MEMORY_SCOPE_SYSTEM);
}
__device__ __forceinline__ u32 ld_agt(const char* p) {
  return __hip_atomic_load((const u32*)p, __ATOMIC_RELAXED, __HIP_MEMORY_SCOPE_AGENT);
}
__device__ __forceinline__ void st_sys(char* p, u32 v) {
  __hip_atomic_store((u32*)p, v, __ATOMIC_RELAXED, __HIP_MEMORY_SCOPE_SYSTEM);
}
__device__ __forceinline__ void st_agt(char* p, u32 v) {
  __hip_atomic_store((u32*)p, v, __ATOMIC_RELAXED, __HIP_MEMORY_SCOPE_AGENT);
}

#define STAMP(slot) do {} while (0)

__device__ __forceinline__ bool wave_wait(const char* me, u32 off, bool act, u32 tgt, u32 call) {
  bool ok = !act || ld_sys(me + off) >= tgt;
  if (__all(ok)) return true;
  const u64 t0 = __builtin_amdgcn_s_memrealtime();
  for (;;) {
    __builtin_amdgcn_s_sleep(6);
    if (!ok) ok = ld_sys(me + off) >= tgt;
    if (__all(ok)) return true;
    if (ld_agt(me + F_TMO) == call || __builtin_amdgcn_s_memrealtime() - t0 > DEADLINE) {
      st_agt((char*)me + F_TMO, call);
      return false;
    }
  }
}

__device__ __forceinline__ bool wg_wait(char* smem, const char* me, u32 off, bool act, u32 tgt, u32 call, int wave) {
  volatile LDS_AS u32* word = (volatile LDS_AS u32*)((LDS_AS char*)smem + LDS_CTL);
  if (wave == 0) {
    bool r = wave_wait(me, off, act, tgt, call);
    r = r && ld_agt(me + F_TMO) != call;
    __builtin_amdgcn_fence(__ATOMIC_ACQUIRE, "");
    waitv0();
    *word = r ? 1u : 2u;
  }
  bar();
  const u32 r = __builtin_amdgcn_readfirstlane(*word);
  bar();
  return r == 1u;
}

__device__ __forceinline__ u32 ffbl(u32 x) {
  u32 r;
  asm("v_ffbl_b32 %0, %1" : "=v"(r) : "v"(x));
  return r;
}
__device__ __forceinline__ u32 perm(u32 hi, u32 lo, u32 sel) { return __builtin_amdgcn_perm(hi, lo, sel); }

__device__ __forceinline__ void gemm_tile(char* smem, rsrc_t rws, int rank, u32 par, u32 g, u32 cg_self, u32 c,
                                          int dskip, int nk, int wave, int lane, i32x4 (&acc)[4][4]) {
  const u32 va = (u32)wave * 2048u + (u32)lane * 16u;
  const u32 vb = (u32)wave * 4096u + (u32)lane * 16u;
  u32 sa, sb;
  auto locate = [&](int kb) {
    const int qi = kb >> 2;
    const u32 kc = kb & 3;
    const int d = qi + (qi >= dskip ? 1 : 0);
    const u32 q = (u32)(rank + d) & 7u;
    sa = (d == 0 ? par + P_EX : D_IX + q * (2 * MiB)) + (g * 4 + kc) * XBLK;
    sb = (d == 0 ? par + P_EW + cg_self * (4 * WBLK) : par + P_IW + (q * 8 + c) * (4 * WBLK)) + kc * WBLK;
    if (kb >= nk) sa = sb = OOB;
  };
  auto dma = [&](int n, u32 buf) {
    LDS_AS char* la = (LDS_AS char*)smem + buf * STAGE + wave * 2048;
    LDS_AS char* lb = (LDS_AS char*)smem + buf * STAGE + XBLK + wave * 4096;
    if (n == 0) __builtin_amdgcn_raw_ptr_buffer_load_lds(rws, la, 16, va, sa, 0, 0);
    if (n == 1) __builtin_amdgcn_raw_ptr_buffer_load_lds(rws, la, 16, va, sa, 1024, 0);
    if (n == 2) __builtin_amdgcn_raw_ptr_buffer_load_lds(rws, lb, 16, vb, sb, 0, 0);
    if (n == 3) __builtin_amdgcn_raw_ptr_buffer_load_lds(rws, lb, 16, vb, sb, 1024, 0);
    if (n == 4) __builtin_amdgcn_raw_ptr_buffer_load_lds(rws, lb, 16, vb, sb, 2048, 0);
    if (n == 5) __builtin_amdgcn_raw_ptr_buffer_load_lds(rws, lb, 16, vb, sb, 3072, 0);
  };
#pragma unroll
  for (int m = 0; m < 4; ++m)
#pragma unroll
    for (int t = 0; t < 4; ++t) acc[m][t] = (i32x4){0, 0, 0, 0};
  const u32 ra = (u32)(wave >> 2) * 4096u + (u32)lane * 16u;
  const u32 rb = XBLK + (u32)(wave & 3) * 4096u + (u32)lane * 16u;
  i32x4 f0[8], f1[8];
  auto rd = [&](i32x4 (&f)[8], int n, u32 buf, int ks) {
    const int j = n == 0 ? 0 : (n < 5 ? n + 3 : n - 4);
    const char* p = smem + buf * STAGE + (j < 4 ? ra + ks * 8192 + j * 1024 : rb + ks * 16384 + (j - 4) * 1024);
    f[j] = *(const i32x4*)p;
  };
  auto half = [&](i32x4 (&fc)[8], i32x4 (&fn)[8], u32 rbuf, int rks, int d0, u32 dbuf) {
#pragma unroll
    for (int i = 0; i < 16; ++i) {
      acc[i >> 2][i & 3] = __builtin_amdgcn_mfma_i32_16x16x64_i8(fc[i >> 2], fc[4 + (i & 3)], acc[i >> 2][i & 3], 0, 0, 0);
      if (i < 8) rd(fn, i, rbuf, rks);
      if (i == 8 || i == 10 || i == 12) dma(d0 + ((i - 8) >> 1), dbuf);
      __builtin_amdgcn_sched_barrier(0);
    }
  };
  locate(0);
#pragma unroll
  for (int n = 0; n < 6; ++n) dma(n, 0);
  locate(1);
#pragma unroll
  for (int n = 0; n < 6; ++n) dma(n, 1);
  locate(2);
#pragma unroll
  for (int n = 0; n < 3; ++n) dma(n, 2);
  asm volatile("s_waitcnt vmcnt(9)" ::: "memory");
  bar();
#pragma unroll
  for (int n = 0; n < 8; ++n) rd(f0, n, 0, 0);
  u32 buf = 0;
  for (int kb = 0; kb < nk; ++kb) {
    const u32 b1 = buf == 2 ? 0u : buf + 1, b2 = buf == 0 ? 2u : buf - 1;
    __builtin_amdgcn_sched_barrier(0);
    half(f0, f1, buf, 1, 3, b2);
    locate(kb + 3);
    asm volatile("s_waitcnt vmcnt(6)" ::: "memory");
    bar();
    __builtin_amdgcn_sched_barrier(0);
    half(f1, f0, b1, 0, 0, buf);
    buf = b1;
  }
  waitv0();
}

constexpr u32 NPULL = 84;
__device__ __forceinline__ u32 import_id(u32 o, u32 q) {
  if (o >= 14 && o < 26) return 64 + 4 + (o - 14);
  if (o >= 40 && o < 56) return 64 + 16 + (o - 40);
  u32 s, e;
  if (o < 14) { s = 0; e = o; }
  else if (o < 40) { s = 1; e = o - 26; }
  else if (o < 70) { s = 2; e = o - 56; }
  else { s = 3; e = o - 70; }
  u32 p = e >> 1;
  p += (p >= q) ? 1 : 0;
  return (4 * p + s) * 2 + (e & 1);
}
__device__ __forceinline__ void tile_sc(u32 i, u32& s, u32& c) {
  if (i < 8) { s = i >> 2; c = i & 3; }
  else if (i < 16) { s = i & 1; c = 4 + ((i - 8) >> 1); }
  else { s = i >> 3; c = i & 7; }
}
__device__ __forceinline__ u32 draw(char* p) {
  u32 t;
  u64 save;
  asm volatile("s_mov_b64 %1, exec\n\ts_mov_b64 exec, 1\n\tglobal_atomic_add %0, %2, %3, off sc0\n\ts_mov_b64 exec, %1"
               : "=&v"(t), "=&s"(save) : "v"(p), "v"(1u) : "memory");
  return t;
}
__device__ __forceinline__ u32 take(u32 t) {
  asm volatile("s_nop 0" : "+v"(t));
  return __builtin_amdgcn_readfirstlane(t);
}
__device__ __forceinline__ u32 lds_add(volatile LDS_AS u32* p, u32 x) {
  u32 old = 0;
  if ((threadIdx.x & 63) == 0) old = __hip_atomic_fetch_add((LDS_AS u32*)p, x, __ATOMIC_RELAXED, __HIP_MEMORY_SCOPE_WORKGROUP);
  return __builtin_amdgcn_readfirstlane(old);
}
__device__ __forceinline__ void bump(char* p) {
  u64 save;
  asm volatile("s_mov_b64 %0, exec\n\ts_mov_b64 exec, 1\n\tglobal_atomic_add %1, %2, off\n\ts_mov_b64 exec, %0"
               : "=&s"(save) : "v"(p), "v"(1u) : "memory");
}

__global__ void __launch_bounds__(NTHR, 2) gemm_rs_kernel(Params P) {
  __shared__ __attribute__((aligned(16))) char smem[LDS_BYTES];
  const int tid = threadIdx.x, lane = tid & 63;
  const int wave = __builtin_amdgcn_readfirstlane(tid >> 6);
  const u32 b = blockIdx.x;
  const u32 rank = (u32)P.rank & 7u;
  char* const me = P.me;
  asm volatile("" :: "s"(P.x), "s"(P.w), "s"(me), "s"(P.out), "s"(P.ws_bytes), "s"(P.rank));
  const rsrc_t rme = mk_rsrc(me, P.ws_bytes);
  const rsrc_t rout = mk_rsrc(P.out, OUT_BYTES);
  volatile LDS_AS u32* ctl = (volatile LDS_AS u32*)((LDS_AS char*)smem + LDS_CTL);
  STAMP(0);

  const u32 gi = b - NSTR;
  const u32 cnt = ld_agt(me + F_CNT + b * 128);
  u32 call = 0, par = 0;
  if (b < NSTR) {
    if (tid < 128) ctl[tid] = tid == C_END ? 0xFFFFFFFFu : 0u;
    call = __builtin_amdgcn_readfirstlane(cnt) + 1;
    par = D_PAR + (call & 1) * PAR_BYTES;
    bar();
    if (tid == 0) st_agt(me + F_CNT + b * 128, call);
  }
  for (int e = 0; e < 2 && b >= NSTR; ++e) {
    u32 wp = 0xFFFFu, xp = 0xFFFFu;
    if (gi < 32) { if (e == 0) wp = (gi >> 2) * 32 + (gi & 3); }
    else if (gi < 48) { if (e == 0) xp = ((gi - 32) >> 1) * 8 + (gi & 1); }
    else {
      const u32 u = gi - 48, v = e == 0 ? u : 152 + u;
      if (v < 224) wp = (v / 28) * 32 + 4 + v % 28;
      else if (v < 272) xp = ((v - 224) / 6) * 8 + 2 + (v - 224) % 6;
      if (e == 0) __builtin_amdgcn_s_sleep(EXPORT_DELAY);
    }
    const u32 vs = wave * 4096 + lane * 16;
    if (wp != 0xFFFFu) {
      const u32 cg = wp >> 2, kc = wp & 3, dj = (cg >> 3) & 7;
      u32 d[16];
      const rsrc_t rw = mk_rsrc(P.w, (u32)KSL * NCOL);
      const u32 vw = (kc * 128 + (wave >> 2) * 64 + (lane >> 4) * 16) * NCOL + cg * 256 + (wave & 3) * 64 + (lane & 15) * 4;
#pragma unroll
      for (int r = 0; r < 16; ++r) d[r] = __builtin_amdgcn_raw_buffer_load_b32(rw, vw, r * NCOL, 0);
      char* const peer = P.ws[dj];
      call = __builtin_amdgcn_readfirstlane(cnt) + 1;
      par = D_PAR + (call & 1) * PAR_BYTES;
      const bool push = (cg & 7) == 0 && dj != rank;
      u32x4 o[4];
#pragma unroll
      for (int r4 = 0; r4 < 4; ++r4) {
        const u32 t0 = perm(d[4 * r4 + 1], d[4 * r4], 0x05010400u), t1 = perm(d[4 * r4 + 1], d[4 * r4], 0x07030602u);
        const u32 t2 = perm(d[4 * r4 + 3], d[4 * r4 + 2], 0x05010400u), t3 = perm(d[4 * r4 + 3], d[4 * r4 + 2], 0x07030602u);
        o[0][r4] = perm(t2, t0, 0x05040100u);
        o[1][r4] = perm(t2, t0, 0x07060302u);
        o[2][r4] = perm(t3, t1, 0x05040100u);
        o[3][r4] = perm(t3, t1, 0x07060302u);
      }
      if (push) {
        const rsrc_t rj = mk_rsrc(peer, P.ws_bytes);
#pragma unroll
        for (int t = 0; t < 4; ++t) st128(o[t], rj, vs + t * 1024, par + P_IW + (rank * 32 + kc) * WBLK, AUX_SYS);
      }
#pragma unroll
      for (int t = 0; t < 4; ++t) st128(o[t], rme, vs + t * 1024, par + P_EW + wp * WBLK, AUX_SYS);
      waitv0();
      bar();
      if (tid < 2) {
        char* f = tid == 0 ? peer + (push ? F_I : F_S) + (rank * 96 + 64 + (cg & 7) * 4 + kc) * 128 : me + F_LW + wp * 128;
        st_sys(f, (push && tid == 0) ? call * 8 : call);
      }
    } else if (xp != 0xFFFFu) {
      u32x4 o[4];
      const rsrc_t rx = mk_rsrc(P.x, 4096u * KSL);
      const u32 vx = ((xp >> 1) * 128 + (wave & 1) * 64 + (lane & 15)) * KSL + ((xp & 1) * 2 + (wave >> 2)) * 128 + ((wave >> 1) & 1) * 64 + (lane >> 4) * 16;
#pragma unroll
      for (int u = 0; u < 4; ++u) o[u] = __builtin_amdgcn_raw_buffer_load_b128(rx, vx, u * 16 * KSL, 0);
      char* const peer = P.ws[lane & 7];
      call = __builtin_amdgcn_readfirstlane(cnt) + 1;
      par = D_PAR + (call & 1) * PAR_BYTES;
#pragma unroll
      for (int t = 0; t < 4; ++t) st128(o[t], rme, vs + t * 1024, par + P_EX + xp * (2 * XBLK), AUX_SYS);
      waitv0();
      bar();
      if (tid < 8) st_sys(peer + F_S + (rank * 96 + xp) * 128, call);
    }
    if (e == 0 && tid == 64) st_agt(me + F_CNT + b * 128, call);
  }
  STAMP(1);

  if (b < NSTR) {
    const u32 q = (rank + 1 + b % 7) & 7u, idx = b / 7;
    const rsrc_t rq = mk_rsrc(P.ws[q], P.ws_bytes);
    const u32 fo0 = F_S + (q * 96 + import_id(lane, q)) * 128;
    const u32 fo1 = F_S + (q * 96 + import_id(64 + (lane < NPULL - 64 ? lane : 0), q)) * 128;
    bool good = wave_wait(me, fo0, lane == ((idx * P1W + wave) >> 3), call, call);
    asm volatile("" ::: "memory");
    STAMP(2);
    if (good && wave < P1W) {
      const u32 tbase = (call - 1) * (NPULL * 8) - WPL * P1W;
      u32 k = idx * P1W + wave, prev = 0xFFFFFFFFu;
      bool all = false;
      while (k < NPULL * 8) {
        const u32 tk = draw(me + F_T1 + q * 128);
        const u32 id = __builtin_amdgcn_readfirstlane(import_id(k >> 3, q));
        u32 so, dso;
        if (id < 64) { so = par + P_EX + id * (2 * XBLK); dso = D_IX + q * (2 * MiB) + id * (2 * XBLK); }
        else { so = par + P_EW + (rank * 32 + (id - 64)) * WBLK; dso = par + P_IW + (q * 32 + (id - 64)) * WBLK; }
        const u32 vs = (k & 7) * 4096 + lane * 16;
#pragma unroll
        for (int it = 0; it < 4; ++it) {
          const u32x4 v = __builtin_amdgcn_raw_buffer_load_b128(rq, vs + it * 1024, so, AUX_SYS);
          if (it == 3 && !all) {
            const u32 f0 = ld_sys(me + fo0), f1 = ld_sys(me + fo1);
            all = __all(f0 >= call && f1 >= call);
          }
          waitv0();
          if (it == 0 && prev != 0xFFFFFFFFu) bump(me + F_I + (q * 96 + prev) * 128);
          st128(v, rme, vs + it * 1024, dso, AUX_AGT);
        }
        prev = id;
        k = take(tk) - tbase;
        if (!all && k < NPULL * 8) {
          const u32 o = k >> 3;
          good = wave_wait(me, o < 64 ? fo0 : fo1, lane == (o & 63), call, call);
          asm volatile("" ::: "memory");
          if (!good) break;
        }
      }
      waitv0();
      bump(me + F_I + (q * 96 + prev) * 128);
    }
    STAMP(3);
    const u32 role = wave < P1W ? lds_add(ctl + C_ELECT, 1u) : 2u;
    if (!good) ctl[C_FAIL] = 1;
    if (role == 0) {
      const u32 tbase = (call - 1) * 256u - WPL;
      const u32 v16 = lane * 16, vB = lane < 32 ? lane * 16 : OOB;
      u32 z = idx, n = 0;
      bool pend = false;
      while (z < 256) {
        const u32 i = z >> 3, w8 = z & 7, slot = n % NSLOT;
        bool fail = false;
        if (!((ctl[C_READY] >> i) & 1) || ctl[C_DONE + slot] != 4 * (n / NSLOT)) {
          waitv0();
          if (pend) { ctl[C_FULL + (n - 1) % NSLOT] = n; pend = false; }
          while (ctl[C_DONE + slot] != 4 * (n / NSLOT)) __builtin_amdgcn_s_sleep(1);
          while (!((ctl[C_READY] >> i) & 1)) {
            if (ctl[C_FAIL]) { fail = true; break; }
            __builtin_amdgcn_s_sleep(1);
          }
        }
        const u32 raw = fail ? 0u : (ctl[C_RAWM + i] >> w8) & 1;
        ctl[C_ASG + slot] = ((n + 1) << 12) | (fail ? 512u : 0u) | (raw << 8) | z;
        u32 s, c;
        tile_sc(i, s, c);
        const u32 so = (fail || raw) ? OOB : par + P_PK + (((rank * 4 + s) * 8 + c) * 8 + w8) * SUB;
        LDS_AS char* ld = (LDS_AS char*)smem + slot * SLOTB;
        asm volatile("s_waitcnt vmcnt(%0)" :: "i"(LWIN - 1) : "memory");
        const u32 tk = draw(me + F_T2 + q * 128);
#define PULL(k, vofs, grp, imm) \
        asm volatile("s_waitcnt vmcnt(%0)" :: "i"(LWIN - 1) : "memory"); \
        if ((k) + 1 == LWIN - 1 && pend) ctl[C_FULL + (n - 1) % NSLOT] = n; \
        __builtin_amdgcn_raw_ptr_buffer_load_lds(rq, ld + (grp) * 4096, 16, vofs, so + (grp) * 4096, imm, AUX_SYS);
        PULL(0, v16, 2, 0) PULL(1, v16, 2, 1024) PULL(2, vB, 2, 2048)
        PULL(3, v16, 0, 0) PULL(4, v16, 0, 1024) PULL(5, v16, 0, 2048) PULL(6, v16, 0, 3072)
        PULL(7, v16, 1, 0) PULL(8, v16, 1, 1024) PULL(9, v16, 1, 2048) PULL(10, v16, 1, 3072)
#undef PULL
        asm volatile("s_waitcnt vmcnt(11)" ::: "memory");
        z = take(tk) - tbase;
        pend = true;
        ++n;
      }
      waitv0();
      if (pend) ctl[C_FULL + (n - 1) % NSLOT] = n;
      ctl[C_END] = n;
    } else {
      if (role == 1) {
        const u64 t0 = __builtin_amdgcn_s_memrealtime();
        for (;;) {
          const u32 l = lane & 31;
          const u64 of = __hip_atomic_load((const u64*)(me + F_O + (q * 32 + l) * 128), __ATOMIC_RELAXED, __HIP_MEMORY_SCOPE_SYSTEM);
          const u32 pf = ld_agt(me + F_P + (q * 32 + l) * 128);
          const bool tmo = ld_agt(me + F_TMO) == call;
          const u32 ready = (u32)__ballot((u32)(of >> 32) >= call && pf >= call);
          if (tmo || ctl[C_FAIL] || __builtin_amdgcn_s_memrealtime() - t0 > DEADLINE) {
            st_agt(me + F_TMO, call);
            ctl[C_FAIL] = 1;
            break;
          }
          if (lane < 32) ctl[C_RAWM + lane] = (u32)of;
          asm volatile("s_waitcnt lgkmcnt(0)" ::: "memory");
          ctl[C_READY] = ready;
          if (ready == 0xFFFFFFFFu) break;
          __builtin_amdgcn_s_sleep(16);
        }
      }
      const float scale = P.sx[0] * P.sw[0];
      const u32 vo = (u32)(lane >> 4) * 4 * (NCOL * 4) + (lane & 15) * 16;
      for (;;) {
        const u32 item = lds_add(ctl + C_ITEM, 1u);
        const u32 n = item >> 2, m = item & 3, slot = n % NSLOT;
        u32 a;
        for (;;) {
          a = __builtin_amdgcn_readfirstlane(ctl[C_ASG + slot]);
          if ((a >> 12) == n + 1 || n >= __builtin_amdgcn_readfirstlane(ctl[C_END])) break;
          __builtin_amdgcn_s_sleep(1);
        }
        if ((a >> 12) != n + 1) break;
        asm volatile("" ::: "memory");
        const u32 z = a & 255, w8 = z & 7;
        u32 s, c;
        tile_sc(z >> 3, s, c);
        const u32 oo = ((s * 128 + (w8 >> 2) * 64 + m * 16) * NCOL + q * 2048 + c * 256 + (w8 & 3) * 64) * 4;
        u32x4 y[4];
        if (a & 512u) {
#pragma unroll
          for (int r = 0; r < 4; ++r) y[r] = (u32x4){0x7FC00000u, 0x7FC00000u, 0x7FC00000u, 0x7FC00000u};
          lds_add(ctl + C_DONE + slot, 1u);
        } else {
          u32x4 pt[4];
          i32x4 v[4];
#pragma unroll
          for (int r = 0; r < 4; ++r) pt[r] = __builtin_amdgcn_raw_buffer_load_b128(rout, vo + r * (NCOL * 4), oo, AUX_AGT);
          if (a & 256u) {
            const u32 so = par + P_RAW + (((rank * 4 + s) * 8 + c) * 8 + w8) * 16384 + m * 4096;
#pragma unroll
            for (int r = 0; r < 4; ++r) v[r] = __builtin_bit_cast(i32x4, __builtin_amdgcn_raw_buffer_load_b128(rq, lane * 16 + r * 1024, so, AUX_SYS));
            lds_add(ctl + C_DONE + slot, 1u);
          } else {
            while (__builtin_amdgcn_readfirstlane(ctl[C_FULL + slot]) != n + 1) __builtin_amdgcn_s_sleep(1);
            asm volatile("" ::: "memory");
            const char* ls = smem + slot * SLOTB;
            const u32x2 bb = *(const u32x2*)(ls + 10240 + lane * 8);
            const u32x4 n0 = *(const u32x4*)(ls + 8192 + lane * 16), n1 = *(const u32x4*)(ls + 9216 + lane * 16);
            const u32x4 l0 = *(const u32x4*)(ls + m * 2048 + lane * 16), l1 = *(const u32x4*)(ls + m * 2048 + 1024 + lane * 16);
            asm volatile("s_waitcnt lgkmcnt(0)" :: "v"(bb), "v"(n0), "v"(n1), "v"(l0), "v"(l1) : "memory");
            lds_add(ctl + C_DONE + slot, 1u);
            u32 cnt = 0;
            if (m > 0) cnt += __builtin_popcount(n0[0] & 0x88888888u) + __builtin_popcount(n0[1] & 0x88888888u);
            if (m > 1) cnt += __builtin_popcount(n0[2] & 0x88888888u) + __builtin_popcount(n0[3] & 0x88888888u);
            if (m > 2) cnt += __builtin_popcount(n1[0] & 0x88888888u) + __builtin_popcount(n1[1] & 0x88888888u);
            u32 pos = 0, wd = bb[0], pc = __builtin_popcount(wd);
            if (cnt >= pc) { cnt -= pc; wd = bb[1]; pos = 32; }
#pragma unroll
            for (int wdt = 16; wdt >= 1; wdt >>= 1) {
              pc = __builtin_popcount(wd & ((1u << wdt) - 1u));
              if (cnt >= pc) { cnt -= pc; wd >>= wdt; pos += wdt; }
            }
            u64 B = (((u64)bb[1] << 32) | bb[0]) >> (pos & 63u);
            const u32 nd0 = m == 0 ? n0[0] : m == 1 ? n0[2] : m == 2 ? n1[0] : n1[2];
            const u32 nd1 = m == 0 ? n0[1] : m == 1 ? n0[3] : m == 2 ? n1[1] : n1[3];
#pragma unroll
            for (int r = 0; r < 4; ++r) {
              const int nd = (int)(r < 2 ? nd0 : nd1);
#pragma unroll
              for (int t = 0; t < 4; ++t) {
                const int sh = 8 * t + 4 * (r & 1);
                const u32 h = ffbl((u32)B ^ 1u) & (u32)__builtin_amdgcn_sbfe(nd, sh + 3, 1);
                B >>= (h & 63u);
                const u32 hi = (u32)__builtin_amdgcn_sbfe(nd, sh, 3) ^ (h << 2);
                const u32 lo = r < 2 ? l0[(r & 1) * 2 + (t >> 1)] : l1[(r & 1) * 2 + (t >> 1)];
                v[r][t] = (int)perm(hi, lo, (t & 1) ? 0x05040302u : 0x05040100u);
              }
            }
          }
#pragma unroll
          for (int r = 0; r < 4; ++r)
#pragma unroll
            for (int t = 0; t < 4; ++t) y[r][t] = __builtin_bit_cast(u32, (float)(v[r][t] + (int)pt[r][t]) * scale);
        }
#pragma unroll
        for (int r = 0; r < 4; ++r) st128(y[r], rout, vo + r * (NCOL * 4), oo, AUX_AGT);
      }
    }
    STAMP(4);
    return;
  }

  const u32 vo = ((u32)(wave >> 2) * 64 + (lane >> 4) * 4) * (NCOL * 4) + ((wave & 3) * 64 + (lane & 15) * 4) * 4;
  i32x4 acc[4][4];
  for (int job = 0; job < 4; ++job) {
    const bool partial = job < 2;
    u32 t;
    if (job == 0) t = gi;
    else if (job == 1) { if (gi < NGEM - 24) continue; t = gi + 24; }
    else if (job == 2) { if (gi >= 192) break; t = gi; }
    else { if (gi >= 64) break; t = 192 + gi; }
    u32 g, c, cg_self, foff, s, dst, oi, tgt = call;
    int dskip, nk;
    bool act;
    if (partial) {
      const u32 r = t >> 3;
      oi = (r / 7) * 8 + (t & 7);
      tile_sc(oi, s, c);
      dst = (rank + 1 + r % 7) & 7u;
      g = rank * 4 + s;
      cg_self = dst * 8 + c;
      dskip = 8; nk = 4;
      act = lane < 6;
      foff = lane < 4 ? F_LW + (cg_self * 4 + lane) * 128 : F_S + (rank * 96 + g * 2 + (lane & 1)) * 128;
    } else {
      u32 pd;
      if (t < 64) { oi = (t >> 5) * 4 + (t & 3); pd = (t >> 2) & 7; }
      else { oi = (t >> 6) * 8 + (t & 7); pd = (t >> 3) & 7; }
      tile_sc(oi, s, c);
      dst = (rank + 1 + pd) & 7u;
      g = dst * 4 + s;
      cg_self = rank * 8 + c;
      dskip = dst == rank ? 8 : (int)((dst - rank) & 7u);
      nk = dst == rank ? 32 : 28;
      const u32 q = lane / 6, e = lane % 6;
      act = lane < 48 && (q != dst || dst == rank);
      const u32 id = e < 4 ? 64 + c * 4 + e : g * 2 + (e & 1);
      foff = (q == rank ? F_S : F_I) + ((q & 7) * 96 + id) * 128;
      tgt = q == rank ? call : call * 8;
    }
    const bool good = wg_wait(smem, me, foff, act, tgt, call, wave);
    STAMP(2 + job * 3);
    if (good) gemm_tile(smem, rme, (int)rank, par, g, cg_self, c, dskip, nk, wave, lane, acc);
    STAMP(3 + job * 3);
    if (partial) {
      if (good) {
        const u32 oo = (s * 128 * NCOL + dst * 2048 + c * 256) * 4;
#pragma unroll
        for (int m = 0; m < 4; ++m)
#pragma unroll
          for (int r = 0; r < 4; ++r) {
            const u32x4 y = {(u32)acc[m][0][r], (u32)acc[m][1][r], (u32)acc[m][2][r], (u32)acc[m][3][r]};
            st128(y, rout, vo + (m * 16 + r) * (NCOL * 4), oo, AUX_AGT);
          }
        waitv0();
        bar();
        if (tid == 0) st_agt(me + F_P + (dst * 32 + oi) * 128, call);
      }
    } else if (dst == rank) {
      const float scale = P.sx[0] * P.sw[0];
      const u32 oo = (s * 128 * NCOL + rank * 2048 + c * 256) * 4;
#pragma unroll
      for (int m = 0; m < 4; ++m)
#pragma unroll
        for (int r = 0; r < 4; ++r) {
          u32x4 y;
#pragma unroll
          for (int tt = 0; tt < 4; ++tt) y[tt] = good ? __builtin_bit_cast(u32, (float)acc[m][tt][r] * scale) : 0x7FC00000u;
          st128(y, rout, vo + (m * 16 + r) * (NCOL * 4), oo, AUX_AGT);
        }
    } else if (good) {
      if (tid == 0) ctl[C_RAW] = 0;
      bar();
      const u32 sub = (g * 8 + c) * 8 + wave;
      u64 B = 1;
      u32 sum = 0, any = 0;
      u32x4 nb[2], lo[4][2];
#pragma unroll
      for (int m = 3; m >= 0; --m) {
        u32 hb[4];
#pragma unroll
        for (int r = 3; r >= 0; --r) {
          u32 wv[4];
#pragma unroll
          for (int tt = 3; tt >= 0; --tt) {
            const int v = acc[m][tt][r];
            const u32 sg = (u32)(v >> 31), h = (u32)(v >> 18) ^ sg;
            B = (B << (h & 63u)) | 1u;
            sum += h;
            any |= h;
            wv[tt] = ((u32)v & 0x30000u) | (sg & 0x40000u) | ((0u - h) & 0x80000u);
          }
          lo[m][r >> 1][(r & 1) * 2] = perm(acc[m][1][r], acc[m][0][r], 0x05040100u);
          lo[m][r >> 1][(r & 1) * 2 + 1] = perm(acc[m][3][r], acc[m][2][r], 0x05040100u);
          hb[r] = perm(perm(wv[3], wv[2], 0x0c0c0602u), perm(wv[1], wv[0], 0x0c0c0602u), 0x05040100u);
        }
        nb[m >> 1][(m & 1) * 2] = hb[0] | (hb[1] << 4);
        nb[m >> 1][(m & 1) * 2 + 1] = hb[2] | (hb[3] << 4);
      }
      if (__any(sum > 63u || any > 31u)) {
        if (lane == 0) __hip_atomic_fetch_or((u32*)(smem + LDS_CTL) + C_RAW, 1u << wave, __ATOMIC_RELAXED, __HIP_MEMORY_SCOPE_WORKGROUP);
#pragma unroll
        for (int m = 0; m < 4; ++m)
#pragma unroll
          for (int r = 0; r < 4; ++r) {
            const u32x4 y = {(u32)acc[m][0][r], (u32)acc[m][1][r], (u32)acc[m][2][r], (u32)acc[m][3][r]};
            st128(y, rme, lane * 16 + (m * 4 + r) * 1024, par + P_RAW + sub * 16384, AUX_SYS);
          }
      } else {
        const u32 so = par + P_PK + sub * SUB;
#pragma unroll
        for (int m = 0; m < 4; ++m) {
          st128(lo[m][0], rme, lane * 16 + m * 2048, so, AUX_SYS);
          st128(lo[m][1], rme, lane * 16 + m * 2048 + 1024, so, AUX_SYS);
        }
        st128(nb[0], rme, lane * 16 + 8192, so, AUX_SYS);
        st128(nb[1], rme, lane * 16 + 9216, so, AUX_SYS);
        __builtin_amdgcn_raw_buffer_store_b64((u32x2){(u32)B, (u32)(B >> 32)}, rme, lane * 8 + 10240, so, AUX_SYS);
      }
      waitv0();
      bar();
      if (tid == 0) {
        const u64 f = ((u64)call << 32) | ctl[C_RAW];
        __hip_atomic_store((u64*)(P.ws[dst] + F_O + (rank * 32 + oi) * 128), f, __ATOMIC_RELAXED, __HIP_MEMORY_SCOPE_SYSTEM);
      }
    }
    STAMP(4 + job * 3);
  }
}

}

extern "C" void kernel_launch(int rank, int nranks, ncclComm_t comm,
                              hipStream_t stream,
                              void* const* d_in, const int* in_sizes, int n_in,
                              void* d_out, int out_size,
                              const int* in_dtypes, int out_dtype,
                              void* const* d_out_all,
                              void* const* d_ws_all, size_t ws_bytes) {
  if (nranks != NR || (unsigned)rank >= (unsigned)NR || n_in < 4 || ws_bytes < (size_t)WS_NEED + MiB ||
      (size_t)out_size * 4 != OUT_BYTES || in_sizes[0] != 4096 * KSL || in_sizes[1] != KSL * NCOL) {
    fprintf(stderr, "kernel_launch: unexpected shapes\n");
    return;
  }
  Params p;
  p.x = (const signed char*)d_in[0];
  p.w = (const signed char*)d_in[1];
  p.sx = (const float*)d_in[2];
  p.sw = (const float*)d_in[3];
  p.out = (float*)d_out;
  for (int i = 0; i < NR; ++i) p.ws[i] = (char*)d_ws_all[i];
  p.me = p.ws[rank];
  p.ws_bytes = WS_NEED + MiB;
  p.rank = rank;
  gemm_rs_kernel<<<NWG, NTHR, 0, stream>>>(p);
  HIP_CHECK(hipGetLastError());
}
```

```cpp
#include <hip/hip_runtime.h>
#include <rccl/rccl.h>
#include <cstdio>

#define HIP_CHECK(expr) do { hipError_t _e = (expr); if (_e != hipSuccess) { \
    fprintf(stderr, "HIP error %s at %s:%d\n", hipGetErrorString(_e), __FILE__, __LINE__); \
    return; } } while (0)

typedef unsigned u32;
typedef unsigned long long u64;
typedef u32 u32x4 __attribute__((ext_vector_type(4)));
typedef u32 u32x2 __attribute__((ext_vector_type(2)));
typedef int i32x4 __attribute__((ext_vector_type(4)));
typedef short s16x2 __attribute__((ext_vector_type(2)));
typedef unsigned short u16x2 __attribute__((ext_vector_type(2)));
typedef __amdgpu_buffer_rsrc_t rsrc_t;
#define LDS_AS __attribute__((address_space(3)))
#define UNI(x) __builtin_amdgcn_readfirstlane(x)

namespace {

constexpr int NR = 8;
constexpr int KSL = 512;
constexpr int NCOL = 16384;
constexpr int NWG = 256, NTHR = 512;
constexpr int WPL = 8;
constexpr int NSTR = 7 * WPL;
constexpr int NGEM = NWG - NSTR;
#define EXPORT_DELAY 24
#define GAUX 17
#define RDMASK 0x0B6Du
#define DMAMASK 0x2082u
#define V_SEEN 1
#define NPUSH 6
#define OAUX 16
#define WSLEEP 4
#define WIDE 0
#define LW1 8
#define LW2 8

constexpr u32 KiB = 1024u, MiB = 1024u * 1024u;
constexpr u32 F_CNT = 0;
constexpr u32 F_TMO = 32 * KiB;
constexpr u32 F_S = 64 * KiB;
constexpr u32 F_I = 160 * KiB;
constexpr u32 F_LW = 256 * KiB;
constexpr u32 F_O = 288 * KiB;
constexpr u32 F_P = 320 * KiB;
constexpr u32 F_T = 352 * KiB;
constexpr u32 XBLK = 16 * KiB, WBLK = 32 * KiB;
constexpr u32 D_IX = 1 * MiB;
constexpr u32 D_PAR = D_IX + 16 * MiB;
constexpr u32 P_EX = 0;
constexpr u32 P_EW = 2 * MiB;
constexpr u32 SUB = 10496;
constexpr u32 P_PK = 10 * MiB;
constexpr u32 P_RAW = 32 * MiB;
constexpr u32 P_IW = 64 * MiB;
constexpr u32 PAR_BYTES = 72 * MiB;
constexpr u32 WS_NEED = D_PAR + 2 * PAR_BYTES;
constexpr u32 OUT_BYTES = 512u * NCOL * 4u;

constexpr u32 STAGE = XBLK + WBLK;
constexpr u32 LDS_CTL = 3 * STAGE;
constexpr u32 LDS_BYTES = LDS_CTL + 512;
constexpr u32 NSLOT = 8, SLOTB = 11264;
constexpr u32 LDS_PT = NSLOT * SLOTB;
static_assert(LDS_PT + 7 * 8192 <= LDS_CTL, "LDS");
constexpr int C_RAW = 8, C_IN = 12, C_ITEM = 17, C_END = 18, C_READY = 19, C_FAIL = 20,
              C_LAND = 21, C_RAWM = 24, C_ASG = 64, C_BASE = 80, C_DONE = 96;

constexpr u64 DEADLINE = 40000000ull;
constexpr int AUX_SYS = 17, AUX_AGT = 16;
constexpr u32 OOB = 0x70000000u;

struct Params {
  const signed char* x;
  const signed char* w;
  const float* sx;
  const float* sw;
  float* out;
  char* me;
  char* ws[NR];
  u32 ws_bytes;
  int rank;
};

__device__ __forceinline__ rsrc_t mk_rsrc(const void* p, u32 bytes) {
  return __builtin_amdgcn_make_buffer_rsrc((void*)p, (short)0, (int)bytes, 0x00020000);
}
#define st128(v, r, vo, so, aux) __builtin_amdgcn_raw_buffer_store_b128((v), (r), (vo) + (so), 0, (aux))
__device__ __forceinline__ void waitv0() { asm volatile("s_waitcnt vmcnt(0)" ::: "memory"); }
__device__ __forceinline__ void bar() {
  asm volatile("s_waitcnt lgkmcnt(0)" ::: "memory");
  __builtin_amdgcn_s_barrier();
  asm volatile("" ::: "memory");
}
__device__ __forceinline__ u32 ld_sys(const char* p) {
  return __hip_atomic_load((const u32*)p, __ATOMIC_RELAXED, __HIP_MEMORY_SCOPE_SYSTEM);
}
__device__ __forceinline__ u32 ld_agt(const char* p) {
  return __hip_atomic_load((const u32*)p, __ATOMIC_RELAXED, __HIP_MEMORY_SCOPE_AGENT);
}
__device__ __forceinline__ void st_sys(char* p, u32 v) {
  __hip_atomic_store((u32*)p, v, __ATOMIC_RELAXED, __HIP_MEMORY_SCOPE_SYSTEM);
}
__device__ __forceinline__ void st_agt(char* p, u32 v) {
  __hip_atomic_store((u32*)p, v, __ATOMIC_RELAXED, __HIP_MEMORY_SCOPE_AGENT);
}

#define STAMP(slot) do {} while (0)
#define STAMPV(slot, v) do {} while (0)

__device__ __forceinline__ bool wave_wait(const char* me, u32 off, bool act, u32 tgt, u32 call) {
  bool ok = !act || ld_sys(me + off) >= tgt;
  if (__all(ok)) return true;
  const u64 t0 = __builtin_amdgcn_s_memrealtime();
  for (;;) {
    __builtin_amdgcn_s_sleep(6);
    if (!ok) ok = ld_sys(me + off) >= tgt;
    if (__all(ok)) return true;
    if (UNI(ld_agt(me + F_TMO)) == call || __builtin_amdgcn_s_memrealtime() - t0 > DEADLINE) {
      st_agt((char*)me + F_TMO, call);
      return false;
    }
  }
}

__device__ __forceinline__ bool wg_wait(char* smem, const char* me, u32 off, bool act, u32 tgt, u32 call, int wave) {
  volatile LDS_AS u32* word = (volatile LDS_AS u32*)((LDS_AS char*)smem + LDS_CTL);
  if (wave == 0) {
    bool r = wave_wait(me, off, act, tgt, call);
    r = r && UNI(ld_agt(me + F_TMO)) != call;
    *word = r ? 1u : 2u;
  }
  bar();
  const u32 r = UNI(*word);
  bar();
  return r == 1u;
}

__device__ __forceinline__ u32 ffbl(u32 x) {
  u32 r;
  asm("v_ffbl_b32 %0, %1" : "=v"(r) : "v"(x));
  return r;
}
__device__ __forceinline__ u32 wave_scan(u32 x) {
  x += __builtin_amdgcn_update_dpp(0u, x, 0x111, 0xF, 0xF, false);
  x += __builtin_amdgcn_update_dpp(0u, x, 0x112, 0xF, 0xF, false);
  x += __builtin_amdgcn_update_dpp(0u, x, 0x114, 0xF, 0xF, false);
  x += __builtin_amdgcn_update_dpp(0u, x, 0x118, 0xF, 0xF, false);
  x += __builtin_amdgcn_update_dpp(0u, x, 0x142, 0xA, 0xF, false);
  x += __builtin_amdgcn_update_dpp(0u, x, 0x143, 0xC, 0xF, false);
  return x;
}
__device__ __forceinline__ u32 code_window(u32 lp, int lane, u32& total) {
  const u32 end = 224u + wave_scan((lane & 1) ? 0u : lp);
  total = __builtin_amdgcn_readlane(end, 63);
  return (lane & 1) ? end - 64u : end - lp;
}
__device__ __forceinline__ u64 rev64(u64 x) {
  return ((u64)__builtin_bitreverse32((u32)x) << 32) | __builtin_bitreverse32((u32)(x >> 32));
}
__device__ __forceinline__ u32 perm(u32 hi, u32 lo, u32 sel) { return __builtin_amdgcn_perm(hi, lo, sel); }

__device__ __forceinline__ void gemm_tile(char* smem, rsrc_t rws, int rank, u32 par, u32 g, u32 cg_self, u32 c,
                                          int dskip, int k0, int nk, int wave, int lane, i32x4 (&acc)[4][4]) {
  const u32 va = (u32)wave * 2048u + (u32)lane * 16u;
  const u32 vb = (u32)wave * 4096u + (u32)lane * 16u;
  u32 sa, sb;
  auto locate = [&](int kb) {
    const int qi = kb >> 2;
    const u32 kc = kb & 3;
    const int d = qi + (qi >= dskip ? 1 : 0);
    const u32 q = (u32)(rank + d) & 7u;
    sa = (d == 0 ? par + P_EX : D_IX + q * (2 * MiB)) + (g * 4 + kc) * XBLK;
    sb = (d == 0 ? par + P_EW + cg_self * (4 * WBLK) : par + P_IW + (q * 8 + c) * (4 * WBLK)) + kc * WBLK;
    if (kb >= nk) sa = sb = OOB;
  };
  auto dma = [&](int n, u32 buf) {
    LDS_AS char* la = (LDS_AS char*)smem + buf * STAGE + wave * 2048;
    LDS_AS char* lb = (LDS_AS char*)smem + buf * STAGE + XBLK + wave * 4096;
    if (n == 0) __builtin_amdgcn_raw_ptr_buffer_load_lds(rws, la, 16, va, sa, 0, GAUX);
    if (n == 1) __builtin_amdgcn_raw_ptr_buffer_load_lds(rws, la, 16, va, sa, 1024, GAUX);
    if (n == 2) __builtin_amdgcn_raw_ptr_buffer_load_lds(rws, lb, 16, vb, sb, 0, GAUX);
    if (n == 3) __builtin_amdgcn_raw_ptr_buffer_load_lds(rws, lb, 16, vb, sb, 1024, GAUX);
    if (n == 4) __builtin_amdgcn_raw_ptr_buffer_load_lds(rws, lb, 16, vb, sb, 2048, GAUX);
    if (n == 5) __builtin_amdgcn_raw_ptr_buffer_load_lds(rws, lb, 16, vb, sb, 3072, GAUX);
  };
  const u32 ra = (u32)(wave >> 2) * 4096u + (u32)lane * 16u;
  const u32 rb = XBLK + (u32)(wave & 3) * 4096u + (u32)lane * 16u;
  i32x4 f0[8], f1[8];
  auto rd = [&](i32x4 (&f)[8], int n, u32 buf, int ks) {
    const int j = n == 0 ? 0 : (n < 5 ? n + 3 : n - 4);
    const char* p = smem + buf * STAGE + (j < 4 ? ra + ks * 8192 + j * 1024 : rb + ks * 16384 + (j - 4) * 1024);
    f[j] = *(const i32x4*)p;
  };
  auto half = [&](i32x4 (&fc)[8], i32x4 (&fn)[8], u32 rbuf, int rks, int d0, u32 dbuf) {
#pragma unroll
    for (int i = 0; i < 16; ++i) {
      acc[i >> 2][i & 3] = __builtin_amdgcn_mfma_i32_16x16x64_i8(fc[i >> 2], fc[4 + (i & 3)], acc[i >> 2][i & 3], 0, 0, 0);
      if ((RDMASK >> i) & 1) rd(fn, __builtin_popcount(RDMASK & ((1u << i) - 1u)), rbuf, rks);
      if ((DMAMASK >> i) & 1) dma(d0 + __builtin_popcount(DMAMASK & ((1u << i) - 1u)), dbuf);
      __builtin_amdgcn_sched_barrier(0);
    }
  };
  locate(k0);
#pragma unroll
  for (int n = 0; n < 6; ++n) dma(n, 0);
  locate(k0 + 1);
#pragma unroll
  for (int n = 0; n < 6; ++n) dma(n, 1);
  locate(k0 + 2);
#pragma unroll
  for (int n = 0; n < 3; ++n) dma(n, 2);
  asm volatile("s_waitcnt vmcnt(9)" ::: "memory");
  bar();
#pragma unroll
  for (int n = 0; n < 8; ++n) rd(f0, n, 0, 0);
  u32 buf = 0;
  for (int kb = k0; kb < nk; ++kb) {
    const u32 b1 = buf == 2 ? 0u : buf + 1, b2 = buf == 0 ? 2u : buf - 1;
    __builtin_amdgcn_sched_barrier(0);
    half(f0, f1, buf, 1, 3, b2);
    locate(kb + 3);
    asm volatile("s_waitcnt vmcnt(6)" ::: "memory");
    bar();
    __builtin_amdgcn_sched_barrier(0);
    half(f1, f0, b1, 0, 0, buf);
    buf = b1;
  }
  waitv0();
}

constexpr u32 NPULL = 88 - NPUSH;
__device__ __forceinline__ u32 import_id(u32 o, u32 q) {
  if (o >= 14) {
    o += NPUSH - 4;
    if (o < 18) return 64 + 4 + (o - 14);
    if (o >= 32 && o < 56) return 64 + 8 + (o - 32);
  }
  u32 s, e;
  if (o < 14) { s = 0; e = o; }
  else if (o < 32) { s = 1; e = o - 18; }
  else if (o < 70) { s = 2; e = o - 56; }
  else { s = 3; e = o - 70; }
  u32 p = e >> 1;
  p += (p >= q) ? 1 : 0;
  return (4 * p + s) * 2 + (e & 1);
}
__device__ __forceinline__ void tile_sc(u32 i, u32& s, u32& c) {
  if (i < 4) { s = i >> 1; c = i & 1; }
  else if (i < 16) { s = i & 1; c = i >> 1; }
  else { s = i >> 3; c = i & 7; }
}
__device__ __forceinline__ u32 draw(char* p) {
  u32 t;
  u64 save;
  asm volatile("s_mov_b64 %1, exec\n\ts_mov_b64 exec, 1\n\tglobal_atomic_add %0, %2, %3, off sc0\n\ts_mov_b64 exec, %1"
               : "=&v"(t), "=&s"(save) : "v"(p), "v"(1u) : "memory");
  return t;
}
__device__ __forceinline__ u32 take(u32 t) {
  asm volatile("s_nop 0" : "+v"(t));
  return UNI(t);
}
__device__ __forceinline__ u32 lds_add(volatile LDS_AS u32* p, u32 x) {
  u32 t;
  u64 save;
  asm volatile("s_mov_b64 %1, exec\n\ts_mov_b64 exec, 1\n\tds_add_rtn_u32 %0, %2, %3\n\ts_waitcnt lgkmcnt(0)\n\ts_mov_b64 exec, %1"
               : "=&v"(t), "=&s"(save) : "v"((u32)(size_t)p), "v"(x) : "memory");
  return UNI(t);
}
__device__ __forceinline__ void lds_inc(volatile LDS_AS u32* p) {
  u64 save;
  asm volatile("s_mov_b64 %0, exec\n\ts_mov_b64 exec, 1\n\tds_add_u32 %1, %2\n\ts_mov_b64 exec, %0"
               : "=&s"(save) : "v"((u32)(size_t)p), "v"(1u) : "memory");
}
__device__ __forceinline__ void bump(char* p) {
  u64 save;
  asm volatile("s_mov_b64 %0, exec\n\ts_mov_b64 exec, 1\n\tglobal_atomic_add %1, %2, off\n\ts_mov_b64 exec, %0"
               : "=&s"(save) : "v"(p), "v"(1u) : "memory");
}

__global__ void __launch_bounds__(NTHR, 2) gemm_rs_kernel(Params P) {
  __shared__ __attribute__((aligned(16))) char smem[LDS_BYTES];
  const int tid = threadIdx.x, lane = tid & 63;
  const int wave = UNI(tid >> 6);
  const u32 b = blockIdx.x;
  const u32 rank = (u32)P.rank & 7u;
  char* const me = P.me;
  asm volatile("" :: "s"(P.x), "s"(P.w), "s"(me), "s"(P.out), "s"(P.ws_bytes), "s"(P.rank));
  const rsrc_t rme = mk_rsrc(me, P.ws_bytes);
  const rsrc_t rout = mk_rsrc(P.out, OUT_BYTES);
  volatile LDS_AS u32* ctl = (volatile LDS_AS u32*)((LDS_AS char*)smem + LDS_CTL);
  STAMP(0);

  const u32 gi = b - NSTR;
  const u32 cnt = ld_agt(me + F_CNT + b * 128);
  if (b < NSTR) {
    if (tid < 128) ctl[tid] = tid == C_END ? 0xFFFFFFFFu : 0u;
    const u32 call = UNI(cnt) + 1;
    bar();
    if (tid == 0) st_agt(me + F_CNT + b * 128, call);
  }
  for (int e = 0; e < 2 && b >= NSTR; ++e) {
    u32 wp = 0xFFFFu, xp = 0xFFFFu, v = 0xFFFFu;
    if (e == 0) {
      if (gi < 64) {
        wp = (gi >> 3) * 32 + (gi & 7);
        if ((gi & 7) >= NPUSH) __builtin_amdgcn_s_sleep(EXPORT_DELAY);
      } else if (gi < 80) {
        xp = ((gi - 64) >> 1) * 8 + (gi & 1);
      } else {
        v = gi - 80;
        __builtin_amdgcn_s_sleep(EXPORT_DELAY);
      }
    } else if (gi >= 80) {
      v = gi + 40;
    }
    if (v < 16) xp = (v >> 1) * 8 + 2 + (v & 1);
    else if (v < 208) wp = ((v - 16) / 24) * 32 + 8 + (v - 16) % 24;
    else if (v < 240) xp = ((v - 208) >> 2) * 8 + 4 + (v & 3);
    const u32 vs = wave * 4096 + lane * 16;
    if (wp != 0xFFFFu) {
      const u32 cg = wp >> 2, kc = wp & 3, dj = (cg >> 3) & 7;
      u32 d[16];
      const rsrc_t rw = mk_rsrc(P.w, (u32)KSL * NCOL);
      const u32 vw = (kc * 128 + (wave >> 2) * 64 + (lane >> 4) * 16) * NCOL + cg * 256 + (wave & 3) * 64 + (lane & 15) * 4;
#pragma unroll
      for (int r = 0; r < 16; ++r) d[r] = __builtin_amdgcn_raw_buffer_load_b32(rw, vw, r * NCOL, 0);
      char* const peer = P.ws[dj];
      const u32 call = UNI(cnt) + 1, par = D_PAR + (call & 1) * PAR_BYTES;
      const bool push = (wp & 31) < NPUSH && dj != rank;
      u32x4 o[4];
#pragma unroll
      for (int r4 = 0; r4 < 4; ++r4) {
        const u32 t0 = perm(d[4 * r4 + 1], d[4 * r4], 0x05010400u), t1 = perm(d[4 * r4 + 1], d[4 * r4], 0x07030602u);
        const u32 t2 = perm(d[4 * r4 + 3], d[4 * r4 + 2], 0x05010400u), t3 = perm(d[4 * r4 + 3], d[4 * r4 + 2], 0x07030602u);
        o[0][r4] = perm(t2, t0, 0x05040100u);
        o[1][r4] = perm(t2, t0, 0x07060302u);
        o[2][r4] = perm(t3, t1, 0x05040100u);
        o[3][r4] = perm(t3, t1, 0x07060302u);
      }
      if (push) {
        const rsrc_t rj = mk_rsrc(peer, P.ws_bytes);
#pragma unroll
        for (int t = 0; t < 4; ++t) st128(o[t], rj, vs + t * 1024, par + P_IW + (rank * 32 + (wp & 31)) * WBLK, AUX_SYS);
      }
#pragma unroll
      for (int t = 0; t < 4; ++t) st128(o[t], rme, vs + t * 1024, par + P_EW + wp * WBLK, AUX_SYS);
      waitv0();
      bar();
      if (tid == 0) st_sys(peer + (push ? F_I : F_S) + (rank * 96 + 64 + (wp & 31)) * 128, push ? call * 16 : call);
      if (tid == 64) st_sys(me + F_LW + wp * 128, call);
    } else if (xp != 0xFFFFu) {
      u32x4 o[4];
      const rsrc_t rx = mk_rsrc(P.x, 4096u * KSL);
      const u32 vx = ((xp >> 1) * 128 + (wave & 1) * 64 + (lane & 15)) * KSL + ((xp & 1) * 2 + (wave >> 2)) * 128 + ((wave >> 1) & 1) * 64 + (lane >> 4) * 16;
#pragma unroll
      for (int u = 0; u < 4; ++u) o[u] = __builtin_amdgcn_raw_buffer_load_b128(rx, vx, u * 16 * KSL, 0);
      const u32 call = UNI(cnt) + 1, par = D_PAR + (call & 1) * PAR_BYTES;
#pragma unroll
      for (int t = 0; t < 4; ++t) st128(o[t], rme, vs + t * 1024, par + P_EX + xp * (2 * XBLK), AUX_SYS);
      waitv0();
      bar();
      if (lane == 0) st_sys(P.ws[wave] + F_S + (rank * 96 + xp) * 128, call);
    }
    if (e == 0 && tid == 64) st_agt(me + F_CNT + b * 128, cnt + 1);
  }
  const u32 call = UNI(cnt) + 1, par = D_PAR + (call & 1) * PAR_BYTES;
  STAMP(1);

  if (b < NSTR) {
    const u32 q = (rank + 1 + b % 7) & 7u, idx = b / 7;
    const rsrc_t rq = mk_rsrc(P.ws[q], P.ws_bytes);
    constexpr u32 NIN = NPULL * 4, NTK = NIN + 256;
    const u32 fo0 = F_S + (q * 96 + import_id(lane, q)) * 128;
    if (wave == 0) {
      const u32 tbase = (call - 1) * NTK - WPL;
      const u32 v16 = lane * 16, vB = lane < 16 ? lane * 16 : OOB;
      u32 z = idx, n = 0, so = 0, slot = 0, raw = 0;
      u64 seen;
      bool ok = true;
      {
        const u64 t0 = __builtin_amdgcn_s_memrealtime();
        for (;;) {
          seen = __ballot(ld_sys(me + fo0) >= call);
          if ((seen >> (idx >> 2)) & 1) break;
          if (UNI(ctl[C_FAIL]) || __builtin_amdgcn_s_memrealtime() - t0 > DEADLINE) { ctl[C_FAIL] = 1; ok = false; break; }
          __builtin_amdgcn_s_sleep(2);
        }
        asm volatile("" ::: "memory");
      }
      int iss = 0, known = 0;
      auto ready = [&](u32 t) -> bool {
        const u32 wd = t < NIN ? ctl[C_IN + (t >> 7)] >> ((t >> 2) & 31) : ctl[C_READY] >> ((t - NIN) >> 3);
        return (UNI(wd) & 1) || (t < 256 && ((seen >> (t >> 2)) & 1));
      };
      auto locate = [&](u32 t) {
        raw = 0;
        if (t < NIN) {
          const u32 id = import_id(t >> 2, q);
          so = par + (id < 64 ? P_EX + id * (2 * XBLK) : P_EW + (rank * 32 + (id - 64)) * WBLK) + (t & 3) * 8192;
        } else {
          const u32 i = (t - NIN) >> 3, w8 = (t - NIN) & 7;
          u32 s, c;
          tile_sc(i, s, c);
          raw = (UNI(ctl[C_RAWM + i]) >> w8) & 1;
          so = raw ? OOB : par + P_PK + (((rank * 4 + s) * 8 + c) * 8 + w8) * SUB;
        }
      };
      locate(z);
      STAMP(2);
      while (z < NTK) {
        u32 fail = 0;
        if (!ok && ready(z) && UNI(ctl[C_DONE + slot]) == 4 * (n / NSLOT)) {
          locate(z);
        } else if (!ok) {
          waitv0();
          ctl[C_LAND] = known = iss;
          while (UNI(ctl[C_DONE + slot]) != 4 * (n / NSLOT)) __builtin_amdgcn_s_sleep(1);
          while (!ready(z)) {
            if (UNI(ctl[C_FAIL])) { fail = 1; break; }
            __builtin_amdgcn_s_sleep(1);
          }
          locate(z);
          if (fail) so = OOB;
        }
        ctl[C_BASE + slot] = iss;
        ctl[C_ASG + slot] = ((n + 1) << 16) | (fail << 13) | (raw << 12) | z;
        LDS_AS char* ld = (LDS_AS char*)smem + slot * SLOTB;
        const u32 sc = so, zc = z;
        u32 tk;
        auto advance = [&]() {
          z = take(tk) - tbase;
          slot = slot + 1 == NSLOT ? 0 : slot + 1;
          if (z < NTK) {
            ok = ready(z) && UNI(ctl[C_DONE + slot]) == 4 * ((n + 1) / NSLOT);
            locate(z);
          }
        };
#define LEFT(N, k) do { asm volatile("s_waitcnt vmcnt(%0)" :: "i"(N) : "memory"); ctl[C_LAND] = max(known, iss + (k) - (N)); } while (0)
#define PULL(k, vofs, grp, imm) \
        asm volatile("s_waitcnt vmcnt(%0)" :: "i"(LWIN - 1) : "memory"); \
        __builtin_amdgcn_raw_ptr_buffer_load_lds(rq, ld + (grp) * 4096, 16, vofs, sc + (grp) * 4096, imm, AUX_SYS); \
        asm volatile("" ::: "memory"); \
        if (!(((k) + 2 + LWIN) & 1)) ctl[C_LAND] = max(known, iss + (k) + 1 - (LWIN - 1)); \
        if ((k) + 1 == LWIN) advance();
#define PULLX(vofs, grp, imm) __builtin_amdgcn_raw_ptr_buffer_load_lds(rq, ld + (grp) * 4096, 16, vofs, sc + (grp) * 4096, imm, AUX_SYS);
        if (zc < NIN) {
          constexpr int LWIN = LW1;
          asm volatile("s_waitcnt vmcnt(%0)" :: "i"(LWIN - 1) : "memory");
          tk = draw(me + F_T + q * 128);
          ctl[C_LAND] = max(known, iss - (LWIN - 1));
          PULL(0, v16, 0, 0) PULL(1, v16, 0, 1024) PULL(2, v16, 0, 2048) PULL(3, v16, 0, 3072)
          PULL(4, v16, 1, 0) PULL(5, v16, 1, 1024) PULL(6, v16, 1, 2048) PULL(7, v16, 1, 3072)
          iss += 9;
        } else if (zc < NTK - WIDE) {
          constexpr int LWIN = LW2;
          asm volatile("s_waitcnt vmcnt(%0)" :: "i"(LWIN - 1) : "memory");
          tk = draw(me + F_T + q * 128);
          ctl[C_LAND] = max(known, iss - (LWIN - 1));
          PULL(0, v16, 2, 0) PULL(1, v16, 2, 1024) PULL(2, vB, 2, 2048)
          PULL(3, v16, 0, 0) PULL(4, v16, 0, 1024) PULL(5, v16, 0, 2048) PULL(6, v16, 0, 3072)
          PULL(7, v16, 1, 0)
          if (z < NTK) {
            PULL(8, v16, 1, 1024) PULL(9, v16, 1, 2048) PULL(10, v16, 1, 3072)
          } else {
            PULLX(v16, 1, 1024) PULLX(v16, 1, 2048) PULLX(v16, 1, 3072)
          }
          iss += 12;
        } else {
          tk = draw(me + F_T + q * 128);
          PULLX(v16, 2, 0) PULLX(v16, 2, 1024) PULLX(vB, 2, 2048)
          PULLX(v16, 0, 0) PULLX(v16, 0, 1024) PULLX(v16, 0, 2048) PULLX(v16, 0, 3072)
          PULLX(v16, 1, 0) PULLX(v16, 1, 1024) PULLX(v16, 1, 2048) PULLX(v16, 1, 3072)
          iss += 12;
          LEFT(16, 0); LEFT(14, 0); LEFT(12, 0); LEFT(11, 0);
          advance();
        }
        ++n;
      }
      ctl[C_END] = n;
      LEFT(8, 0); LEFT(6, 0); LEFT(4, 0); LEFT(2, 0); LEFT(0, 0);
#undef PULL
#undef PULLX
#undef LEFT
      STAMP(6);
      STAMPV(7, twait & ~(1ull << 62));
      STAMPV(8, n);
      STAMPV(9, tz[0]); STAMPV(10, tz[1]); STAMPV(11, tz[2]); STAMPV(12, tz[3]); STAMPV(13, tz[4]);
    } else {
      if (wave == 1) {
        const u32 fo1 = F_S + (q * 96 + import_id(64 + (lane < NPULL - 64 ? lane : 0), q)) * 128;
        const u64 t0 = __builtin_amdgcn_s_memrealtime();
        bool inputs = false;
        for (;;) {
          const bool tmo = UNI(ld_agt(me + F_TMO)) == call;
          bool fin = false;
          if (!inputs) {
            const u32 f0 = ld_sys(me + fo0), f1 = ld_sys(me + fo1);
            const u64 m0 = __ballot(f0 >= call), m1 = __ballot(f1 >= call);
            if (lane < 3) ctl[C_IN + lane] = lane == 0 ? (u32)m0 : lane == 1 ? (u32)(m0 >> 32) : (u32)m1;
            inputs = (m0 & m1) == ~0ull;
          } else {
            const u32 l = lane & 31;
            const u64 of = __hip_atomic_load((const u64*)(me + F_O + (q * 32 + l) * 128), __ATOMIC_RELAXED, __HIP_MEMORY_SCOPE_SYSTEM);
            const u32 pf = ld_agt(me + F_P + (q * 32 + l) * 128);
            const u32 rd = (u32)__ballot((u32)(of >> 32) >= call && pf >= call);
            if (lane < 32) ctl[C_RAWM + lane] = (u32)of;
            asm volatile("s_waitcnt lgkmcnt(0)" ::: "memory");
            ctl[C_READY] = rd;
            fin = rd == 0xFFFFFFFFu;
          }
          if (tmo || UNI(ctl[C_FAIL]) || __builtin_amdgcn_s_memrealtime() - t0 > DEADLINE) {
            st_agt(me + F_TMO, call);
            ctl[C_FAIL] = 1;
            break;
          }
          if (fin) break;
          if (inputs) __builtin_amdgcn_s_sleep(20); else __builtin_amdgcn_s_sleep(WSLEEP);
        }
      }
      const float scale = P.sx[0] * P.sw[0];
      const u32 vo = (u32)(lane >> 4) * 4 * (NCOL * 4) + (lane & 15) * 16;
      auto peek = [&](u32 item) -> u32 {
        const u32 n = item >> 2, a = UNI(ctl[C_ASG + n % NSLOT]);
        return (a >> 16) == n + 1 ? a : 0u;
      };
      auto await = [&](u32 item) -> u32 {
        for (;;) {
          const u32 e = UNI(ctl[C_END]);
          const u32 a = peek(item);
          if (a || (item >> 2) >= e) return a;
          __builtin_amdgcn_s_sleep(1);
        }
      };
      auto landed = [&](u32 slot, u32 k) {
        const u32 need = UNI(ctl[C_BASE + slot]) + k;
        while ((int)(UNI(ctl[C_LAND]) - need) < 0) __builtin_amdgcn_s_sleep(1);
        asm volatile("" ::: "memory");
      };
      auto out_off = [&](u32 a, u32 item) -> u32 {
        const u32 z = (a & 0xFFFu) - NIN, w8 = z & 7;
        u32 s, c;
        tile_sc(z >> 3, s, c);
        return ((s * 128 + (w8 >> 2) * 64 + (item & 3) * 16) * NCOL + q * 2048 + c * 256 + (w8 & 3) * 64) * 4;
      };
      auto fetch = [&](u32 pb, u32 a, u32 item) {
        const u32 oo = ((a & 0xFFFu) >= NIN && !(a & 0x2000u)) ? out_off(a, item) : OOB;
        LDS_AS char* ld = (LDS_AS char*)smem + pb;
        __builtin_amdgcn_raw_ptr_buffer_load_lds(rout, ld, 16, vo, oo, 0, AUX_SYS);
        __builtin_amdgcn_raw_ptr_buffer_load_lds(rout, ld, 16, vo + (NCOL * 4 - 1024), oo, 1024, AUX_SYS);
        __builtin_amdgcn_raw_ptr_buffer_load_lds(rout, ld, 16, vo + 2 * (NCOL * 4 - 1024), oo, 2048, AUX_SYS);
        __builtin_amdgcn_raw_ptr_buffer_load_lds(rout, ld, 16, vo + 3 * (NCOL * 4 - 1024), oo, 3072, AUX_SYS);
      };
#define TICK(x)
      u32 pcur = LDS_PT + (wave - 1) * 8192;
      bool early = false;
      u32 item = lds_add(ctl + C_ITEM, 1u);
      u32 a = await(item);
      fetch(pcur, a, item);
      while (a) {
        const u32 n = item >> 2, m = item & 3, slot = n % NSLOT, zt = a & 0xFFFu;
        if (zt < NIN || (a & 0x3000u)) {
          if (zt < NIN) {
            if (!(a & 0x2000u)) {
              const u32 id = import_id(zt >> 2, q);
              const u32 dso = (id < 64 ? D_IX + q * (2 * MiB) + id * (2 * XBLK) : par + P_IW + (q * 32 + (id - 64)) * WBLK) + (zt & 3) * 8192 + m * 2048;
              landed(slot, 3 + 2 * m);
              const char* ls = smem + slot * SLOTB + m * 2048 + lane * 16;
              const u32x4 v0 = *(const u32x4*)ls, v1 = *(const u32x4*)(ls + 1024);
              st128(v0, rme, lane * 16, dso, AUX_AGT);
              st128(v1, rme, lane * 16 + 1024, dso, AUX_AGT);
              lds_inc(ctl + C_DONE + slot);
              waitv0();
              bump(me + F_I + (q * 96 + id) * 128);
            } else {
              lds_inc(ctl + C_DONE + slot);
            }
          } else {
            lds_inc(ctl + C_DONE + slot);
            waitv0();
            const u32 z = zt - NIN, oo = out_off(a, item);
            u32 s, c;
            tile_sc(z >> 3, s, c);
            const u32 so = par + P_RAW + (((rank * 4 + s) * 8 + c) * 8 + (z & 7)) * 16384 + m * 4096;
#pragma unroll
            for (int r = 0; r < 4; ++r) {
              u32x4 y = {0x7FC00000u, 0x7FC00000u, 0x7FC00000u, 0x7FC00000u};
              if (!(a & 0x2000u)) {
                const u32x4 v = __builtin_amdgcn_raw_buffer_load_b128(rq, lane * 16 + r * 1024, so, AUX_SYS);
                const u32x4 pt = *(const u32x4*)(smem + pcur + r * 1024 + lane * 16);
#pragma unroll
                for (int t = 0; t < 4; ++t) y[t] = __builtin_bit_cast(u32, (float)((int)v[t] + (int)pt[t]) * scale);
              }
              st128(y, rout, vo + r * (NCOL * 4), oo, AUX_AGT);
            }
          }
          item = lds_add(ctl + C_ITEM, 1u);
          a = await(item);
          pcur ^= 4096;
          fetch(pcur, a, item);
          early = false;
          continue;
        }
        const u32 oo = out_off(a, item);
        TICK(tA);
        const u32 item2 = lds_add(ctl + C_ITEM, 1u);
        u32 a2 = 0;
        const u32 base = UNI(ctl[C_BASE + slot]);
        auto until = [&](u32 need) {
          while ((int)(UNI(ctl[C_LAND]) - need) < 0) {
            if (!a2) {
              a2 = peek(item2);
              if (a2) fetch(pcur ^ 4096, a2, item2);
            }
            __builtin_amdgcn_s_sleep(1);
          }
          asm volatile("" ::: "memory");
        };
        until(base + 4);
        const char* ls = smem + slot * SLOTB;
        const u32* cw = (const u32*)(ls + 10240);
        const u32 lb = 7u * (lane >> 1);
        const u32 lp = __builtin_amdgcn_alignbit(cw[(lb >> 5) + 1], cw[lb >> 5], lb & 31u) & 127u;
        const u32x4 n0 = *(const u32x4*)(ls + 8192 + lane * 16), n1 = *(const u32x4*)(ls + 9216 + lane * 16);
        u32 total;
        const u32 wp = code_window(lp, lane, total);
        const u32 c0 = cw[wp >> 5], c1 = cw[(wp >> 5) + 1], c2 = cw[(wp >> 5) + 2];
        u64 B = ((u64)__builtin_amdgcn_alignbit(c2, c1, wp & 31u) << 32) | __builtin_amdgcn_alignbit(c1, c0, wp & 31u);
        if (lane & 1) B = rev64(B);
        B = (B << 1) | 1u;
        auto escs = [](u32 x) -> u32 { return __builtin_popcount(x & (x >> 1) & 0x44444444u); };
        u32 cnt = 0;
        if (m > 0) cnt += escs(n0[0]) + escs(n0[1]);
        if (m > 1) cnt += escs(n0[2]) + escs(n0[3]);
        if (m > 2) cnt += escs(n1[0]) + escs(n1[1]);
        u32 pos = 0, wd = (u32)B, pc = __builtin_popcount(wd);
        if (cnt >= pc) { cnt -= pc; wd = (u32)(B >> 32); pos = 32; }
#pragma unroll
        for (int wdt = 16; wdt >= 1; wdt >>= 1) {
          pc = __builtin_popcount(wd & ((1u << wdt) - 1u));
          if (cnt >= pc) { cnt -= pc; wd >>= wdt; pos += wdt; }
        }
        B >>= (pos & 63u);
        const u32 nd0 = m == 0 ? n0[0] : m == 1 ? n0[2] : m == 2 ? n1[0] : n1[2];
        const u32 nd1 = m == 0 ? n0[1] : m == 1 ? n0[3] : m == 2 ? n1[1] : n1[3];
        if (early && a2) asm volatile("s_waitcnt vmcnt(8)" ::: "memory");
        else if (early || a2) asm volatile("s_waitcnt vmcnt(4)" ::: "memory");
        else asm volatile("s_waitcnt vmcnt(0)" ::: "memory");
        TICK(tC);
        u32x4 pt[4];
#pragma unroll
        for (int r = 0; r < 4; ++r) pt[r] = *(const u32x4*)(smem + pcur + r * 1024 + lane * 16);
#pragma unroll
        for (int r = 0; r < 4; ++r) {
          const u32 nd = r < 2 ? nd0 : nd1, ne = nd & (nd >> 1);
#pragma unroll
          for (int t = 0; t < 4; ++t) {
            const int sh = 8 * t + 4 * (r & 1);
            const u32 esc = (u32)__builtin_amdgcn_sbfe((int)ne, sh + 2, 1);
            const u32 len = ffbl((u32)B ^ 1u) & esc;
            B >>= (len & 63u);
            const u32 h = __builtin_amdgcn_ubfe(nd, sh + 2, 2) + len + esc;
            pt[r][t] += ((u32)__builtin_amdgcn_sbfe((int)nd, sh, 2) ^ (h << 1)) << 16;
          }
        }
        asm volatile("" : "+v"(pt[0]), "+v"(pt[1]), "+v"(pt[2]), "+v"(pt[3]));
        until(base + 6 + 2 * m);
        TICK(tB);
        const u32x4 l0 = *(const u32x4*)(ls + m * 2048 + lane * 16), l1 = *(const u32x4*)(ls + m * 2048 + 1024 + lane * 16);
        asm volatile("s_waitcnt lgkmcnt(0)" :: "v"(l0), "v"(l1) : "memory");
        lds_inc(ctl + C_DONE + slot);
#pragma unroll
        for (int r = 0; r < 4; ++r) {
          u32x4 y;
#pragma unroll
          for (int t = 0; t < 4; ++t) {
            const u32 lo = r < 2 ? l0[(r & 1) * 2 + (t >> 1)] : l1[(r & 1) * 2 + (t >> 1)];
            y[t] = __builtin_bit_cast(u32, (float)(int)(pt[r][t] + ((t & 1) ? lo >> 16 : lo & 0xFFFFu)) * scale);
          }
          st128(y, rout, vo + r * (NCOL * 4), oo, OAUX);
        }
        TICK(tD);
        item = item2;
        a = a2;
        early = a2 != 0;
        pcur ^= 4096;
        if (!a2) {
          a = await(item);
          fetch(pcur, a, item);
        }
      }
      STAMPV(9, tA); STAMPV(10, tB); STAMPV(11, tC); STAMPV(12, tD);
    }
    STAMP(4);
    return;
  }

  const u32 vo = ((u32)(wave >> 2) * 64 + (lane >> 4) * 4) * (NCOL * 4) + ((wave & 3) * 64 + (lane & 15) * 4) * 4;
  i32x4 acc[4][4];
  for (int job = 0; job < 4; ++job) {
    const bool partial = job < 2;
    u32 t;
    if (job == 0) t = gi;
    else if (job == 1) { if (gi < NGEM - 24) continue; t = gi + 24; }
    else if (job == 2) { if (gi >= 192) break; t = gi; }
    else { if (gi >= 64) break; t = 192 + gi; }
    u32 g, c, cg_self, foff, s, dst, oi, tgt = call;
    int dskip, nk;
    bool act, own = true;
    if (partial) {
      const u32 r = t >> 3;
      oi = (r / 7) * 8 + (t & 7);
      tile_sc(oi, s, c);
      dst = (rank + 1 + r % 7) & 7u;
      g = rank * 4 + s;
      cg_self = dst * 8 + c;
      dskip = 8; nk = 4;
      act = lane < 6;
      foff = lane < 4 ? F_LW + (cg_self * 4 + lane) * 128 : F_S + (rank * 96 + g * 2 + (lane & 1)) * 128;
    } else {
      u32 pd;
      if (t < 128) { oi = t >> 3; pd = t & 7; }
      else { oi = (t >> 6) * 8 + (t & 7); pd = (t >> 3) & 7; }
      tile_sc(oi, s, c);
      dst = (rank + 1 + pd) & 7u;
      g = dst * 4 + s;
      cg_self = rank * 8 + c;
      dskip = dst == rank ? 8 : (int)((dst - rank) & 7u);
      nk = dst == rank ? 32 : 28;
      const u32 q = lane / 6, e = lane % 6;
      act = lane < 48 && (q != dst || dst == rank);
      const u32 id = e < 4 ? 64 + c * 4 + e : g * 2 + (e & 1);
      foff = (q == rank ? F_S : F_I) + ((q & 7) * 96 + id) * 128;
      own = q == rank;
      tgt = own ? call : call * 16;
    }
#pragma unroll
    for (int m = 0; m < 4; ++m)
#pragma unroll
      for (int tt = 0; tt < 4; ++tt) acc[m][tt] = (i32x4){0, 0, 0, 0};
    bool good = true;
    for (int part = 0; part < (partial ? 1 : 2) && good; ++part) {
      good = wg_wait(smem, me, foff, act && own == (part == 0), tgt, call, wave);
      if (part || partial) STAMP(2 + job * 3);
      if (good) gemm_tile(smem, rme, (int)rank, par, g, cg_self, c, dskip, part ? 4 : 0, part ? nk : 4, wave, lane, acc);
    }
    STAMP(3 + job * 3);
    if (partial) {
      if (good) {
        const u32 oo = (s * 128 * NCOL + dst * 2048 + c * 256) * 4;
#pragma unroll
        for (int m = 0; m < 4; ++m)
#pragma unroll
          for (int r = 0; r < 4; ++r) {
            const u32x4 y = {(u32)acc[m][0][r], (u32)acc[m][1][r], (u32)acc[m][2][r], (u32)acc[m][3][r]};
            st128(y, rout, vo + (m * 16 + r) * (NCOL * 4), oo, AUX_AGT);
          }
        waitv0();
        bar();
        if (tid == 0) st_agt(me + F_P + (dst * 32 + oi) * 128, call);
      }
    } else if (dst == rank) {
      const float scale = P.sx[0] * P.sw[0];
      const u32 oo = (s * 128 * NCOL + rank * 2048 + c * 256) * 4;
#pragma unroll
      for (int m = 0; m < 4; ++m)
#pragma unroll
        for (int r = 0; r < 4; ++r) {
          u32x4 y;
#pragma unroll
          for (int tt = 0; tt < 4; ++tt) y[tt] = good ? __builtin_bit_cast(u32, (float)acc[m][tt][r] * scale) : 0x7FC00000u;
          st128(y, rout, vo + (m * 16 + r) * (NCOL * 4), oo, AUX_AGT);
        }
    } else if (good) {
      if (tid == 0) ctl[C_RAW] = 0;
      bar();
      const u32 sub = (g * 8 + c) * 8 + wave;
      const u32 so = par + P_PK + sub * SUB;
#pragma unroll
      for (int m = 0; m < 4; ++m)
#pragma unroll
        for (int hf = 0; hf < 2; ++hf) {
          const u32x4 lo = {perm(acc[m][1][2 * hf], acc[m][0][2 * hf], 0x05040100u), perm(acc[m][3][2 * hf], acc[m][2][2 * hf], 0x05040100u),
                            perm(acc[m][1][2 * hf + 1], acc[m][0][2 * hf + 1], 0x05040100u), perm(acc[m][3][2 * hf + 1], acc[m][2][2 * hf + 1], 0x05040100u)};
          st128(lo, rme, lane * 16 + m * 2048 + hf * 1024, so, AUX_SYS);
        }
      LDS_AS u32* cw = (LDS_AS u32*)((LDS_AS char*)smem + wave * 512);
      cw[lane] = 0;
      if (lane < 8) cw[64 + lane] = 0;
      u64 B = 1;
      u32 any = 0;
      u16x2 sum2 = {0, 0};
      u32x4 nb[2];
#pragma unroll
      for (int m = 3; m >= 0; --m) {
        u32 hb[4];
#pragma unroll
        for (int r = 3; r >= 0; --r) {
          u32 nv[2];
#pragma unroll
          for (int pr = 1; pr >= 0; --pr) {
            const u32 qv = perm(acc[m][2 * pr + 1][r], acc[m][2 * pr][r], 0x07060302u);
            const s16x2 q2 = __builtin_bit_cast(s16x2, qv);
            const u32 sg = __builtin_bit_cast(u32, q2 >> (s16x2){15, 15});
            const u16x2 h = __builtin_bit_cast(u16x2, __builtin_bit_cast(u32, q2 >> (s16x2){1, 1}) ^ sg);
            const u16x2 len = __builtin_elementwise_sub_sat(h, (u16x2){2, 2});
            const u32 lw = __builtin_bit_cast(u32, len);
            B = (B << ((lw >> 16) & 63u)) | 1u;
            B = (B << (lw & 63u)) | 1u;
            sum2 += len;
            any |= lw;
            const u32 t2 = __builtin_bit_cast(u32, __builtin_elementwise_min(h, (u16x2){3, 3})) << 2;
            nv[pr] = (qv & 0x00010001u) | (sg & 0x00020002u) | t2;
          }
          hb[r] = perm(nv[1], nv[0], 0x06040200u);
        }
        nb[m >> 1][(m & 1) * 2] = hb[0] | (hb[1] << 4);
        nb[m >> 1][(m & 1) * 2 + 1] = hb[2] | (hb[3] << 4);
      }
      const u32 sum = (u32)sum2[0] + sum2[1];
      const u32 lp = sum + __builtin_amdgcn_update_dpp(0u, sum, 0xB1, 0xF, 0xF, false);
      u32 total;
      const u32 wp = code_window(lp, lane, total);
      if (__any(sum > 63u || (any & 0xFFE0FFE0u)) || total > 2048u) {
        if (lane == 0) __hip_atomic_fetch_or((u32*)(smem + LDS_CTL) + C_RAW, 1u << wave, __ATOMIC_RELAXED, __HIP_MEMORY_SCOPE_WORKGROUP);
#pragma unroll
        for (int m = 0; m < 4; ++m)
#pragma unroll
          for (int r = 0; r < 4; ++r) {
            const u32x4 y = {(u32)acc[m][0][r], (u32)acc[m][1][r], (u32)acc[m][2][r], (u32)acc[m][3][r]};
            st128(y, rme, lane * 16 + (m * 4 + r) * 1024, par + P_RAW + sub * 16384, AUX_SYS);
          }
      } else {
        st128(nb[0], rme, lane * 16 + 8192, so, AUX_SYS);
        st128(nb[1], rme, lane * 16 + 9216, so, AUX_SYS);
        const u64 V = (lane & 1) ? rev64(B >> 1) : B >> 1;
        const u32 sf = wp & 31u, lb = 7u * (lane >> 1);
        const u64 L = (u64)((lane & 1) ? 0u : lp) << (lb & 31u);
        __hip_atomic_fetch_or(cw + (lb >> 5), (u32)L, __ATOMIC_RELAXED, __HIP_MEMORY_SCOPE_WAVEFRONT);
        __hip_atomic_fetch_or(cw + (lb >> 5) + 1, (u32)(L >> 32), __ATOMIC_RELAXED, __HIP_MEMORY_SCOPE_WAVEFRONT);
        __hip_atomic_fetch_or(cw + (wp >> 5), (u32)V << sf, __ATOMIC_RELAXED, __HIP_MEMORY_SCOPE_WAVEFRONT);
        __hip_atomic_fetch_or(cw + (wp >> 5) + 1, (u32)((V << sf) >> 32), __ATOMIC_RELAXED, __HIP_MEMORY_SCOPE_WAVEFRONT);
        __hip_atomic_fetch_or(cw + (wp >> 5) + 2, (u32)(((V >> 32) << sf) >> 32), __ATOMIC_RELAXED, __HIP_MEMORY_SCOPE_WAVEFRONT);
        asm volatile("s_waitcnt lgkmcnt(0)" ::: "memory");
        __builtin_amdgcn_raw_buffer_store_b32(*(volatile LDS_AS u32*)(cw + lane), rme, lane * 4 + 10240, so, AUX_SYS);
      }
      waitv0();
      bar();
      if (tid == 0) {
        const u64 f = ((u64)call << 32) | ctl[C_RAW];
        __hip_atomic_store((u64*)(P.ws[dst] + F_O + (rank * 32 + oi) * 128), f, __ATOMIC_RELAXED, __HIP_MEMORY_SCOPE_SYSTEM);
      }
    }
    STAMP(4 + job * 3);
  }
}

}

extern "C" void kernel_launch(int rank, int nranks, ncclComm_t comm,
                              hipStream_t stream,
                              void* const* d_in, const int* in_sizes, int n_in,
                              void* d_out, int out_size,
                              const int* in_dtypes, int out_dtype,
                              void* const* d_out_all,
                              void* const* d_ws_all, size_t ws_bytes) {
  if (nranks != NR || (unsigned)rank >= (unsigned)NR || n_in < 4 || ws_bytes < (size_t)WS_NEED + MiB ||
      (size_t)out_size * 4 != OUT_BYTES || in_sizes[0] != 4096 * KSL || in_sizes[1] != KSL * NCOL) {
    fprintf(stderr, "kernel_launch: unexpected shapes\n");
    return;
  }
  Params p;
  p.x = (const signed char*)d_in[0];
  p.w = (const signed char*)d_in[1];
  p.sx = (const float*)d_in[2];
  p.sw = (const float*)d_in[3];
  p.out = (float*)d_out;
  for (int i = 0; i < NR; ++i) p.ws[i] = (char*)d_ws_all[i];
  p.me = p.ws[rank];
  p.ws_bytes = WS_NEED + MiB;
  p.rank = rank;
  gemm_rs_kernel<<<NWG, NTHR, 0, stream>>>(p);
  HIP_CHECK(hipGetLastError());
}
```

```cpp
#include <hip/hip_runtime.h>
#include <rccl/rccl.h>
#include <cstdio>

#define HIP_CHECK(expr) do { hipError_t _e = (expr); if (_e != hipSuccess) { \
    fprintf(stderr, "HIP error %s at %s:%d\n", hipGetErrorString(_e), __FILE__, __LINE__); \
    return; } } while (0)

typedef unsigned u32;
typedef unsigned long long u64;
typedef u32 u32x4 __attribute__((ext_vector_type(4)));
typedef u32 u32x2 __attribute__((ext_vector_type(2)));
typedef int i32x4 __attribute__((ext_vector_type(4)));
typedef short s16x2 __attribute__((ext_vector_type(2)));
typedef unsigned short u16x2 __attribute__((ext_vector_type(2)));
typedef __amdgpu_buffer_rsrc_t rsrc_t;
#define LDS_AS __attribute__((address_space(3)))
#define UNI(x) __builtin_amdgcn_readfirstlane(x)

namespace {

constexpr int NR = 8;
constexpr int KSL = 512;
constexpr int NCOL = 16384;
constexpr int NWG = 256, NTHR = 512;
constexpr int WPL = 8;
constexpr int NSTR = 7 * WPL;
constexpr int NGEM = NWG - NSTR;
#define EXPORT_DELAY 24
#define GAUX 17
#define RDMASK 0x0B6Du
#define DMAMASK 0x2082u
#define NPUSH 7
#define OAUX 16
#define WSLEEP 4
#define BROT 0
#define GSLEEP 32
#define OSLEEP 20
#define V_SC 0
#define PLAN_AT 150
#define LAG_MIN 6
#define EXT_MAX 12
#define LW1 8
#define LW2 7

constexpr u32 KiB = 1024u, MiB = 1024u * 1024u;
constexpr u32 F_CNT = 0;
constexpr u32 F_TMO = 32 * KiB;
constexpr u32 F_S = 64 * KiB;
constexpr u32 F_I = 160 * KiB;
constexpr u32 F_LW = 256 * KiB;
constexpr u32 F_O = 288 * KiB;
constexpr u32 F_P = 320 * KiB;
constexpr u32 F_T = 352 * KiB;
constexpr u32 F_PLAN = 354 * KiB;
constexpr u32 XBLK = 16 * KiB, WBLK = 32 * KiB;
constexpr u32 D_PAR = 1 * MiB;
constexpr u32 P_EX = 0;
constexpr u32 P_EW = 2 * MiB;
constexpr u32 SUB = 10496;
constexpr u32 P_PK = 10 * MiB;
constexpr u32 P_RAW = 32 * MiB;
constexpr u32 P_IW = 64 * MiB;
constexpr u32 P_IX = 72 * MiB;
constexpr u32 PAR_BYTES = 88 * MiB;
constexpr u32 WS_NEED = D_PAR + 2 * PAR_BYTES;
constexpr u32 OUT_BYTES = 512u * NCOL * 4u;

constexpr u32 STAGE = XBLK + WBLK;
constexpr u32 LDS_CTL = 3 * STAGE;
constexpr u32 LDS_BYTES = LDS_CTL + 768;
constexpr u32 NSLOT = 8, SLOTB = 11264;
constexpr u32 LDS_PT = NSLOT * SLOTB;
static_assert(LDS_PT + 7 * 8192 <= LDS_CTL, "LDS");
constexpr int C_RAW = 8, C_IN = 12, C_ITEM = 17, C_END = 18, C_READY = 19, C_FAIL = 20,
              C_LAND = 21, C_NEAR = 22, C_HAVE = 23, C_RAWM = 24, C_PLAN = 56, C_TAB = 128, C_ASG = 64, C_BASE = 80, C_DONE = 96;

constexpr u64 DEADLINE = 150000000ull;
constexpr int AUX_SYS = 17, AUX_AGT = 16;
constexpr u32 OOB = 0x70000000u;

struct Params {
  const signed char* x;
  const signed char* w;
  const float* sx;
  const float* sw;
  float* out;
  char* me;
  char* ws[NR];
  u32 ws_bytes;
  int rank;
};

__device__ __forceinline__ rsrc_t mk_rsrc(const void* p, u32 bytes) {
  return __builtin_amdgcn_make_buffer_rsrc((void*)p, (short)0, (int)bytes, 0x00020000);
}
#define st128(v, r, vo, so, aux) __builtin_amdgcn_raw_buffer_store_b128((v), (r), (vo) + (so), 0, (aux))
__device__ __forceinline__ void waitv0() { asm volatile("s_waitcnt vmcnt(0)" ::: "memory"); }
__device__ __forceinline__ void bar() {
  asm volatile("s_waitcnt lgkmcnt(0)" ::: "memory");
  __builtin_amdgcn_s_barrier();
  asm volatile("" ::: "memory");
}
__device__ __forceinline__ u32 ld_sys(const char* p) {
  return __hip_atomic_load((const u32*)p, __ATOMIC_RELAXED, __HIP_MEMORY_SCOPE_SYSTEM);
}
__device__ __forceinline__ u32 ld_agt(const char* p) {
  return __hip_atomic_load((const u32*)p, __ATOMIC_RELAXED, __HIP_MEMORY_SCOPE_AGENT);
}
__device__ __forceinline__ void st_sys(char* p, u32 v) {
  __hip_atomic_store((u32*)p, v, __ATOMIC_RELAXED, __HIP_MEMORY_SCOPE_SYSTEM);
}
__device__ __forceinline__ void st_agt(char* p, u32 v) {
  __hip_atomic_store((u32*)p, v, __ATOMIC_RELAXED, __HIP_MEMORY_SCOPE_AGENT);
}

#define STAMP(slot) do {} while (0)
#define STAMPV(slot, v) do {} while (0)

__device__ __forceinline__ bool wave_wait(const char* me, u32 off, bool act, u32 tgt, u32 call) {
  bool ok = !act || ld_sys(me + off) >= tgt;
  if (__all(ok)) return true;
  const u64 t0 = __builtin_amdgcn_s_memrealtime();
  for (;;) {
    __builtin_amdgcn_s_sleep(GSLEEP);
    if (!ok) ok = ld_sys(me + off) >= tgt;
    if (__all(ok)) return true;
    if (UNI(ld_agt(me + F_TMO)) == call || __builtin_amdgcn_s_memrealtime() - t0 > DEADLINE) {
      st_agt((char*)me + F_TMO, call);
      return false;
    }
  }
}

__device__ __forceinline__ bool wg_wait(char* smem, const char* me, u32 off, bool act, u32 tgt, u32 call, int wave) {
  volatile LDS_AS u32* word = (volatile LDS_AS u32*)((LDS_AS char*)smem + LDS_CTL);
  if (wave == 0) {
    bool r = wave_wait(me, off, act, tgt, call);
    r = r && UNI(ld_agt(me + F_TMO)) != call;
    *word = r ? 1u : 2u;
  }
  bar();
  const u32 r = UNI(*word);
  bar();
  return r == 1u;
}

__device__ __forceinline__ u32 ffbl(u32 x) {
  u32 r;
  asm("v_ffbl_b32 %0, %1" : "=v"(r) : "v"(x));
  return r;
}
__device__ __forceinline__ u32 wave_scan(u32 x) {
  x += __builtin_amdgcn_update_dpp(0u, x, 0x111, 0xF, 0xF, false);
  x += __builtin_amdgcn_update_dpp(0u, x, 0x112, 0xF, 0xF, false);
  x += __builtin_amdgcn_update_dpp(0u, x, 0x114, 0xF, 0xF, false);
  x += __builtin_amdgcn_update_dpp(0u, x, 0x118, 0xF, 0xF, false);
  x += __builtin_amdgcn_update_dpp(0u, x, 0x142, 0xA, 0xF, false);
  x += __builtin_amdgcn_update_dpp(0u, x, 0x143, 0xC, 0xF, false);
  return x;
}
__device__ __forceinline__ u32 code_window(u32 lp, int lane, u32& total) {
  const u32 end = 224u + wave_scan((lane & 1) ? 0u : lp);
  total = __builtin_amdgcn_readlane(end, 63);
  return (lane & 1) ? end - 64u : end - lp;
}
__device__ __forceinline__ u64 rev64(u64 x) {
  return ((u64)__builtin_bitreverse32((u32)x) << 32) | __builtin_bitreverse32((u32)(x >> 32));
}
__device__ __forceinline__ u32 perm(u32 hi, u32 lo, u32 sel) { return __builtin_amdgcn_perm(hi, lo, sel); }

__device__ __forceinline__ void gemm_tile(char* smem, rsrc_t rws, int rank, u32 par, u32 g, u32 cg_self, u32 c,
                                          int dskip, int k0, int nk, int wave, int lane, i32x4 (&acc)[4][4]) {
  const u32 va = (u32)wave * 2048u + (u32)lane * 16u;
  const u32 vb = (u32)wave * 4096u + (u32)lane * 16u;
  u32 sa, sb;
  auto locate = [&](int kb) {
    const int qi = kb >> 2;
    const u32 kc = kb & 3;
    const int d = qi + (qi >= dskip ? 1 : 0);
    const u32 q = (u32)(rank + d) & 7u;
    sa = (d == 0 ? par + P_EX : par + P_IX + q * (2 * MiB)) + (g * 4 + kc) * XBLK;
    sb = (d == 0 ? par + P_EW + cg_self * (4 * WBLK) : par + P_IW + (q * 8 + c) * (4 * WBLK)) + kc * WBLK;
    if (kb >= nk) sa = sb = OOB;
  };
  auto dma = [&](int n, u32 buf) {
    LDS_AS char* la = (LDS_AS char*)smem + buf * STAGE + wave * 2048;
    LDS_AS char* lb = (LDS_AS char*)smem + buf * STAGE + XBLK + wave * 4096;
    if (n == 0) __builtin_amdgcn_raw_ptr_buffer_load_lds(rws, la, 16, va, sa, 0, GAUX);
    if (n == 1) __builtin_amdgcn_raw_ptr_buffer_load_lds(rws, la, 16, va, sa, 1024, GAUX);
    if (n == 2) __builtin_amdgcn_raw_ptr_buffer_load_lds(rws, lb, 16, vb, sb, 0, GAUX);
    if (n == 3) __builtin_amdgcn_raw_ptr_buffer_load_lds(rws, lb, 16, vb, sb, 1024, GAUX);
    if (n == 4) __builtin_amdgcn_raw_ptr_buffer_load_lds(rws, lb, 16, vb, sb, 2048, GAUX);
    if (n == 5) __builtin_amdgcn_raw_ptr_buffer_load_lds(rws, lb, 16, vb, sb, 3072, GAUX);
  };
  const u32 ra = (u32)(wave >> 2) * 4096u + (u32)lane * 16u;
  const u32 rb = XBLK + (u32)(wave & 3) * 4096u + (u32)lane * 16u;
  i32x4 f0[8], f1[8];
  auto rd = [&](i32x4 (&f)[8], int n, u32 buf, int ks) {
    const int j = n == 0 ? 0 : (n < 5 ? n + 3 : n - 4);
    const char* p = smem + buf * STAGE + (j < 4 ? ra + ks * 8192 + j * 1024 : rb + ks * 16384 + (j - 4) * 1024);
    f[j] = *(const i32x4*)p;
  };
  auto half = [&](i32x4 (&fc)[8], i32x4 (&fn)[8], u32 rbuf, int rks, int d0, u32 dbuf) {
#pragma unroll
    for (int i = 0; i < 16; ++i) {
      acc[i >> 2][i & 3] = __builtin_amdgcn_mfma_i32_16x16x64_i8(fc[i >> 2], fc[4 + (i & 3)], acc[i >> 2][i & 3], 0, 0, 0);
      if ((RDMASK >> i) & 1) rd(fn, __builtin_popcount(RDMASK & ((1u << i) - 1u)), rbuf, rks);
      if ((DMAMASK >> i) & 1) dma(d0 + __builtin_popcount(DMAMASK & ((1u << i) - 1u)), dbuf);
      __builtin_amdgcn_sched_barrier(0);
    }
  };
  locate(k0);
#pragma unroll
  for (int n = 0; n < 6; ++n) dma(n, 0);
  locate(k0 + 1);
#pragma unroll
  for (int n = 0; n < 6; ++n) dma(n, 1);
  locate(k0 + 2);
#pragma unroll
  for (int n = 0; n < 3; ++n) dma(n, 2);
  asm volatile("s_waitcnt vmcnt(9)" ::: "memory");
  bar();
#pragma unroll
  for (int n = 0; n < 8; ++n) rd(f0, n, 0, 0);
  u32 buf = 0;
  for (int kb = k0; kb < nk; ++kb) {
    const u32 b1 = buf == 2 ? 0u : buf + 1, b2 = buf == 0 ? 2u : buf - 1;
    __builtin_amdgcn_sched_barrier(0);
    half(f0, f1, buf, 1, 3, b2);
    locate(kb + 3);
    asm volatile("s_waitcnt vmcnt(6)" ::: "memory");
    bar();
    __builtin_amdgcn_sched_barrier(0);
    half(f1, f0, b1, 0, 0, buf);
    buf = b1;
  }
  waitv0();
}

constexpr u32 NPULL = 88 - NPUSH;
constexpr u32 NIN = NPULL * 4, CUT_MAX = 28;
constexpr u32 X3 = NPULL - 14;
__device__ __forceinline__ u32 import_id(u32 o, u32 q, u32 rank) {
  if (o >= 14) {
    o += NPUSH - 4;
    if (o < 18) return 64 + 4 + (o - 14);
    if (o >= 32 && o < 56) return 64 + 8 + (o - 32);
  }
  u32 s, e;
  if (o < 14) { s = 0; e = o; }
  else if (o < 32) { s = 1; e = o - 18; }
  else if (o < 70) { s = 2; e = o - 56; }
  else { s = 3; e = o - 70 + 2 * ((rank - q - 1) & 7u); e -= e >= 14 ? 14 : 0; }
  u32 p = e >> 1;
  p += (p >= q) ? 1 : 0;
  return (4 * p + s) * 2 + (e & 1);
}
__device__ __forceinline__ void tile_sc(u32 i, u32& s, u32& c) {
  if (i < 4) { s = i >> 1; c = i & 1; }
  else if (i < 16) { s = i & 1; c = i >> 1; }
  else { s = i >> 3; c = i & 7; }
}
__device__ __forceinline__ u32 draw(char* p) {
  u32 t;
  u64 save;
  asm volatile("s_mov_b64 %1, exec\n\ts_mov_b64 exec, 1\n\tglobal_atomic_add %0, %2, %3, off sc0\n\ts_mov_b64 exec, %1"
               : "=&v"(t), "=&s"(save) : "v"(p), "v"(1u) : "memory");
  return t;
}
__device__ __forceinline__ u32 take(u32 t) {
  u32 r;
  asm volatile("s_nop 0\n\tv_readfirstlane_b32 %0, %1\n\ts_nop 4" : "=s"(r) : "v"(t));
  return r;
}
__device__ __forceinline__ u32 lds_add(volatile LDS_AS u32* p, u32 x) {
  u32 t;
  u64 save;
  asm volatile("s_mov_b64 %1, exec\n\ts_mov_b64 exec, 1\n\tds_add_rtn_u32 %0, %2, %3\n\ts_waitcnt lgkmcnt(0)\n\ts_mov_b64 exec, %1"
               : "=&v"(t), "=&s"(save) : "v"((u32)(size_t)p), "v"(x) : "memory");
  return UNI(t);
}
__device__ __forceinline__ u32 peek_word(char* p) {
  u32 t;
  asm volatile("global_atomic_add %0, %1, %2, off sc0\n\ts_waitcnt vmcnt(0)" : "=&v"(t) : "v"(p), "v"(0u) : "memory");
  return t;
}
__device__ __forceinline__ void lds_inc(volatile LDS_AS u32* p) {
  u64 save;
  asm volatile("s_mov_b64 %0, exec\n\ts_mov_b64 exec, 1\n\tds_add_u32 %1, %2\n\ts_mov_b64 exec, %0"
               : "=&s"(save) : "v"((u32)(size_t)p), "v"(1u) : "memory");
}
__device__ __forceinline__ void bump(char* p) {
  u64 save;
  asm volatile("s_mov_b64 %0, exec\n\ts_mov_b64 exec, 1\n\tglobal_atomic_add %1, %2, off\n\ts_mov_b64 exec, %0"
               : "=&s"(save) : "v"(p), "v"(1u) : "memory");
}

__global__ void __launch_bounds__(NTHR, 2) gemm_rs_kernel(Params P) {
  __shared__ __attribute__((aligned(16))) char smem[LDS_BYTES];
  const int tid = threadIdx.x, lane = tid & 63;
  const int wave = UNI(tid >> 6);
  const u32 b = blockIdx.x;
  const u32 rank = (u32)P.rank & 7u;
  char* const me = P.me;
  asm volatile("" :: "s"(P.x), "s"(P.w), "s"(me), "s"(P.out), "s"(P.ws_bytes), "s"(P.rank));
  const rsrc_t rme = mk_rsrc(me, P.ws_bytes);
  const rsrc_t rout = mk_rsrc(P.out, OUT_BYTES);
  volatile LDS_AS u32* ctl = (volatile LDS_AS u32*)((LDS_AS char*)smem + LDS_CTL);
  STAMP(0);

  const u32 gi = b - NSTR;
  const u32 cnt = ld_agt(me + F_CNT + b * 128);
  if (b < NSTR) {
    if (tid < 192) ctl[tid] = tid == C_END ? 0xFFFFFFFFu : 0u;
    const u32 call = UNI(cnt) + 1;
    bar();
    if (tid == 0) st_agt(me + F_CNT + b * 128, call);
  }
  for (int e = 0; e < 2 && b >= NSTR; ++e) {
    u32 wp = 0xFFFFu, xp = 0xFFFFu, v = 0xFFFFu;
    if (e == 0) {
      if (gi < 64) {
        wp = (gi >> 3) * 32 + (gi & 7);
        if ((gi & 7) >= NPUSH) __builtin_amdgcn_s_sleep(EXPORT_DELAY);
      } else if (gi < 80) {
        xp = ((gi - 64) >> 1) * 8 + (gi & 1);
      } else {
        v = gi - 80;
        __builtin_amdgcn_s_sleep(EXPORT_DELAY);
      }
    } else if (gi >= 80) {
      v = gi + 40;
    }
    if (v < 16) xp = (v >> 1) * 8 + 2 + (v & 1);
    else if (v < 208) wp = ((v - 16) / 24) * 32 + 8 + (v - 16) % 24;
    else if (v < 240) xp = ((v - 208) >> 2) * 8 + 4 + (v & 3);
    const u32 vs = wave * 4096 + lane * 16;
    if (wp != 0xFFFFu) {
      const u32 cg = wp >> 2, kc = wp & 3, dj = (cg >> 3) & 7;
      u32 d[16];
      const rsrc_t rw = mk_rsrc(P.w, (u32)KSL * NCOL);
      const u32 vw = (kc * 128 + (wave >> 2) * 64 + (lane >> 4) * 16) * NCOL + cg * 256 + (wave & 3) * 64 + (lane & 15) * 4;
#pragma unroll
      for (int r = 0; r < 16; ++r) d[r] = __builtin_amdgcn_raw_buffer_load_b32(rw, vw, r * NCOL, 0);
      char* const peer = P.ws[dj];
      const u32 call = UNI(cnt) + 1, par = D_PAR + (call & 1) * PAR_BYTES;
      const bool push = (wp & 31) < NPUSH && dj != rank;
      u32x4 o[4];
#pragma unroll
      for (int r4 = 0; r4 < 4; ++r4) {
        const u32 t0 = perm(d[4 * r4 + 1], d[4 * r4], 0x05010400u), t1 = perm(d[4 * r4 + 1], d[4 * r4], 0x07030602u);
        const u32 t2 = perm(d[4 * r4 + 3], d[4 * r4 + 2], 0x05010400u), t3 = perm(d[4 * r4 + 3], d[4 * r4 + 2], 0x07030602u);
        o[0][r4] = perm(t2, t0, 0x05040100u);
        o[1][r4] = perm(t2, t0, 0x07060302u);
        o[2][r4] = perm(t3, t1, 0x05040100u);
        o[3][r4] = perm(t3, t1, 0x07060302u);
      }
      if (push) {
        const rsrc_t rj = mk_rsrc(peer, P.ws_bytes);
#pragma unroll
        for (int t = 0; t < 4; ++t) st128(o[t], rj, vs + t * 1024, par + P_IW + (rank * 32 + (wp & 31)) * WBLK, AUX_SYS);
      }
#pragma unroll
      for (int t = 0; t < 4; ++t) st128(o[t], rme, vs + t * 1024, par + P_EW + wp * WBLK, AUX_SYS);
      waitv0();
      bar();
      if (tid == 0) st_sys(peer + (push ? F_I : F_S) + (rank * 96 + 64 + (wp & 31)) * 128, push ? call * 4 : call);
      if (tid == 64) st_sys(me + F_LW + wp * 128, call);
    } else if (xp != 0xFFFFu) {
      u32x4 o[4];
      const rsrc_t rx = mk_rsrc(P.x, 4096u * KSL);
      const u32 vx = ((xp >> 1) * 128 + (wave & 1) * 64 + (lane & 15)) * KSL + ((xp & 1) * 2 + (wave >> 2)) * 128 + ((wave >> 1) & 1) * 64 + (lane >> 4) * 16;
#pragma unroll
      for (int u = 0; u < 4; ++u) o[u] = __builtin_amdgcn_raw_buffer_load_b128(rx, vx, u * 16 * KSL, 0);
      const u32 call = UNI(cnt) + 1, par = D_PAR + (call & 1) * PAR_BYTES;
#pragma unroll
      for (int t = 0; t < 4; ++t) st128(o[t], rme, vs + t * 1024, par + P_EX + xp * (2 * XBLK), AUX_SYS);
      waitv0();
      bar();
      if (lane == 0) st_sys(P.ws[wave] + F_S + (rank * 96 + xp) * 128, call);
    }
    if (e == 0 && tid == 64) st_agt(me + F_CNT + b * 128, cnt + 1);
  }
  const u32 call = UNI(cnt) + 1, par = D_PAR + (call & 1) * PAR_BYTES;
  STAMP(1);

  if (b < NSTR) {
    const u32 q = (rank + 1 + b % 7) & 7u, idx = b / 7;
    const rsrc_t rq = mk_rsrc(P.ws[q], P.ws_bytes);
    const u32 fo0 = F_S + (q * 96 + import_id(lane, q, rank)) * 128;
    if (wave == 0) {
      const u32 v16 = lane * 16, vB = lane < 16 ? lane * 16 : OOB;
      if (lane == 0) __hip_atomic_fetch_max((u64*)(me + F_T + q * 128), (u64)call << 32, __ATOMIC_RELAXED, __HIP_MEMORY_SCOPE_AGENT);
      u32 t = idx, z = idx, lk = q, xe = 0, n = 0, so = 0, slot = 0, raw = 0;
      u64 seen;
      bool ok = true, slot_free = true;
      {
        const u64 t0 = __builtin_amdgcn_s_memrealtime();
        for (;;) {
          seen = __ballot(ld_sys(me + fo0) >= call);
          if ((seen >> (idx >> 2)) & 1) break;
          if (UNI(ctl[C_FAIL]) || __builtin_amdgcn_s_memrealtime() - t0 > DEADLINE) { ctl[C_FAIL] = 1; ok = false; break; }
          __builtin_amdgcn_s_sleep(2);
        }
        waitv0();
      }
      int iss = 0, known = 0;
      u32 lim = 0;
      auto prepare = [&]() -> bool {
        lk = q;
        z = t;
        raw = 0;
        bool known_unit = true;
        if (t + CUT_MAX >= NIN) {
          if (!lim) lim = UNI(ctl[C_PLAN]);
          known_unit = lim != 0;
          if (t >= lim) {
            z = NIN + t - lim;
          } else {
            const u32 en = UNI(ctl[C_TAB + t - (NIN - CUT_MAX)]);
            z = en & 0x3FFu;
            lk = (en >> 10) & 7u;
            xe = en >> 13;
          }
        }
        const u32 i = ((z - NIN) >> 3) & 31;
        const u32 dn = ctl[C_DONE + slot], wd = ctl[z < NIN ? C_IN + (z >> 7) : C_READY];
        const u32 rw = ctl[C_RAWM + i];
        bool rdy;
        if (z < NIN) {
          const u32 id = import_id(z >> 2, lk, rank);
          so = par + (lk != q ? P_IX + lk * (2 * MiB) + id * (2 * XBLK) : id < 64 ? P_EX + id * (2 * XBLK) : P_EW + (rank * 32 + (id - 64)) * WBLK) + (z & 3) * 8192;
          rdy = lk != q || ((UNI(wd) >> ((z >> 2) & 31)) & 1) || (z < 256 && ((seen >> (z >> 2)) & 1));
        } else {
          const u32 w8 = (z - NIN) & 7;
          u32 s, c;
          tile_sc(i, s, c);
          raw = (UNI(rw) >> w8) & 1;
          so = raw ? OOB : par + P_PK + (((rank * 4 + s) * 8 + c) * 8 + w8) * SUB;
          rdy = (UNI(wd) >> i) & 1;
        }
        slot_free = UNI(dn) == 4 * (n / NSLOT);
        return known_unit && (z >= NIN + 256 || (rdy && slot_free));
      };
      prepare();
      STAMP(2);
#define LEFT(N, k) do { asm volatile("s_waitcnt vmcnt(%0)" :: "i"(N) : "memory"); ctl[C_LAND] = max(known, iss + (k) - (N)); } while (0)
      for (;;) {
        u32 fail = 0;
        if (!ok && !prepare()) {
          LEFT(4, 0);
          if (!prepare()) {
            waitv0();
            ctl[C_LAND] = known = iss;
            for (;;) {
              const u32 f = UNI(ctl[C_FAIL]);
              if (prepare()) break;
              if (f && slot_free && (lim || t + CUT_MAX < NIN)) { fail = 1; so = OOB; break; }
              __builtin_amdgcn_s_sleep(1);
            }
          }
        }
        if (z >= NIN + 256) break;
        u32 src = q;
        if (lk != q && !fail && !((UNI(ctl[C_HAVE]) >> xe) & 1)) {
          const u32 id = import_id(z >> 2, lk, rank);
          const u32 have = UNI(ld_sys(P.ws[q] + F_I + (lk * 96 + id) * 128));
          waitv0();
          ctl[C_LAND] = known = iss;
          if (have < call * 4) {
            src = lk;
            so = par + P_EX + id * (2 * XBLK) + (z & 3) * 8192;
            if (!wave_wait(me, F_S + (lk * 96 + id) * 128, lane == 0, call, call)) { ctl[C_FAIL] = 1; fail = 1; so = OOB; }
          }
          asm volatile("" ::: "memory");
        }
        if (z + 20 >= NIN && z < NIN + 16) ctl[C_NEAR] = 1;
        ctl[C_BASE + slot] = iss;
        ctl[C_ASG + slot] = ((n + 1) << 16) | (lk << 13) | (fail << 11) | (raw << 10) | z;
        LDS_AS char* ld = (LDS_AS char*)smem + slot * SLOTB;
        const u32 sc = so;
        const bool out = z >= NIN;
        const rsrc_t rs = src == q ? rq : mk_rsrc(P.ws[src & 7], P.ws_bytes);
        u32 tk;
        auto advance = [&]() {
          t = take(tk) + WPL;
          slot = slot + 1 == NSLOT ? 0 : slot + 1;
          ++n;
          ok = prepare();
        };
#define PULL(k, vofs, grp, imm) \
        asm volatile("s_waitcnt vmcnt(%0)" :: "i"(LWIN - 1) : "memory"); \
        __builtin_amdgcn_raw_ptr_buffer_load_lds(rs, ld + (grp) * 4096, 16, vofs, sc + (grp) * 4096, imm, AUX_SYS); \
        asm volatile("" ::: "memory"); \
        if (!(((k) + 2 + LWIN) & 1)) ctl[C_LAND] = max(known, iss + (k) + 1 - (LWIN - 1)); \
        if ((k) + 1 == LWIN) advance();
#define PULLX(vofs, grp, imm) __builtin_amdgcn_raw_ptr_buffer_load_lds(rs, ld + (grp) * 4096, 16, vofs, sc + (grp) * 4096, imm, AUX_SYS);
        if (!out) {
          constexpr int LWIN = LW1;
          asm volatile("s_waitcnt vmcnt(%0)" :: "i"(LWIN - 1) : "memory");
          tk = draw(me + F_T + q * 128);
          ctl[C_LAND] = max(known, iss - (LWIN - 1));
          PULL(0, v16, 0, 0) PULL(1, v16, 0, 1024) PULL(2, v16, 0, 2048) PULL(3, v16, 0, 3072)
          PULL(4, v16, 1, 0) PULL(5, v16, 1, 1024) PULL(6, v16, 1, 2048) PULL(7, v16, 1, 3072)
          iss += 9;
        } else {
          constexpr int LWIN = LW2;
          asm volatile("s_waitcnt vmcnt(%0)" :: "i"(LWIN - 1) : "memory");
          tk = draw(me + F_T + q * 128);
          ctl[C_LAND] = max(known, iss - (LWIN - 1));
          PULL(0, v16, 2, 0) PULL(1, v16, 2, 1024) PULL(2, vB, 2, 2048)
          PULL(3, v16, 0, 0) PULL(4, v16, 0, 1024) PULL(5, v16, 0, 2048) PULL(6, v16, 0, 3072)
          PULL(7, v16, 1, 0)
          if (z < NIN + 256) {
            PULL(8, v16, 1, 1024) PULL(9, v16, 1, 2048) PULL(10, v16, 1, 3072)
          } else {
            PULLX(v16, 1, 1024) PULLX(v16, 1, 2048) PULLX(v16, 1, 3072)
          }
          iss += 12;
        }
      }
      ctl[C_END] = n;
      LEFT(8, 0); LEFT(6, 0); LEFT(4, 0); LEFT(2, 0); LEFT(0, 0);
#undef PULL
#undef PULLX
#undef LEFT
      STAMP(6);
      STAMPV(7, twait & ~(1ull << 62));
      STAMPV(8, n);
      STAMPV(9, tz[0]); STAMPV(10, tz[1]); STAMPV(11, tz[2]); STAMPV(12, tz[3]); STAMPV(13, tz[4]); STAMPV(14, nst); STAMPV(15, wz);
    } else {
      if (wave == 1) {
        const u32 fo1 = F_S + (q * 96 + import_id(64 + (lane < NPULL - 64 ? lane : 0), q, rank)) * 128;
        const u64 t0 = __builtin_amdgcn_s_memrealtime();
        bool inputs = false, planned = false;
        u32 en = 0, todo = 0, have = 0, looks = 0;
        auto table = [&](u32 cut, u32 ext) {
          const u32 keep = ~cut & ((1u << CUT_MAX) - 1u), nk = __builtin_popcount(keep);
          if (lane < CUT_MAX && ((keep >> lane) & 1)) ctl[C_TAB + __builtin_popcount(keep & ((1u << lane) - 1u))] = (q << 10) | (NIN - CUT_MAX + lane);
          if (lane < ext) ctl[C_TAB + nk + lane] = (lane << 13) | ((en & 7u) << 10) | (NIN - CUT_MAX + ((en >> 3) & 31u));
          asm volatile("s_waitcnt lgkmcnt(0)" ::: "memory");
          ctl[C_PLAN] = NIN - CUT_MAX + nk + ext;
        };
        for (;;) {
          const u32 tw = ld_agt(me + F_TMO);
          bool fin = false;
          if (!inputs) {
            const u32 f0 = ld_sys(me + fo0), f1 = ld_sys(me + fo1);
            const u64 m0 = __ballot(f0 >= call), m1 = __ballot(f1 >= call);
            if (lane < 3) ctl[C_IN + lane] = lane == 0 ? (u32)m0 : lane == 1 ? (u32)(m0 >> 32) : (u32)m1;
            inputs = (m0 & m1) == ~0ull;
          } else {
            const u32 near = UNI(ctl[C_NEAR]);
            if (!planned) {
              const u32 pw = ld_agt(me + F_PLAN + q * 128 + (lane & 15) * 4);
              if (__all((pw >> 16) == (call & 0xFFFFu))) {
                en = pw & 0xFFFFu;
                todo = (u32)__ballot(lane < 14 && (en >> 8));
                table(__builtin_amdgcn_readlane(en, 14) | (__builtin_amdgcn_readlane(en, 15) << 14), __builtin_popcount(todo));
                planned = true;
              }
            }
            if (todo && near) {
              bool has = false;
              if (lane < 14 && ((todo >> lane) & 1))
                has = ld_sys(P.ws[q] + F_I + ((en & 7u) * 96 + import_id((NIN - CUT_MAX + ((en >> 3) & 31u)) >> 2, en & 7u, rank)) * 128) >= call * 4;
              const u32 got = (u32)__ballot(has);
              have |= got;
              ctl[C_HAVE] = have;
              todo = ++looks < 3 ? todo & ~got : 0u;
            }
            const u32 l = lane & 31;
            const u64 of = __hip_atomic_load((const u64*)(me + F_O + (q * 32 + l) * 128), __ATOMIC_RELAXED, __HIP_MEMORY_SCOPE_SYSTEM);
            const u32 pf = ld_agt(me + F_P + (q * 32 + l) * 128);
            const u32 rd = (u32)__ballot((u32)(of >> 32) >= call && pf >= call);
            if (lane < 32) ctl[C_RAWM + lane] = (u32)of;
            asm volatile("s_waitcnt lgkmcnt(0)" ::: "memory");
            ctl[C_READY] = rd;
            fin = rd == 0xFFFFFFFFu && planned;
          }
          if (UNI(tw) == call || UNI(ctl[C_FAIL]) || __builtin_amdgcn_s_memrealtime() - t0 > DEADLINE) {
            st_agt(me + F_TMO, call);
            if (!planned) table(0u, 0u);
            ctl[C_FAIL] = 1;
            break;
          }
          if (fin) break;
          if (inputs) __builtin_amdgcn_s_sleep(OSLEEP); else __builtin_amdgcn_s_sleep(WSLEEP);
        }
      }
      const float scale = P.sx[0] * P.sw[0];
      const u32 vo = (u32)(lane >> 4) * 4 * (NCOL * 4) + (lane & 15) * 16;
      auto peek = [&](u32 item) -> u32 {
        const u32 n = item >> 2, a = UNI(ctl[C_ASG + n % NSLOT]);
        return (a >> 16) == n + 1 ? a : 0u;
      };
      auto await = [&](u32 item) -> u32 {
        for (;;) {
          const u32 e = UNI(ctl[C_END]);
          const u32 a = peek(item);
          if (a || (item >> 2) >= e) return a;
          __builtin_amdgcn_s_sleep(1);
        }
      };
      auto landed = [&](u32 slot, u32 k) {
        const u32 need = UNI(ctl[C_BASE + slot]) + k;
        while ((int)(UNI(ctl[C_LAND]) - need) < 0) __builtin_amdgcn_s_sleep(1);
        asm volatile("" ::: "memory");
      };
      auto out_off = [&](u32 a, u32 item) -> u32 {
        const u32 z = (a & 0x3FFu) - NIN, w8 = z & 7;
        u32 s, c;
        tile_sc(z >> 3, s, c);
        return ((s * 128 + (w8 >> 2) * 64 + (item & 3) * 16) * NCOL + q * 2048 + c * 256 + (w8 & 3) * 64) * 4;
      };
      auto fetch = [&](u32 pb, u32 a, u32 item) {
        const u32 oo = ((a & 0x3FFu) >= NIN && !(a & 0x800u)) ? out_off(a, item) : OOB;
        LDS_AS char* ld = (LDS_AS char*)smem + pb;
        __builtin_amdgcn_raw_ptr_buffer_load_lds(rout, ld, 16, vo, oo, 0, AUX_SYS);
        __builtin_amdgcn_raw_ptr_buffer_load_lds(rout, ld, 16, vo + (NCOL * 4 - 1024), oo, 1024, AUX_SYS);
        __builtin_amdgcn_raw_ptr_buffer_load_lds(rout, ld, 16, vo + 2 * (NCOL * 4 - 1024), oo, 2048, AUX_SYS);
        __builtin_amdgcn_raw_ptr_buffer_load_lds(rout, ld, 16, vo + 3 * (NCOL * 4 - 1024), oo, 3072, AUX_SYS);
      };
#define TICK(x)
      u32 pcur = LDS_PT + (wave - 1) * 8192;
      bool early = false;
      u32 item = lds_add(ctl + C_ITEM, 1u);
      u32 a = await(item);
      fetch(pcur, a, item);
      while (a) {
        const u32 n = item >> 2, m = item & 3, slot = n % NSLOT, zt = a & 0x3FFu;
        if (zt < NIN || (a & 0xC00u)) {
          if (zt < NIN) {
            if (!(a & 0x800u)) {
              const u32 lk = (a >> 13) & 7, id = import_id(zt >> 2, lk, rank);
              const u32 dso = par + (id < 64 ? P_IX + lk * (2 * MiB) + id * (2 * XBLK) : P_IW + (lk * 32 + (id - 64)) * WBLK) + (zt & 3) * 8192 + m * 2048;
              landed(slot, 3 + 2 * m);
              const char* ls = smem + slot * SLOTB + m * 2048 + lane * 16;
              const u32x4 v0 = *(const u32x4*)ls, v1 = *(const u32x4*)(ls + 1024);
              {
                st128(v0, rme, lane * 16, dso, AUX_SYS);
                st128(v1, rme, lane * 16 + 1024, dso, AUX_SYS);
              }
              waitv0();
              if ((lds_add(ctl + C_DONE + slot, 1u) & 3u) == 3u) bump(me + F_I + (lk * 96 + id) * 128);
            } else {
              lds_inc(ctl + C_DONE + slot);
            }
          } else {
            lds_inc(ctl + C_DONE + slot);
            waitv0();
            const u32 z = zt - NIN, oo = out_off(a, item);
            u32 s, c;
            tile_sc(z >> 3, s, c);
            const u32 so = par + P_RAW + (((rank * 4 + s) * 8 + c) * 8 + (z & 7)) * 16384 + m * 4096;
#pragma unroll
            for (int r = 0; r < 4; ++r) {
              u32x4 y = {0x7FC00000u, 0x7FC00000u, 0x7FC00000u, 0x7FC00000u};
              if (!(a & 0x800u)) {
                const u32x4 v = __builtin_amdgcn_raw_buffer_load_b128(rq, lane * 16 + r * 1024, so, AUX_SYS);
                const u32x4 pt = *(const u32x4*)(smem + pcur + r * 1024 + lane * 16);
#pragma unroll
                for (int t = 0; t < 4; ++t) y[t] = __builtin_bit_cast(u32, (float)((int)v[t] + (int)pt[t]) * scale);
              }
              st128(y, rout, vo + r * (NCOL * 4), oo, AUX_AGT);
            }
          }
          item = lds_add(ctl + C_ITEM, 1u);
          a = await(item);
          pcur ^= 4096;
          fetch(pcur, a, item);
          early = false;
          continue;
        }
        const u32 oo = out_off(a, item);
        TICK(tA);
        const u32 item2 = lds_add(ctl + C_ITEM, 1u);
        u32 a2 = 0;
        const u32 base = UNI(ctl[C_BASE + slot]);
        auto until = [&](u32 need) {
          while ((int)(UNI(ctl[C_LAND]) - need) < 0) {
            if (!a2) {
              a2 = peek(item2);
              if (a2) fetch(pcur ^ 4096, a2, item2);
            }
            __builtin_amdgcn_s_sleep(1);
          }
          asm volatile("" ::: "memory");
        };
        until(base + 4);
        const char* ls = smem + slot * SLOTB;
        const u32* cw = (const u32*)(ls + 10240);
        const u32 lb = 7u * (lane >> 1);
        const u32 lp = __builtin_amdgcn_alignbit(cw[(lb >> 5) + 1], cw[lb >> 5], lb & 31u) & 127u;
        const u32x4 n0 = *(const u32x4*)(ls + 8192 + lane * 16), n1 = *(const u32x4*)(ls + 9216 + lane * 16);
        u32 total;
        const u32 wp = code_window(lp, lane, total);
        const u32 c0 = cw[wp >> 5], c1 = cw[(wp >> 5) + 1], c2 = cw[(wp >> 5) + 2];
        u64 B = ((u64)__builtin_amdgcn_alignbit(c2, c1, wp & 31u) << 32) | __builtin_amdgcn_alignbit(c1, c0, wp & 31u);
        if (lane & 1) B = rev64(B);
        B = (B << 1) | 1u;
        auto escs = [](u32 x) -> u32 { return __builtin_popcount(x & (x >> 1) & 0x44444444u); };
        u32 cnt = 0;
        if (m > 0) cnt += escs(n0[0]) + escs(n0[1]);
        if (m > 1) cnt += escs(n0[2]) + escs(n0[3]);
        if (m > 2) cnt += escs(n1[0]) + escs(n1[1]);
        u32 pos = 0, wd = (u32)B, pc = __builtin_popcount(wd);
        if (cnt >= pc) { cnt -= pc; wd = (u32)(B >> 32); pos = 32; }
#pragma unroll
        for (int wdt = 16; wdt >= 1; wdt >>= 1) {
          pc = __builtin_popcount(wd & ((1u << wdt) - 1u));
          if (cnt >= pc) { cnt -= pc; wd >>= wdt; pos += wdt; }
        }
        B >>= (pos & 63u);
        const u32 nd0 = m == 0 ? n0[0] : m == 1 ? n0[2] : m == 2 ? n1[0] : n1[2];
        const u32 nd1 = m == 0 ? n0[1] : m == 1 ? n0[3] : m == 2 ? n1[1] : n1[3];
        if (early && a2) asm volatile("s_waitcnt vmcnt(8)" ::: "memory");
        else if (early || a2) asm volatile("s_waitcnt vmcnt(4)" ::: "memory");
        else asm volatile("s_waitcnt vmcnt(0)" ::: "memory");
        TICK(tC);
        u32x4 pt[4];
#pragma unroll
        for (int r = 0; r < 4; ++r) pt[r] = *(const u32x4*)(smem + pcur + r * 1024 + lane * 16);
#pragma unroll
        for (int r = 0; r < 4; ++r) {
          const u32 nd = r < 2 ? nd0 : nd1, ne = nd & (nd >> 1);
#pragma unroll
          for (int t = 0; t < 4; ++t) {
            const int sh = 8 * t + 4 * (r & 1);
            const u32 esc = (u32)__builtin_amdgcn_sbfe((int)ne, sh + 2, 1);
            const u32 len = ffbl((u32)B ^ 1u) & esc;
            B >>= (len & 63u);
            const u32 h = __builtin_amdgcn_ubfe(nd, sh + 2, 2) + len + esc;
            pt[r][t] += ((u32)__builtin_amdgcn_sbfe((int)nd, sh, 2) ^ (h << 1)) << 16;
          }
        }
        asm volatile("" : "+v"(pt[0]), "+v"(pt[1]), "+v"(pt[2]), "+v"(pt[3]));
        until(base + 6 + 2 * m);
        TICK(tB);
        const u32x4 l0 = *(const u32x4*)(ls + m * 2048 + lane * 16), l1 = *(const u32x4*)(ls + m * 2048 + 1024 + lane * 16);
        asm volatile("s_waitcnt lgkmcnt(0)" :: "v"(l0), "v"(l1) : "memory");
        lds_inc(ctl + C_DONE + slot);
#pragma unroll
        for (int r = 0; r < 4; ++r) {
          u32x4 y;
#pragma unroll
          for (int t = 0; t < 4; ++t) {
            const u32 lo = r < 2 ? l0[(r & 1) * 2 + (t >> 1)] : l1[(r & 1) * 2 + (t >> 1)];
            y[t] = __builtin_bit_cast(u32, (float)(int)(pt[r][t] + ((t & 1) ? lo >> 16 : lo & 0xFFFFu)) * scale);
          }
          st128(y, rout, vo + r * (NCOL * 4), oo, OAUX);
        }
        TICK(tD);
        item = item2;
        a = a2;
        early = a2 != 0;
        pcur ^= 4096;
        if (!a2) {
          a = await(item);
          fetch(pcur, a, item);
        }
      }
      STAMPV(9, tA); STAMPV(10, tB); STAMPV(11, tC); STAMPV(12, tD);
    }
    STAMP(4);
    return;
  }

  const u32 vo = ((u32)(wave >> 2) * 64 + (lane >> 4) * 4) * (NCOL * 4) + ((wave & 3) * 64 + (lane & 15) * 4) * 4;
  i32x4 acc[4][4];
  for (int job = 0; job < 4; ++job) {
    const bool partial = job < 2;
    u32 t;
    if (job == 0) t = gi;
    else if (job == 1) { if (gi < NGEM - 24) continue; t = gi + 24; }
    else if (job == 2) { if (gi >= 192) break; t = gi; }
    else { if (gi >= 64) break; t = 192 + gi; }
    u32 g, c, cg_self, foff, s, dst, oi, tgt = call;
    int dskip, nk;
    bool act, own = true;
    if (partial) {
      const u32 r = t >> 3;
      oi = (r / 7) * 8 + (t & 7);
      tile_sc(oi, s, c);
      dst = (rank + 1 + r % 7) & 7u;
      g = rank * 4 + s;
      cg_self = dst * 8 + c;
      dskip = 8; nk = 4;
      act = lane < 6;
      foff = lane < 4 ? F_LW + (cg_self * 4 + lane) * 128 : F_S + (rank * 96 + g * 2 + (lane & 1)) * 128;
    } else {
      u32 pd;
      if (t < 128) { oi = t >> 3; pd = t & 7; }
      else { oi = (t >> 6) * 8 + (t & 7); pd = (t >> 3) & 7; }
      tile_sc(oi, s, c);
      dst = (rank + 1 + pd) & 7u;
      g = dst * 4 + s;
      cg_self = rank * 8 + c;
      dskip = dst == rank ? 8 : (int)((dst - rank) & 7u);
      nk = dst == rank ? 32 : 28;
      const u32 q = lane / 6, e = lane % 6;
      act = lane < 48 && (q != dst || dst == rank);
      const u32 id = e < 4 ? 64 + c * 4 + e : g * 2 + (e & 1);
      foff = (q == rank ? F_S : F_I) + ((q & 7) * 96 + id) * 128;
      own = q == rank;
      tgt = own ? call : call * 4;
    }
#pragma unroll
    for (int m = 0; m < 4; ++m)
#pragma unroll
      for (int tt = 0; tt < 4; ++tt) acc[m][tt] = (i32x4){0, 0, 0, 0};
    bool good = true;
    for (int part = 0; part < (partial ? 1 : 2) && good; ++part) {
      good = wg_wait(smem, me, foff, act && own == (part == 0), tgt, call, wave);
      if (part || partial) STAMP(2 + job * 3);
      if (good) gemm_tile(smem, rme, (int)rank, par, g, cg_self, c, dskip, part ? 4 : 0, part ? nk : 4, wave, lane, acc);
    }
    STAMP(3 + job * 3);
    if (partial) {
      if (good) {
        const u32 oo = (s * 128 * NCOL + dst * 2048 + c * 256) * 4;
#pragma unroll
        for (int m = 0; m < 4; ++m)
#pragma unroll
          for (int r = 0; r < 4; ++r) {
            const u32x4 y = {(u32)acc[m][0][r], (u32)acc[m][1][r], (u32)acc[m][2][r], (u32)acc[m][3][r]};
            st128(y, rout, vo + (m * 16 + r) * (NCOL * 4), oo, AUX_AGT);
          }
        waitv0();
        bar();
        if (tid == 0) st_agt(me + F_P + (dst * 32 + oi) * 128, call);
      }
    } else if (dst == rank) {
      const float scale = P.sx[0] * P.sw[0];
      const u32 oo = (s * 128 * NCOL + rank * 2048 + c * 256) * 4;
#pragma unroll
      for (int m = 0; m < 4; ++m)
#pragma unroll
        for (int r = 0; r < 4; ++r) {
          u32x4 y;
#pragma unroll
          for (int tt = 0; tt < 4; ++tt) y[tt] = good ? __builtin_bit_cast(u32, (float)acc[m][tt][r] * scale) : 0x7FC00000u;
          st128(y, rout, vo + (m * 16 + r) * (NCOL * 4), oo, AUX_AGT);
        }
    } else if (good) {
      if (tid == 0) ctl[C_RAW] = 0;
      bar();
      const u32 sub = (g * 8 + c) * 8 + wave;
      const u32 so = par + P_PK + sub * SUB;
#pragma unroll
      for (int m = 0; m < 4; ++m)
#pragma unroll
        for (int hf = 0; hf < 2; ++hf) {
          const u32x4 lo = {perm(acc[m][1][2 * hf], acc[m][0][2 * hf], 0x05040100u), perm(acc[m][3][2 * hf], acc[m][2][2 * hf], 0x05040100u),
                            perm(acc[m][1][2 * hf + 1], acc[m][0][2 * hf + 1], 0x05040100u), perm(acc[m][3][2 * hf + 1], acc[m][2][2 * hf + 1], 0x05040100u)};
          st128(lo, rme, lane * 16 + m * 2048 + hf * 1024, so, AUX_SYS);
        }
      LDS_AS u32* cw = (LDS_AS u32*)((LDS_AS char*)smem + wave * 512);
      cw[lane] = 0;
      if (lane < 8) cw[64 + lane] = 0;
      u64 B = 1;
      u32 any = 0;
      u16x2 sum2 = {0, 0};
      u32x4 nb[2];
#pragma unroll
      for (int m = 3; m >= 0; --m) {
        u32 hb[4];
#pragma unroll
        for (int r = 3; r >= 0; --r) {
          u32 nv[2];
#pragma unroll
          for (int pr = 1; pr >= 0; --pr) {
            const u32 qv = perm(acc[m][2 * pr + 1][r], acc[m][2 * pr][r], 0x07060302u);
            const s16x2 q2 = __builtin_bit_cast(s16x2, qv);
            const u32 sg = __builtin_bit_cast(u32, q2 >> (s16x2){15, 15});
            const u16x2 h = __builtin_bit_cast(u16x2, __builtin_bit_cast(u32, q2 >> (s16x2){1, 1}) ^ sg);
            const u16x2 len = __builtin_elementwise_sub_sat(h, (u16x2){2, 2});
            const u32 lw = __builtin_bit_cast(u32, len);
            B = (B << ((lw >> 16) & 63u)) | 1u;
            B = (B << (lw & 63u)) | 1u;
            sum2 += len;
            any |= lw;
            const u32 t2 = __builtin_bit_cast(u32, __builtin_elementwise_min(h, (u16x2){3, 3})) << 2;
            nv[pr] = (qv & 0x00010001u) | (sg & 0x00020002u) | t2;
          }
          hb[r] = perm(nv[1], nv[0], 0x06040200u);
        }
        nb[m >> 1][(m & 1) * 2] = hb[0] | (hb[1] << 4);
        nb[m >> 1][(m & 1) * 2 + 1] = hb[2] | (hb[3] << 4);
      }
      const u32 sum = (u32)sum2[0] + sum2[1];
      const u32 lp = sum + __builtin_amdgcn_update_dpp(0u, sum, 0xB1, 0xF, 0xF, false);
      u32 total;
      const u32 wp = code_window(lp, lane, total);
      if (__any(sum > 63u || (any & 0xFFE0FFE0u)) || total > 2048u) {
        if (lane == 0) __hip_atomic_fetch_or((u32*)(smem + LDS_CTL) + C_RAW, 1u << wave, __ATOMIC_RELAXED, __HIP_MEMORY_SCOPE_WORKGROUP);
#pragma unroll
        for (int m = 0; m < 4; ++m)
#pragma unroll
          for (int r = 0; r < 4; ++r) {
            const u32x4 y = {(u32)acc[m][0][r], (u32)acc[m][1][r], (u32)acc[m][2][r], (u32)acc[m][3][r]};
            st128(y, rme, lane * 16 + (m * 4 + r) * 1024, par + P_RAW + sub * 16384, AUX_SYS);
          }
      } else {
        st128(nb[0], rme, lane * 16 + 8192, so, AUX_SYS);
        st128(nb[1], rme, lane * 16 + 9216, so, AUX_SYS);
        const u64 V = (lane & 1) ? rev64(B >> 1) : B >> 1;
        const u32 sf = wp & 31u, lb = 7u * (lane >> 1);
        const u64 L = (u64)((lane & 1) ? 0u : lp) << (lb & 31u);
        __hip_atomic_fetch_or(cw + (lb >> 5), (u32)L, __ATOMIC_RELAXED, __HIP_MEMORY_SCOPE_WAVEFRONT);
        __hip_atomic_fetch_or(cw + (lb >> 5) + 1, (u32)(L >> 32), __ATOMIC_RELAXED, __HIP_MEMORY_SCOPE_WAVEFRONT);
        __hip_atomic_fetch_or(cw + (wp >> 5), (u32)V << sf, __ATOMIC_RELAXED, __HIP_MEMORY_SCOPE_WAVEFRONT);
        __hip_atomic_fetch_or(cw + (wp >> 5) + 1, (u32)((V << sf) >> 32), __ATOMIC_RELAXED, __HIP_MEMORY_SCOPE_WAVEFRONT);
        __hip_atomic_fetch_or(cw + (wp >> 5) + 2, (u32)(((V >> 32) << sf) >> 32), __ATOMIC_RELAXED, __HIP_MEMORY_SCOPE_WAVEFRONT);
        asm volatile("s_waitcnt lgkmcnt(0)" ::: "memory");
        __builtin_amdgcn_raw_buffer_store_b32(*(volatile LDS_AS u32*)(cw + lane), rme, lane * 4 + 10240, so, AUX_SYS);
      }
      waitv0();
      bar();
      if (tid == 0) {
        const u64 f = ((u64)call << 32) | ctl[C_RAW];
        __hip_atomic_store((u64*)(P.ws[dst] + F_O + (rank * 32 + oi) * 128), f, __ATOMIC_RELAXED, __HIP_MEMORY_SCOPE_SYSTEM);
      }
    }
    STAMP(4 + job * 3);
  }
  if (gi == NGEM - 1 && wave == 0) {
    u32 cnt = 0, cut = 0, ext = 0;
    u64 e0 = 0, e1 = 0;
    const u64 t0 = __builtin_amdgcn_s_memrealtime();
    for (;;) {
      if (lane < 8 && lane != rank) cnt = peek_word(me + F_T + lane * 128);
      if (__any(cnt + PLAN_AT >= NIN) || __builtin_amdgcn_s_memrealtime() - t0 > DEADLINE) break;
      __builtin_amdgcn_s_sleep(48);
    }
    auto top8 = [&](u32 x) -> u32 {
      x = max(x, (u32)__builtin_amdgcn_update_dpp(0u, x, 0x111, 0xF, 0xF, false));
      x = max(x, (u32)__builtin_amdgcn_update_dpp(0u, x, 0x112, 0xF, 0xF, false));
      x = max(x, (u32)__builtin_amdgcn_update_dpp(0u, x, 0x114, 0xF, 0xF, false));
      return __builtin_amdgcn_readlane(x, 7);
    };
    cnt = min(cnt, 4000u);
    u32 eff = cnt;
    for (int it = 0; it < 40; ++it) {
      const u32 gv = top8((lane != rank && ext == 0) ? ((4095u - eff) << 3) | lane : 0u);
      const u32 v = gv & 7u, lo = 4095u - (gv >> 3);
      const u32 gap = (cnt - __builtin_amdgcn_readlane(cnt, v)) >> 2, held = gap < 11u ? min(7u, 11u - gap) : 0u;
      u32 sh = 2 * (((rank - v - 1) & 7u) + 7 - ((lane - v - 1) & 7u));
      sh -= sh >= 14 ? 14 : 0;
      const u32 win = ((1u << held) - 1u) * 0x4001u;
      u32 cand = (win >> (sh + 7 >= 14 ? sh - 7 : sh + 7)) & 127u;
      cand = (cand | (cand << 12)) & 0x000F000Fu;
      cand = (cand | (cand << 6)) & 0x03030303u;
      cand = (cand | (cand << 3)) & 0x11111111u;
      cand = (cand * 15u) & ~__builtin_amdgcn_readlane(cut, v);
      const u32 hv = top8((lane != rank && lane != v && cut == 0 && ext < EXT_MAX && cand) ? (eff << 3) | lane : 0u);
      const u32 h = hv & 7u;
      if ((hv >> 3) < lo + LAG_MIN) break;
      const u32 u = __builtin_ctz(__builtin_amdgcn_readlane(cand, h));
      if (lane == h) {
        const u64 x = (u64)(v | (u << 3)) << (8 * (ext & 7));
        if (ext < 8) e0 |= x; else e1 |= x;
        ++ext;
        --eff;
      }
      if (lane == v) { cut |= 1u << u; ++eff; }
    }
#pragma unroll
    for (int half = 0; half < 2; ++half) {
      const int lnk = half * 4 + (lane >> 4), k = lane & 15;
      const u64 s0 = __shfl(e0, lnk), s1 = __shfl(e1, lnk), es = k < 8 ? s0 : s1;
      const u32 cs = __shfl(cut, lnk), xs = __shfl(ext, lnk);
      const u32 w = k < 14 ? ((u32)k < xs ? 0x100u | ((u32)(es >> (8 * (k & 7))) & 255u) : 0u) : (cs >> (k == 15 ? 14 : 0)) & 0x3FFFu;
      st_agt(me + F_PLAN + lnk * 128 + k * 4, (call << 16) | w);
    }
    STAMP(15);
  }
}

}

extern "C" void kernel_launch(int rank, int nranks, ncclComm_t comm,
                              hipStream_t stream,
                              void* const* d_in, const int* in_sizes, int n_in,
                              void* d_out, int out_size,
                              const int* in_dtypes, int out_dtype,
                              void* const* d_out_all,
                              void* const* d_ws_all, size_t ws_bytes) {
  if (nranks != NR || (unsigned)rank >= (unsigned)NR || n_in < 4 || ws_bytes < (size_t)WS_NEED + MiB ||
      (size_t)out_size * 4 != OUT_BYTES || in_sizes[0] != 4096 * KSL || in_sizes[1] != KSL * NCOL) {
    fprintf(stderr, "kernel_launch: unexpected shapes\n");
    return;
  }
  Params p;
  p.x = (const signed char*)d_in[0];
  p.w = (const signed char*)d_in[1];
  p.sx = (const float*)d_in[2];
  p.sw = (const float*)d_in[3];
  p.out = (float*)d_out;
  for (int i = 0; i < NR; ++i) p.ws[i] = (char*)d_ws_all[i];
  p.me = p.ws[rank];
  p.ws_bytes = WS_NEED + MiB;
  p.rank = rank;
  gemm_rs_kernel<<<NWG, NTHR, 0, stream>>>(p);
  HIP_CHECK(hipGetLastError());
}
```

```cpp
#include <hip/hip_runtime.h>
#include <rccl/rccl.h>
#include <cstdio>

#define HIP_CHECK(expr) do { hipError_t _e = (expr); if (_e != hipSuccess) { \
    fprintf(stderr, "HIP error %s at %s:%d\n", hipGetErrorString(_e), __FILE__, __LINE__); \
    return; } } while (0)

typedef unsigned u32;
typedef unsigned long long u64;
typedef u32 u32x4 __attribute__((ext_vector_type(4)));
typedef int i32x4 __attribute__((ext_vector_type(4)));
typedef short s16x2 __attribute__((ext_vector_type(2)));
typedef unsigned short u16x2 __attribute__((ext_vector_type(2)));
typedef __amdgpu_buffer_rsrc_t rsrc_t;
#define LDS_AS __attribute__((address_space(3)))
#define UNI(x) __builtin_amdgcn_readfirstlane(x)

namespace {

constexpr int NR = 8;
constexpr int KSL = 512;
constexpr int NCOL = 16384;
constexpr int NWG = 256, NTHR = 512;
constexpr int WPL = 8;
constexpr int NSTR = 7 * WPL;
constexpr int NGEM = NWG - NSTR;
constexpr int EXPORT_DELAY = 80;
constexpr u32 NPUSH = 8;
constexpr int LW1 = 8, LW2 = 7;
constexpr u32 PLAN_AT = 180;
constexpr u32 LAG_MIN = 6;
constexpr u32 EXT_MAX = 24;
constexpr u32 SAFE = 26;
constexpr u32 DLY_MAX = 56;
constexpr u32 RDMASK = 0x0B6Du, DMAMASK = 0x2082u;

constexpr u32 KiB = 1024u, MiB = 1024u * 1024u;
constexpr u32 F_CNT = 0;
constexpr u32 F_TMO = 32 * KiB;
constexpr u32 F_S = 64 * KiB;
constexpr u32 F_I = 160 * KiB;
constexpr u32 F_LW = 256 * KiB;
constexpr u32 F_O = 288 * KiB;
constexpr u32 F_P = 320 * KiB;
constexpr u32 F_T = 352 * KiB;
constexpr u32 F_PLAN = 354 * KiB;
constexpr u32 XBLK = 16 * KiB, WBLK = 32 * KiB;
constexpr u32 D_PAR = 1 * MiB;
constexpr u32 P_EX = 0;
constexpr u32 P_EW = 2 * MiB;
constexpr u32 SUB = 10496;
constexpr u32 P_PK = 10 * MiB;
constexpr u32 P_RAW = 32 * MiB;
constexpr u32 P_IW = 64 * MiB;
constexpr u32 P_IX = 72 * MiB;
constexpr u32 PAR_BYTES = 88 * MiB;
constexpr u32 WS_NEED = D_PAR + 2 * PAR_BYTES;
constexpr u32 OUT_BYTES = 512u * NCOL * 4u;

constexpr u32 STAGE = XBLK + WBLK;
constexpr u32 LDS_CTL = 3 * STAGE;
constexpr u32 LDS_BYTES = LDS_CTL + 1024;
constexpr u32 NSLOT = 8, SLOTB = 11264;
constexpr u32 LDS_PT = NSLOT * SLOTB;
static_assert(LDS_PT + 7 * 8192 <= LDS_CTL, "LDS");
constexpr int C_RAW = 8, C_IN = 12, C_ITEM = 17, C_END = 18, C_READY = 19, C_FAIL = 20,
              C_LAND = 21, C_NEAR = 22, C_HAVE = 23, C_RAWM = 24, C_PLAN = 56, C_TAB = 128, C_ASG = 64, C_BASE = 80, C_DONE = 96;

constexpr u64 DEADLINE = 150000000ull;
constexpr int AUX_SYS = 17, AUX_AGT = 16;
constexpr u32 OOB = 0x70000000u;

struct Params {
  const signed char* x;
  const signed char* w;
  const float* sx;
  const float* sw;
  float* out;
  char* me;
  char* ws[NR];
  u32 ws_bytes;
  int rank;
};

__device__ __forceinline__ rsrc_t mk_rsrc(const void* p, u32 bytes) {
  return __builtin_amdgcn_make_buffer_rsrc((void*)p, (short)0, (int)bytes, 0x00020000);
}
#define st128(v, r, vo, so, aux) __builtin_amdgcn_raw_buffer_store_b128((v), (r), (vo) + (so), 0, (aux))
__device__ __forceinline__ void waitv0() { asm volatile("s_waitcnt vmcnt(0)" ::: "memory"); }
__device__ __forceinline__ void bar() {
  asm volatile("s_waitcnt lgkmcnt(0)" ::: "memory");
  __builtin_amdgcn_s_barrier();
  asm volatile("" ::: "memory");
}
__device__ __forceinline__ u32 ld_sys(const char* p) {
  return __hip_atomic_load((const u32*)p, __ATOMIC_RELAXED, __HIP_MEMORY_SCOPE_SYSTEM);
}
__device__ __forceinline__ u32 ld_agt(const char* p) {
  return __hip_atomic_load((const u32*)p, __ATOMIC_RELAXED, __HIP_MEMORY_SCOPE_AGENT);
}
__device__ __forceinline__ void st_sys(char* p, u32 v) {
  __hip_atomic_store((u32*)p, v, __ATOMIC_RELAXED, __HIP_MEMORY_SCOPE_SYSTEM);
}
__device__ __forceinline__ void st_agt(char* p, u32 v) {
  __hip_atomic_store((u32*)p, v, __ATOMIC_RELAXED, __HIP_MEMORY_SCOPE_AGENT);
}

__device__ __forceinline__ bool wave_wait(const char* me, u32 off, bool act, u32 tgt, u32 call) {
  bool ok = !act || ld_sys(me + off) >= tgt;
  if (__all(ok)) return true;
  const u64 t0 = __builtin_amdgcn_s_memrealtime();
  for (;;) {
    __builtin_amdgcn_s_sleep(32);
    if (!ok) ok = ld_sys(me + off) >= tgt;
    if (__all(ok)) return true;
    if (UNI(ld_agt(me + F_TMO)) == call || __builtin_amdgcn_s_memrealtime() - t0 > DEADLINE) {
      st_agt((char*)me + F_TMO, call);
      return false;
    }
  }
}

__device__ __forceinline__ bool wg_wait(char* smem, const char* me, u32 off, bool act, u32 tgt, u32 call, int wave) {
  volatile LDS_AS u32* word = (volatile LDS_AS u32*)((LDS_AS char*)smem + LDS_CTL);
  if (wave == 0) {
    bool r = wave_wait(me, off, act, tgt, call);
    r = r && UNI(ld_agt(me + F_TMO)) != call;
    *word = r ? 1u : 2u;
  }
  bar();
  const u32 r = UNI(*word);
  bar();
  return r == 1u;
}

__device__ __forceinline__ u32 ffbl(u32 x) {
  u32 r;
  asm("v_ffbl_b32 %0, %1" : "=v"(r) : "v"(x));
  return r;
}
__device__ __forceinline__ u32 wave_scan(u32 x) {
  x += __builtin_amdgcn_update_dpp(0u, x, 0x111, 0xF, 0xF, false);
  x += __builtin_amdgcn_update_dpp(0u, x, 0x112, 0xF, 0xF, false);
  x += __builtin_amdgcn_update_dpp(0u, x, 0x114, 0xF, 0xF, false);
  x += __builtin_amdgcn_update_dpp(0u, x, 0x118, 0xF, 0xF, false);
  x += __builtin_amdgcn_update_dpp(0u, x, 0x142, 0xA, 0xF, false);
  x += __builtin_amdgcn_update_dpp(0u, x, 0x143, 0xC, 0xF, false);
  return x;
}
__device__ __forceinline__ u32 code_window(u32 lp, int lane, u32& total) {
  const u32 end = 224u + wave_scan((lane & 1) ? 0u : lp);
  total = __builtin_amdgcn_readlane(end, 63);
  return (lane & 1) ? end - 64u : end - lp;
}
__device__ __forceinline__ u64 rev64(u64 x) {
  return ((u64)__builtin_bitreverse32((u32)x) << 32) | __builtin_bitreverse32((u32)(x >> 32));
}
__device__ __forceinline__ u32 perm(u32 hi, u32 lo, u32 sel) { return __builtin_amdgcn_perm(hi, lo, sel); }

__device__ __forceinline__ void gemm_tile(char* smem, rsrc_t rws, int rank, u32 par, u32 g, u32 cg_self, u32 c,
                                          int dskip, int k0, int nk, int wave, int lane, i32x4 (&acc)[4][4]) {
  const u32 va = (u32)wave * 2048u + (u32)lane * 16u;
  const u32 vb = (u32)wave * 4096u + (u32)lane * 16u;
  u32 sa, sb;
  auto locate = [&](int kb) {
    const int qi = kb >> 2;
    const u32 kc = kb & 3;
    const int d = qi + (qi >= dskip ? 1 : 0);
    const u32 q = (u32)(rank + d) & 7u;
    sa = (d == 0 ? par + P_EX : par + P_IX + q * (2 * MiB)) + (g * 4 + kc) * XBLK;
    sb = (d == 0 ? par + P_EW + cg_self * (4 * WBLK) : par + P_IW + (q * 8 + c) * (4 * WBLK)) + kc * WBLK;
    if (kb >= nk) sa = sb = OOB;
  };
  auto dma = [&](int n, u32 buf) {
    LDS_AS char* la = (LDS_AS char*)smem + buf * STAGE + wave * 2048;
    LDS_AS char* lb = (LDS_AS char*)smem + buf * STAGE + XBLK + wave * 4096;
    if (n == 0) __builtin_amdgcn_raw_ptr_buffer_load_lds(rws, la, 16, va, sa, 0, AUX_SYS);
    if (n == 1) __builtin_amdgcn_raw_ptr_buffer_load_lds(rws, la, 16, va, sa, 1024, AUX_SYS);
    if (n == 2) __builtin_amdgcn_raw_ptr_buffer_load_lds(rws, lb, 16, vb, sb, 0, AUX_SYS);
    if (n == 3) __builtin_amdgcn_raw_ptr_buffer_load_lds(rws, lb, 16, vb, sb, 1024, AUX_SYS);
    if (n == 4) __builtin_amdgcn_raw_ptr_buffer_load_lds(rws, lb, 16, vb, sb, 2048, AUX_SYS);
    if (n == 5) __builtin_amdgcn_raw_ptr_buffer_load_lds(rws, lb, 16, vb, sb, 3072, AUX_SYS);
  };
  const u32 ra = (u32)(wave >> 2) * 4096u + (u32)lane * 16u;
  const u32 rb = XBLK + (u32)(wave & 3) * 4096u + (u32)lane * 16u;
  i32x4 f0[8], f1[8];
  auto rd = [&](i32x4 (&f)[8], int n, u32 buf, int ks) {
    const int j = n == 0 ? 0 : (n < 5 ? n + 3 : n - 4);
    const char* p = smem + buf * STAGE + (j < 4 ? ra + ks * 8192 + j * 1024 : rb + ks * 16384 + (j - 4) * 1024);
    f[j] = *(const i32x4*)p;
  };
  auto half = [&](i32x4 (&fc)[8], i32x4 (&fn)[8], u32 rbuf, int rks, int d0, u32 dbuf) {
#pragma unroll
    for (int i = 0; i < 16; ++i) {
      acc[i >> 2][i & 3] = __builtin_amdgcn_mfma_i32_16x16x64_i8(fc[i >> 2], fc[4 + (i & 3)], acc[i >> 2][i & 3], 0, 0, 0);
      if ((RDMASK >> i) & 1) rd(fn, __builtin_popcount(RDMASK & ((1u << i) - 1u)), rbuf, rks);
      if ((DMAMASK >> i) & 1) dma(d0 + __builtin_popcount(DMAMASK & ((1u << i) - 1u)), dbuf);
      __builtin_amdgcn_sched_barrier(0);
    }
  };
  locate(k0);
#pragma unroll
  for (int n = 0; n < 6; ++n) dma(n, 0);
  locate(k0 + 1);
#pragma unroll
  for (int n = 0; n < 6; ++n) dma(n, 1);
  locate(k0 + 2);
#pragma unroll
  for (int n = 0; n < 3; ++n) dma(n, 2);
  asm volatile("s_waitcnt vmcnt(9)" ::: "memory");
  bar();
#pragma unroll
  for (int n = 0; n < 8; ++n) rd(f0, n, 0, 0);
  u32 buf = 0;
  for (int kb = k0; kb < nk; ++kb) {
    const u32 b1 = buf == 2 ? 0u : buf + 1, b2 = buf == 0 ? 2u : buf - 1;
    __builtin_amdgcn_sched_barrier(0);
    half(f0, f1, buf, 1, 3, b2);
    locate(kb + 3);
    asm volatile("s_waitcnt vmcnt(6)" ::: "memory");
    bar();
    __builtin_amdgcn_sched_barrier(0);
    half(f1, f0, b1, 0, 0, buf);
    buf = b1;
  }
  waitv0();
}

constexpr u32 NPULL = 88 - NPUSH;
constexpr u32 NIN = NPULL * 4, CUT_MAX = 28;
__device__ __forceinline__ u32 import_id(u32 o, u32 q, u32 rank) {
  if (o >= 14) {
    o += NPUSH - 4;
    if (o < 18) return 64 + 4 + (o - 14);
    if (o >= 32 && o < 56) return 64 + 8 + (o - 32);
  }
  u32 s, e;
  if (o < 14) { s = 0; e = o; }
  else if (o < 32) { s = 1; e = o - 18; }
  else if (o < 70) { s = 2; e = o - 56; }
  else { s = 3; e = o - 70 + 2 * ((rank - q - 1) & 7u); e -= e >= 14 ? 14 : 0; }
  u32 p = e >> 1;
  p += (p >= q) ? 1 : 0;
  return (4 * p + s) * 2 + (e & 1);
}
__device__ __forceinline__ void tile_sc(u32 i, u32& s, u32& c) {
  if (i < 4) { s = i >> 1; c = i & 1; }
  else if (i < 16) { s = i & 1; c = i >> 1; }
  else { s = i >> 3; c = i & 7; }
}
__device__ __forceinline__ u32 draw(char* p) {
  u32 t;
  u64 save;
  asm volatile("s_mov_b64 %1, exec\n\ts_mov_b64 exec, 1\n\tglobal_atomic_add %0, %2, %3, off sc0\n\ts_mov_b64 exec, %1"
               : "=&v"(t), "=&s"(save) : "v"(p), "v"(1u) : "memory");
  return t;
}
__device__ __forceinline__ u32 take(u32 t) {
  u32 r;
  asm volatile("s_nop 0\n\tv_readfirstlane_b32 %0, %1\n\ts_nop 4" : "=s"(r) : "v"(t));
  return r;
}
__device__ __forceinline__ u32 lds_add(volatile LDS_AS u32* p, u32 x) {
  u32 t;
  u64 save;
  asm volatile("s_mov_b64 %1, exec\n\ts_mov_b64 exec, 1\n\tds_add_rtn_u32 %0, %2, %3\n\ts_waitcnt lgkmcnt(0)\n\ts_mov_b64 exec, %1"
               : "=&v"(t), "=&s"(save) : "v"((u32)(size_t)p), "v"(x) : "memory");
  return UNI(t);
}
__device__ __forceinline__ u32 peek_word(char* p) {
  u32 t;
  asm volatile("global_atomic_add %0, %1, %2, off sc0\n\ts_waitcnt vmcnt(0)" : "=&v"(t) : "v"(p), "v"(0u) : "memory");
  return t;
}
__device__ __forceinline__ void lds_inc(volatile LDS_AS u32* p) {
  u64 save;
  asm volatile("s_mov_b64 %0, exec\n\ts_mov_b64 exec, 1\n\tds_add_u32 %1, %2\n\ts_mov_b64 exec, %0"
               : "=&s"(save) : "v"((u32)(size_t)p), "v"(1u) : "memory");
}
__device__ __forceinline__ void bump(char* p) {
  u64 save;
  asm volatile("s_mov_b64 %0, exec\n\ts_mov_b64 exec, 1\n\tglobal_atomic_add %1, %2, off\n\ts_mov_b64 exec, %0"
               : "=&s"(save) : "v"(p), "v"(1u) : "memory");
}

__global__ void __launch_bounds__(NTHR, 2) gemm_rs_kernel(Params P) {
  __shared__ __attribute__((aligned(16))) char smem[LDS_BYTES];
  const int tid = threadIdx.x, lane = tid & 63;
  const int wave = UNI(tid >> 6);
  const u32 b = blockIdx.x;
  const u32 rank = (u32)P.rank & 7u;
  char* const me = P.me;
  asm volatile("" :: "s"(P.x), "s"(P.w), "s"(me), "s"(P.out), "s"(P.ws_bytes), "s"(P.rank));
  const rsrc_t rme = mk_rsrc(me, P.ws_bytes);
  const rsrc_t rout = mk_rsrc(P.out, OUT_BYTES);
  volatile LDS_AS u32* ctl = (volatile LDS_AS u32*)((LDS_AS char*)smem + LDS_CTL);

  const u32 gi = b - NSTR;
  const u32 cnt = ld_agt(me + F_CNT + b * 128);
  if (b < NSTR) {
    if (tid < 256) ctl[tid] = tid == C_END ? 0xFFFFFFFFu : 0u;
    waitv0();
    bar();
    if (tid == 0) st_agt(me + F_CNT + b * 128, cnt + 1);
  }
  for (int e = 0; e < 2 && b >= NSTR; ++e) {
    u32 wp = 0xFFFFu, xp = 0xFFFFu, v = 0xFFFFu;
    if (e == 0) {
      if (gi < 64) {
        wp = (gi & 7) * 32 + (gi >> 3);
        if ((gi >> 3) >= NPUSH) __builtin_amdgcn_s_sleep(EXPORT_DELAY);
      } else if (gi < 80) {
        xp = ((gi - 64) >> 1) * 8 + (gi & 1);
      } else {
        v = gi - 80;
        __builtin_amdgcn_s_sleep(EXPORT_DELAY);
      }
    } else if (gi >= 80) {
      v = gi + 40;
    }
    if (v < 16) xp = (v >> 1) * 8 + 2 + (v & 1);
    else if (v < 208) wp = ((v - 16) / 24) * 32 + 8 + (v - 16) % 24;
    else if (v < 240) xp = ((v - 208) >> 2) * 8 + 4 + (v & 3);
    const u32 vs = wave * 4096 + lane * 16;
    if (wp != 0xFFFFu) {
      const u32 cg = wp >> 2, kc = wp & 3, dj = (cg >> 3) & 7;
      u32 d[16];
      const rsrc_t rw = mk_rsrc(P.w, (u32)KSL * NCOL);
      const u32 vw = (kc * 128 + (wave >> 2) * 64 + (lane >> 4) * 16) * NCOL + cg * 256 + (wave & 3) * 64 + (lane & 15) * 4;
#pragma unroll
      for (int r = 0; r < 16; ++r) d[r] = __builtin_amdgcn_raw_buffer_load_b32(rw, vw, r * NCOL, 0);
      char* const peer = P.ws[dj];
      const u32 call = UNI(cnt) + 1, par = D_PAR + (call & 1) * PAR_BYTES;
      const bool push = (wp & 31) < NPUSH && dj != rank;
      u32x4 o[4];
#pragma unroll
      for (int r4 = 0; r4 < 4; ++r4) {
        const u32 t0 = perm(d[4 * r4 + 1], d[4 * r4], 0x05010400u), t1 = perm(d[4 * r4 + 1], d[4 * r4], 0x07030602u);
        const u32 t2 = perm(d[4 * r4 + 3], d[4 * r4 + 2], 0x05010400u), t3 = perm(d[4 * r4 + 3], d[4 * r4 + 2], 0x07030602u);
        o[0][r4] = perm(t2, t0, 0x05040100u);
        o[1][r4] = perm(t2, t0, 0x07060302u);
        o[2][r4] = perm(t3, t1, 0x05040100u);
        o[3][r4] = perm(t3, t1, 0x07060302u);
      }
      if (push) {
        const rsrc_t rj = mk_rsrc(peer, P.ws_bytes);
#pragma unroll
        for (int t = 0; t < 4; ++t) st128(o[t], rj, vs + t * 1024, par + P_IW + (rank * 32 + (wp & 31)) * WBLK, AUX_SYS);
      }
#pragma unroll
      for (int t = 0; t < 4; ++t) st128(o[t], rme, vs + t * 1024, par + P_EW + wp * WBLK, AUX_SYS);
      waitv0();
      bar();
      if (tid == 0) st_sys(peer + (push ? F_I : F_S) + (rank * 96 + 64 + (wp & 31)) * 128, push ? call * 4 : call);
      if (tid == 64) st_sys(me + F_LW + wp * 128, call);
    } else if (xp != 0xFFFFu) {
      u32x4 o[4];
      const rsrc_t rx = mk_rsrc(P.x, 4096u * KSL);
      const u32 vx = ((xp >> 1) * 128 + (wave & 1) * 64 + (lane & 15)) * KSL + ((xp & 1) * 2 + (wave >> 2)) * 128 + ((wave >> 1) & 1) * 64 + (lane >> 4) * 16;
#pragma unroll
      for (int u = 0; u < 4; ++u) o[u] = __builtin_amdgcn_raw_buffer_load_b128(rx, vx, u * 16 * KSL, 0);
      const u32 call = UNI(cnt) + 1, par = D_PAR + (call & 1) * PAR_BYTES;
#pragma unroll
      for (int t = 0; t < 4; ++t) st128(o[t], rme, vs + t * 1024, par + P_EX + xp * (2 * XBLK), AUX_SYS);
      waitv0();
      bar();
      if (lane == 0) st_sys(P.ws[wave] + F_S + (rank * 96 + xp) * 128, call);
    }
    if (e == 0 && tid == 64) st_agt(me + F_CNT + b * 128, cnt + 1);
  }
  const u32 call = UNI(cnt) + 1, par = D_PAR + (call & 1) * PAR_BYTES;

  if (b < NSTR) {
    const u32 q = (rank + 1 + b % 7) & 7u, idx = b / 7;
    const rsrc_t rq = mk_rsrc(P.ws[q], P.ws_bytes);
    const u32 fo0 = F_S + (q * 96 + import_id(lane, q, rank)) * 128;
    if (wave == 0) {
      __builtin_amdgcn_s_setprio(3);
      const u32 v16 = lane * 16, vB = lane < 16 ? lane * 16 : OOB;
      if (lane == 0) __hip_atomic_fetch_max((u64*)(me + F_T + q * 128), (u64)call << 32, __ATOMIC_RELAXED, __HIP_MEMORY_SCOPE_AGENT);
      u32 t = idx, z = idx, lk = q, xe = 0, n = 0, so = 0, slot = 0, raw = 0;
      u64 seen;
      bool ok = true, slot_free = true;
      {
        const u64 t0 = __builtin_amdgcn_s_memrealtime();
        for (;;) {
          seen = __ballot(ld_sys(me + fo0) >= call);
          if ((seen >> (idx >> 2)) & 1) break;
          if (UNI(ctl[C_FAIL]) || __builtin_amdgcn_s_memrealtime() - t0 > DEADLINE) { ctl[C_FAIL] = 1; ok = false; break; }
          __builtin_amdgcn_s_sleep(2);
        }
        waitv0();
      }
      int iss = 0, known = 0;
      u32 lim = 0, xs = 0;
      auto prepare = [&]() -> bool {
        lk = q;
        z = t;
        raw = 0;
        bool known_unit = true;
        if (t + CUT_MAX >= NIN) {
          if (!lim) { lim = UNI(ctl[C_PLAN]); xs = UNI(ctl[C_PLAN + 1]); }
          known_unit = lim != 0;
          if (t >= (lim >> 16)) {
            z = NIN + t - (lim & 0xFFFFu);
          } else {
            const u32 en = UNI(ctl[C_TAB + t - (NIN - CUT_MAX)]);
            z = en & 0x3FFu;
            lk = (en >> 10) & 7u;
            xe = en >> 13;
          }
        }
        const u32 i = ((z - NIN) >> 3) & 31;
        const u32 dn = ctl[C_DONE + slot], wd = ctl[z < NIN ? C_IN + (z >> 7) : C_READY];
        const u32 rw = ctl[C_RAWM + i];
        bool rdy;
        if (z < NIN) {
          const u32 id = import_id(z >> 2, lk, rank);
          so = par + (lk != q ? P_IX + lk * (2 * MiB) + id * (2 * XBLK) : id < 64 ? P_EX + id * (2 * XBLK) : P_EW + (rank * 32 + (id - 64)) * WBLK) + (z & 3) * 8192;
          rdy = lk != q || ((UNI(wd) >> ((z >> 2) & 31)) & 1) || (z < 256 && ((seen >> (z >> 2)) & 1));
        } else {
          const u32 w8 = (z - NIN) & 7;
          u32 s, c;
          tile_sc(i, s, c);
          raw = (UNI(rw) >> w8) & 1;
          so = raw ? OOB : par + P_PK + (((rank * 4 + s) * 8 + c) * 8 + w8) * SUB;
          rdy = (UNI(wd) >> i) & 1;
        }
        slot_free = UNI(dn) == 4 * (n / NSLOT);
        return known_unit && (z >= NIN + 256 || (rdy && slot_free));
      };
      prepare();
#define LEFT(N, k) do { asm volatile("s_waitcnt vmcnt(%0)" :: "i"(N) : "memory"); ctl[C_LAND] = max(known, iss + (k) - (N)); } while (0)
      for (;;) {
        u32 fail = 0;
        if (!ok && !prepare()) {
          LEFT(4, 0);
          if (!prepare()) {
            waitv0();
            ctl[C_LAND] = known = iss;
            for (;;) {
              const u32 f = UNI(ctl[C_FAIL]);
              if (prepare()) break;
              if (f && slot_free && (lim || t + CUT_MAX < NIN)) { fail = 1; so = OOB; break; }
              __builtin_amdgcn_s_sleep(1);
            }
          }
        }
        if (z >= NIN + 256) break;
        u32 src = q;
        if (lk != q && !fail && !((UNI(ctl[C_HAVE]) >> xe) & 1)) {
          const u32 id = import_id(z >> 2, lk, rank);
          const u32 have = UNI(ld_sys(P.ws[q] + F_I + (lk * 96 + id) * 128));
          waitv0();
          ctl[C_LAND] = known = iss;
          if (have < call * 4) {
            src = lk;
            so = par + P_EX + id * (2 * XBLK) + (z & 3) * 8192;
            if (!wave_wait(me, F_S + (lk * 96 + id) * 128, lane == 0, call, call)) { ctl[C_FAIL] = 1; fail = 1; so = OOB; }
          }
          asm volatile("" ::: "memory");
        }
        if (lim && t + 20 >= xs && t < xs + 16) ctl[C_NEAR] = 1;
        ctl[C_BASE + slot] = iss;
        ctl[C_ASG + slot] = ((n + 1) << 16) | (lk << 13) | (fail << 11) | (raw << 10) | z;
        LDS_AS char* ld = (LDS_AS char*)smem + slot * SLOTB;
        const u32 sc = so;
        const bool out = z >= NIN;
        const rsrc_t rs = src == q ? rq : mk_rsrc(P.ws[src & 7], P.ws_bytes);
        u32 tk;
        auto advance = [&]() {
          t = take(tk) + WPL;
          slot = slot + 1 == NSLOT ? 0 : slot + 1;
          ++n;
          ok = prepare();
        };
#define PULL(k, vofs, grp, imm) \
        asm volatile("s_waitcnt vmcnt(%0)" :: "i"(LWIN - 1) : "memory"); \
        __builtin_amdgcn_raw_ptr_buffer_load_lds(rs, ld + (grp) * 4096, 16, vofs, sc + (grp) * 4096, imm, AUX_SYS); \
        asm volatile("" ::: "memory"); \
        if (!(((k) + 2 + LWIN) & 1)) ctl[C_LAND] = max(known, iss + (k) + 1 - (LWIN - 1)); \
        if ((k) + 1 == LWIN) advance();
#define PULLX(vofs, grp, imm) __builtin_amdgcn_raw_ptr_buffer_load_lds(rs, ld + (grp) * 4096, 16, vofs, sc + (grp) * 4096, imm, AUX_SYS);
        if (!out) {
          constexpr int LWIN = LW1;
          asm volatile("s_waitcnt vmcnt(%0)" :: "i"(LWIN - 1) : "memory");
          tk = draw(me + F_T + q * 128);
          ctl[C_LAND] = max(known, iss - (LWIN - 1));
          PULL(0, v16, 0, 0) PULL(1, v16, 0, 1024) PULL(2, v16, 0, 2048) PULL(3, v16, 0, 3072)
          PULL(4, v16, 1, 0) PULL(5, v16, 1, 1024) PULL(6, v16, 1, 2048) PULL(7, v16, 1, 3072)
          iss += 9;
        } else {
          constexpr int LWIN = LW2;
          asm volatile("s_waitcnt vmcnt(%0)" :: "i"(LWIN - 1) : "memory");
          tk = draw(me + F_T + q * 128);
          ctl[C_LAND] = max(known, iss - (LWIN - 1));
          PULL(0, v16, 2, 0) PULL(1, v16, 2, 1024) PULL(2, vB, 2, 2048)
          PULL(3, v16, 0, 0) PULL(4, v16, 0, 1024) PULL(5, v16, 0, 2048) PULL(6, v16, 0, 3072)
          PULL(7, v16, 1, 0)
          if (z < NIN + 256) {
            PULL(8, v16, 1, 1024) PULL(9, v16, 1, 2048) PULL(10, v16, 1, 3072)
          } else {
            PULLX(v16, 1, 1024) PULLX(v16, 1, 2048) PULLX(v16, 1, 3072)
          }
          iss += 12;
        }
      }
      ctl[C_END] = n;
      LEFT(8, 0); LEFT(6, 0); LEFT(4, 0); LEFT(2, 0); LEFT(0, 0);
#undef PULL
#undef PULLX
#undef LEFT
    } else {
      if (wave == 1) {
        const u32 fo1 = F_S + (q * 96 + import_id(64 + (lane < NPULL - 64 ? lane : 0), q, rank)) * 128;
        const u64 t0 = __builtin_amdgcn_s_memrealtime();
        bool inputs = false, planned = false;
        u32 en = 0, todo = 0, have = 0, looks = 0;
        auto table = [&](u32 cut, u32 ext, u32 dly) {
          const u32 keep = ~cut & ((1u << CUT_MAX) - 1u), nk = __builtin_popcount(keep);
          if (lane < CUT_MAX && ((keep >> lane) & 1)) ctl[C_TAB + __builtin_popcount(keep & ((1u << lane) - 1u))] = (q << 10) | (NIN - CUT_MAX + lane);
          if (lane < dly) ctl[C_TAB + nk + lane] = (q << 10) | (NIN + lane);
          if (lane < ext) ctl[C_TAB + nk + dly + lane] = (lane << 13) | ((en & 7u) << 10) | (NIN - CUT_MAX + ((en >> 3) & 31u));
          ctl[C_PLAN + 1] = NIN - CUT_MAX + nk + dly;
          asm volatile("s_waitcnt lgkmcnt(0)" ::: "memory");
          ctl[C_PLAN] = ((NIN - CUT_MAX + nk + dly + ext) << 16) | (NIN - CUT_MAX + nk + ext);
        };
        for (;;) {
          const u32 tw = ld_agt(me + F_TMO);
          bool fin = false;
          if (!inputs) {
            const u32 f0 = ld_sys(me + fo0), f1 = ld_sys(me + fo1);
            const u64 m0 = __ballot(f0 >= call), m1 = __ballot(f1 >= call);
            if (lane < 3) ctl[C_IN + lane] = lane == 0 ? (u32)m0 : lane == 1 ? (u32)(m0 >> 32) : (u32)m1;
            inputs = (m0 & m1) == ~0ull;
          } else {
            const u32 near = UNI(ctl[C_NEAR]);
            if (!planned) {
              const u32 pw = ld_agt(me + F_PLAN + q * 128 + (lane & 31) * 4);
              if (__all((pw >> 16) == (call & 0xFFFFu))) {
                en = pw & 0xFFFFu;
                todo = (u32)__ballot(lane < 24 && (en >> 8));
                table(__builtin_amdgcn_readlane(en, 30) | (__builtin_amdgcn_readlane(en, 31) << 14), __builtin_popcount(todo), min(__builtin_amdgcn_readlane(en, 29), DLY_MAX));
                planned = true;
              }
            }
            if (todo && near) {
              bool has = false;
              if (lane < 24 && ((todo >> lane) & 1))
                has = ld_sys(P.ws[q] + F_I + ((en & 7u) * 96 + import_id((NIN - CUT_MAX + ((en >> 3) & 31u)) >> 2, en & 7u, rank)) * 128) >= call * 4;
              const u32 got = (u32)__ballot(has);
              have |= got;
              ctl[C_HAVE] = have;
              todo = ++looks < 3 ? todo & ~got : 0u;
            }
            const u32 l = lane & 31;
            const u64 of = __hip_atomic_load((const u64*)(me + F_O + (q * 32 + l) * 128), __ATOMIC_RELAXED, __HIP_MEMORY_SCOPE_SYSTEM);
            const u32 pf = ld_agt(me + F_P + (q * 32 + l) * 128);
            const u32 rd = (u32)__ballot((u32)(of >> 32) >= call && pf >= call);
            if (lane < 32) ctl[C_RAWM + lane] = (u32)of;
            asm volatile("s_waitcnt lgkmcnt(0)" ::: "memory");
            ctl[C_READY] = rd;
            fin = rd == 0xFFFFFFFFu && planned;
          }
          if (UNI(tw) == call || UNI(ctl[C_FAIL]) || __builtin_amdgcn_s_memrealtime() - t0 > DEADLINE) {
            st_agt(me + F_TMO, call);
            if (!planned) table(0u, 0u, 0u);
            ctl[C_FAIL] = 1;
            break;
          }
          if (fin) break;
          if (inputs) __builtin_amdgcn_s_sleep(20); else __builtin_amdgcn_s_sleep(4);
        }
      }
      const float scale = P.sx[0] * P.sw[0];
      const u32 vo = (u32)(lane >> 4) * 4 * (NCOL * 4) + (lane & 15) * 16;
      auto peek = [&](u32 item) -> u32 {
        const u32 n = item >> 2, a = UNI(ctl[C_ASG + n % NSLOT]);
        return (a >> 16) == n + 1 ? a : 0u;
      };
      auto await = [&](u32 item) -> u32 {
        for (;;) {
          const u32 e = UNI(ctl[C_END]);
          const u32 a = peek(item);
          if (a || (item >> 2) >= e) return a;
          __builtin_amdgcn_s_sleep(1);
        }
      };
      auto landed = [&](u32 slot, u32 k) {
        const u32 need = UNI(ctl[C_BASE + slot]) + k;
        while ((int)(UNI(ctl[C_LAND]) - need) < 0) __builtin_amdgcn_s_sleep(1);
        asm volatile("" ::: "memory");
      };
      auto out_off = [&](u32 a, u32 item) -> u32 {
        const u32 z = (a & 0x3FFu) - NIN, w8 = z & 7;
        u32 s, c;
        tile_sc(z >> 3, s, c);
        return ((s * 128 + (w8 >> 2) * 64 + (item & 3) * 16) * NCOL + q * 2048 + c * 256 + (w8 & 3) * 64) * 4;
      };
      auto fetch = [&](u32 pb, u32 a, u32 item) {
        const u32 oo = ((a & 0x3FFu) >= NIN && !(a & 0x800u)) ? out_off(a, item) : OOB;
        LDS_AS char* ld = (LDS_AS char*)smem + pb;
        __builtin_amdgcn_raw_ptr_buffer_load_lds(rout, ld, 16, vo, oo, 0, AUX_SYS);
        __builtin_amdgcn_raw_ptr_buffer_load_lds(rout, ld, 16, vo + (NCOL * 4 - 1024), oo, 1024, AUX_SYS);
        __builtin_amdgcn_raw_ptr_buffer_load_lds(rout, ld, 16, vo + 2 * (NCOL * 4 - 1024), oo, 2048, AUX_SYS);
        __builtin_amdgcn_raw_ptr_buffer_load_lds(rout, ld, 16, vo + 3 * (NCOL * 4 - 1024), oo, 3072, AUX_SYS);
      };
      u32 pcur = LDS_PT + (wave - 1) * 8192;
      bool early = false;
      u32 item = lds_add(ctl + C_ITEM, 1u);
      u32 a = await(item);
      fetch(pcur, a, item);
      while (a) {
        const u32 n = item >> 2, m = item & 3, slot = n % NSLOT, zt = a & 0x3FFu;
        if (zt < NIN || (a & 0xC00u)) {
          if (zt < NIN) {
            if (!(a & 0x800u)) {
              const u32 lk = (a >> 13) & 7, id = import_id(zt >> 2, lk, rank);
              const u32 dso = par + (id < 64 ? P_IX + lk * (2 * MiB) + id * (2 * XBLK) : P_IW + (lk * 32 + (id - 64)) * WBLK) + (zt & 3) * 8192 + m * 2048;
              landed(slot, 3 + 2 * m);
              const char* ls = smem + slot * SLOTB + m * 2048 + lane * 16;
              const u32x4 v0 = *(const u32x4*)ls, v1 = *(const u32x4*)(ls + 1024);
              st128(v0, rme, lane * 16, dso, AUX_SYS);
              st128(v1, rme, lane * 16 + 1024, dso, AUX_SYS);
              waitv0();
              if ((lds_add(ctl + C_DONE + slot, 1u) & 3u) == 3u) bump(me + F_I + (lk * 96 + id) * 128);
            } else {
              lds_inc(ctl + C_DONE + slot);
            }
          } else {
            lds_inc(ctl + C_DONE + slot);
            const u32 z = zt - NIN, oo = out_off(a, item);
            u32 s, c;
            tile_sc(z >> 3, s, c);
            const bool miss = a & 0x800u;
            const u32 so = miss ? OOB : par + P_RAW + (((rank * 4 + s) * 8 + c) * 8 + (z & 7)) * 16384 + m * 4096;
            u32x4 v[4];
#pragma unroll
            for (int r = 0; r < 4; ++r) v[r] = __builtin_amdgcn_raw_buffer_load_b128(rq, lane * 16 + r * 1024, so, AUX_SYS);
            waitv0();
#pragma unroll
            for (int r = 0; r < 4; ++r) {
              const u32x4 pt = *(const u32x4*)(smem + pcur + r * 1024 + lane * 16);
              u32x4 y;
#pragma unroll
              for (int t = 0; t < 4; ++t) y[t] = miss ? 0x7FC00000u : __builtin_bit_cast(u32, (float)((int)v[r][t] + (int)pt[t]) * scale);
              st128(y, rout, vo + r * (NCOL * 4), oo, AUX_AGT);
            }
          }
          item = lds_add(ctl + C_ITEM, 1u);
          a = await(item);
          pcur ^= 4096;
          fetch(pcur, a, item);
          early = false;
          continue;
        }
        const u32 oo = out_off(a, item);
        const u32 item2 = lds_add(ctl + C_ITEM, 1u);
        u32 a2 = 0;
        const u32 base = UNI(ctl[C_BASE + slot]);
        auto until = [&](u32 need) {
          while ((int)(UNI(ctl[C_LAND]) - need) < 0) {
            if (!a2) {
              a2 = peek(item2);
              if (a2) fetch(pcur ^ 4096, a2, item2);
            }
            __builtin_amdgcn_s_sleep(1);
          }
          asm volatile("" ::: "memory");
        };
        until(base + 4);
        const char* ls = smem + slot * SLOTB;
        const u32* cw = (const u32*)(ls + 10240);
        const u32 lb = 7u * (lane >> 1);
        const u32 lp = __builtin_amdgcn_alignbit(cw[(lb >> 5) + 1], cw[lb >> 5], lb & 31u) & 127u;
        const u32x4 n0 = *(const u32x4*)(ls + 8192 + lane * 16), n1 = *(const u32x4*)(ls + 9216 + lane * 16);
        u32 total;
        const u32 wp = code_window(lp, lane, total);
        const u32 c0 = cw[wp >> 5], c1 = cw[(wp >> 5) + 1], c2 = cw[(wp >> 5) + 2];
        u64 B = ((u64)__builtin_amdgcn_alignbit(c2, c1, wp & 31u) << 32) | __builtin_amdgcn_alignbit(c1, c0, wp & 31u);
        if (lane & 1) B = rev64(B);
        B = (B << 1) | 1u;
        auto escs = [](u32 x) -> u32 { return __builtin_popcount(x & (x >> 1) & 0x44444444u); };
        u32 cnt = 0;
        if (m > 0) cnt += escs(n0[0]) + escs(n0[1]);
        if (m > 1) cnt += escs(n0[2]) + escs(n0[3]);
        if (m > 2) cnt += escs(n1[0]) + escs(n1[1]);
        u32 pos = 0, wd = (u32)B, pc = __builtin_popcount(wd);
        if (cnt >= pc) { cnt -= pc; wd = (u32)(B >> 32); pos = 32; }
#pragma unroll
        for (int wdt = 16; wdt >= 1; wdt >>= 1) {
          pc = __builtin_popcount(wd & ((1u << wdt) - 1u));
          if (cnt >= pc) { cnt -= pc; wd >>= wdt; pos += wdt; }
        }
        B >>= (pos & 63u);
        const u32 nd0 = m == 0 ? n0[0] : m == 1 ? n0[2] : m == 2 ? n1[0] : n1[2];
        const u32 nd1 = m == 0 ? n0[1] : m == 1 ? n0[3] : m == 2 ? n1[1] : n1[3];
        if (early && a2) asm volatile("s_waitcnt vmcnt(8)" ::: "memory");
        else if (early || a2) asm volatile("s_waitcnt vmcnt(4)" ::: "memory");
        else asm volatile("s_waitcnt vmcnt(0)" ::: "memory");
        u32x4 pt[4];
#pragma unroll
        for (int r = 0; r < 4; ++r) pt[r] = *(const u32x4*)(smem + pcur + r * 1024 + lane * 16);
#pragma unroll
        for (int r = 0; r < 4; ++r) {
          const u32 nd = r < 2 ? nd0 : nd1, ne = nd & (nd >> 1);
#pragma unroll
          for (int t = 0; t < 4; ++t) {
            const int sh = 8 * t + 4 * (r & 1);
            const u32 esc = (u32)__builtin_amdgcn_sbfe((int)ne, sh + 2, 1);
            const u32 len = ffbl((u32)B ^ 1u) & esc;
            B >>= (len & 63u);
            const u32 h = __builtin_amdgcn_ubfe(nd, sh + 2, 2) + len + esc;
            pt[r][t] += ((u32)__builtin_amdgcn_sbfe((int)nd, sh, 2) ^ (h << 1)) << 16;
          }
        }
        asm volatile("" : "+v"(pt[0]), "+v"(pt[1]), "+v"(pt[2]), "+v"(pt[3]));
        until(base + 6 + 2 * m);
        const u32x4 l0 = *(const u32x4*)(ls + m * 2048 + lane * 16), l1 = *(const u32x4*)(ls + m * 2048 + 1024 + lane * 16);
        asm volatile("s_waitcnt lgkmcnt(0)" :: "v"(l0), "v"(l1) : "memory");
        lds_inc(ctl + C_DONE + slot);
#pragma unroll
        for (int r = 0; r < 4; ++r) {
          u32x4 y;
#pragma unroll
          for (int t = 0; t < 4; ++t) {
            const u32 lo = r < 2 ? l0[(r & 1) * 2 + (t >> 1)] : l1[(r & 1) * 2 + (t >> 1)];
            y[t] = __builtin_bit_cast(u32, (float)(int)(pt[r][t] + ((t & 1) ? lo >> 16 : lo & 0xFFFFu)) * scale);
          }
          st128(y, rout, vo + r * (NCOL * 4), oo, AUX_AGT);
        }
        item = item2;
        a = a2;
        early = a2 != 0;
        pcur ^= 4096;
        if (!a2) {
          a = await(item);
          fetch(pcur, a, item);
        }
      }
    }
    return;
  }

  const u32 vo = ((u32)(wave >> 2) * 64 + (lane >> 4) * 4) * (NCOL * 4) + ((wave & 3) * 64 + (lane & 15) * 4) * 4;
  i32x4 acc[4][4];
  for (int job = 0; job < 4; ++job) {
    const bool partial = job < 2;
    u32 t;
    if (job == 0) t = gi;
    else if (job == 1) { if (gi < NGEM - 24) continue; t = gi + 24; }
    else if (job == 2) { if (gi >= 192) break; t = gi; }
    else { if (gi >= 64) break; t = 192 + gi; }
    u32 g, c, cg_self, foff, s, dst, oi, tgt = call;
    int dskip, nk;
    bool act, own = true;
    if (partial) {
      const u32 r = t >> 3;
      oi = (r / 7) * 8 + (t & 7);
      tile_sc(oi, s, c);
      dst = (rank + 1 + r % 7) & 7u;
      g = rank * 4 + s;
      cg_self = dst * 8 + c;
      dskip = 8; nk = 4;
      act = lane < 6;
      foff = lane < 4 ? F_LW + (cg_self * 4 + lane) * 128 : F_S + (rank * 96 + g * 2 + (lane & 1)) * 128;
    } else {
      u32 pd;
      if (t < 128) { oi = t >> 3; pd = t & 7; }
      else { oi = (t >> 6) * 8 + (t & 7); pd = (t >> 3) & 7; }
      tile_sc(oi, s, c);
      dst = (rank + 1 + pd) & 7u;
      g = dst * 4 + s;
      cg_self = rank * 8 + c;
      dskip = dst == rank ? 8 : (int)((dst - rank) & 7u);
      nk = dst == rank ? 32 : 28;
      const u32 q = lane / 6, e = lane % 6;
      act = lane < 48 && (q != dst || dst == rank);
      const u32 id = e < 4 ? 64 + c * 4 + e : g * 2 + (e & 1);
      foff = (q == rank ? F_S : F_I) + ((q & 7) * 96 + id) * 128;
      own = q == rank;
      tgt = own ? call : call * 4;
    }
#pragma unroll
    for (int m = 0; m < 4; ++m)
#pragma unroll
      for (int tt = 0; tt < 4; ++tt) acc[m][tt] = (i32x4){0, 0, 0, 0};
    bool good = true;
    for (int part = 0; part < (partial ? 1 : 2) && good; ++part) {
      good = wg_wait(smem, me, foff, act && own == (part == 0), tgt, call, wave);
      if (good) gemm_tile(smem, rme, (int)rank, par, g, cg_self, c, dskip, part ? 4 : 0, part ? nk : 4, wave, lane, acc);
    }
    if (partial) {
      if (good) {
        const u32 oo = (s * 128 * NCOL + dst * 2048 + c * 256) * 4;
#pragma unroll
        for (int m = 0; m < 4; ++m)
#pragma unroll
          for (int r = 0; r < 4; ++r) {
            const u32x4 y = {(u32)acc[m][0][r], (u32)acc[m][1][r], (u32)acc[m][2][r], (u32)acc[m][3][r]};
            st128(y, rout, vo + (m * 16 + r) * (NCOL * 4), oo, AUX_AGT);
          }
        waitv0();
        bar();
        if (tid == 0) st_agt(me + F_P + (dst * 32 + oi) * 128, call);
      }
    } else if (dst == rank) {
      const float scale = P.sx[0] * P.sw[0];
      const u32 oo = (s * 128 * NCOL + rank * 2048 + c * 256) * 4;
#pragma unroll
      for (int m = 0; m < 4; ++m)
#pragma unroll
        for (int r = 0; r < 4; ++r) {
          u32x4 y;
#pragma unroll
          for (int tt = 0; tt < 4; ++tt) y[tt] = good ? __builtin_bit_cast(u32, (float)acc[m][tt][r] * scale) : 0x7FC00000u;
          st128(y, rout, vo + (m * 16 + r) * (NCOL * 4), oo, AUX_AGT);
        }
    } else if (good) {
      if (tid == 0) ctl[C_RAW] = 0;
      bar();
      const u32 sub = (g * 8 + c) * 8 + wave;
      const u32 so = par + P_PK + sub * SUB;
#pragma unroll
      for (int m = 0; m < 4; ++m)
#pragma unroll
        for (int hf = 0; hf < 2; ++hf) {
          const u32x4 lo = {perm(acc[m][1][2 * hf], acc[m][0][2 * hf], 0x05040100u), perm(acc[m][3][2 * hf], acc[m][2][2 * hf], 0x05040100u),
                            perm(acc[m][1][2 * hf + 1], acc[m][0][2 * hf + 1], 0x05040100u), perm(acc[m][3][2 * hf + 1], acc[m][2][2 * hf + 1], 0x05040100u)};
          st128(lo, rme, lane * 16 + m * 2048 + hf * 1024, so, AUX_SYS);
        }
      LDS_AS u32* cw = (LDS_AS u32*)((LDS_AS char*)smem + wave * 512);
      cw[lane] = 0;
      if (lane < 8) cw[64 + lane] = 0;
      u64 B = 1;
      u32 any = 0;
      u16x2 sum2 = {0, 0};
      u32x4 nb[2];
#pragma unroll
      for (int m = 3; m >= 0; --m) {
        u32 hb[4];
#pragma unroll
        for (int r = 3; r >= 0; --r) {
          u32 nv[2];
#pragma unroll
          for (int pr = 1; pr >= 0; --pr) {
            const u32 qv = perm(acc[m][2 * pr + 1][r], acc[m][2 * pr][r], 0x07060302u);
            const s16x2 q2 = __builtin_bit_cast(s16x2, qv);
            const u32 sg = __builtin_bit_cast(u32, q2 >> (s16x2){15, 15});
            const u16x2 h = __builtin_bit_cast(u16x2, __builtin_bit_cast(u32, q2 >> (s16x2){1, 1}) ^ sg);
            const u16x2 len = __builtin_elementwise_sub_sat(h, (u16x2){2, 2});
            const u32 lw = __builtin_bit_cast(u32, len);
            B = (B << ((lw >> 16) & 63u)) | 1u;
            B = (B << (lw & 63u)) | 1u;
            sum2 += len;
            any |= lw;
            const u32 t2 = __builtin_bit_cast(u32, __builtin_elementwise_min(h, (u16x2){3, 3})) << 2;
            nv[pr] = (qv & 0x00010001u) | (sg & 0x00020002u) | t2;
          }
          hb[r] = perm(nv[1], nv[0], 0x06040200u);
        }
        nb[m >> 1][(m & 1) * 2] = hb[0] | (hb[1] << 4);
        nb[m >> 1][(m & 1) * 2 + 1] = hb[2] | (hb[3] << 4);
      }
      const u32 sum = (u32)sum2[0] + sum2[1];
      const u32 lp = sum + __builtin_amdgcn_update_dpp(0u, sum, 0xB1, 0xF, 0xF, false);
      u32 total;
      const u32 wp = code_window(lp, lane, total);
      if (__any(sum > 63u || (any & 0xFFE0FFE0u)) || total > 2048u) {
        if (lane == 0) __hip_atomic_fetch_or((u32*)(smem + LDS_CTL) + C_RAW, 1u << wave, __ATOMIC_RELAXED, __HIP_MEMORY_SCOPE_WORKGROUP);
#pragma unroll
        for (int m = 0; m < 4; ++m)
#pragma unroll
          for (int r = 0; r < 4; ++r) {
            const u32x4 y = {(u32)acc[m][0][r], (u32)acc[m][1][r], (u32)acc[m][2][r], (u32)acc[m][3][r]};
            st128(y, rme, lane * 16 + (m * 4 + r) * 1024, par + P_RAW + sub * 16384, AUX_SYS);
          }
      } else {
        st128(nb[0], rme, lane * 16 + 8192, so, AUX_SYS);
        st128(nb[1], rme, lane * 16 + 9216, so, AUX_SYS);
        const u64 V = (lane & 1) ? rev64(B >> 1) : B >> 1;
        const u32 sf = wp & 31u, lb = 7u * (lane >> 1);
        const u64 L = (u64)((lane & 1) ? 0u : lp) << (lb & 31u);
        __hip_atomic_fetch_or(cw + (lb >> 5), (u32)L, __ATOMIC_RELAXED, __HIP_MEMORY_SCOPE_WAVEFRONT);
        __hip_atomic_fetch_or(cw + (lb >> 5) + 1, (u32)(L >> 32), __ATOMIC_RELAXED, __HIP_MEMORY_SCOPE_WAVEFRONT);
        __hip_atomic_fetch_or(cw + (wp >> 5), (u32)V << sf, __ATOMIC_RELAXED, __HIP_MEMORY_SCOPE_WAVEFRONT);
        __hip_atomic_fetch_or(cw + (wp >> 5) + 1, (u32)((V << sf) >> 32), __ATOMIC_RELAXED, __HIP_MEMORY_SCOPE_WAVEFRONT);
        __hip_atomic_fetch_or(cw + (wp >> 5) + 2, (u32)(((V >> 32) << sf) >> 32), __ATOMIC_RELAXED, __HIP_MEMORY_SCOPE_WAVEFRONT);
        asm volatile("s_waitcnt lgkmcnt(0)" ::: "memory");
        __builtin_amdgcn_raw_buffer_store_b32(*(volatile LDS_AS u32*)(cw + lane), rme, lane * 4 + 10240, so, AUX_SYS);
      }
      waitv0();
      bar();
      if (tid == 0) {
        const u64 f = ((u64)call << 32) | ctl[C_RAW];
        __hip_atomic_store((u64*)(P.ws[dst] + F_O + (rank * 32 + oi) * 128), f, __ATOMIC_RELAXED, __HIP_MEMORY_SCOPE_SYSTEM);
      }
    }
  }
  if (gi == NGEM - 1 && wave == 0) {
    u32 cnt = 0, cut = 0, ext = 0, far = 0;
    u64 e0 = 0, e1 = 0, e2 = 0;
    const u64 t0 = __builtin_amdgcn_s_memrealtime();
    for (;;) {
      if (lane < 8 && lane != rank) cnt = peek_word(me + F_T + lane * 128);
      if (__any(cnt + PLAN_AT >= NIN) || __builtin_amdgcn_s_memrealtime() - t0 > DEADLINE) break;
      __builtin_amdgcn_s_sleep(48);
    }
    auto top8 = [&](u32 x) -> u32 {
      x = max(x, (u32)__builtin_amdgcn_update_dpp(0u, x, 0x111, 0xF, 0xF, false));
      x = max(x, (u32)__builtin_amdgcn_update_dpp(0u, x, 0x112, 0xF, 0xF, false));
      x = max(x, (u32)__builtin_amdgcn_update_dpp(0u, x, 0x114, 0xF, 0xF, false));
      return __builtin_amdgcn_readlane(x, 7);
    };
    cnt = min(cnt, 4000u);
    u32 eff = cnt;
    for (int it = 0; it < 40; ++it) {
      const u32 gv = top8((lane != rank && ext == 0) ? ((4095u - eff) << 3) | lane : 0u);
      const u32 v = gv & 7u, lo = 4095u - (gv >> 3);
      const u32 gap = cnt - __builtin_amdgcn_readlane(cnt, v), held = gap < SAFE + 68u ? min(7u, ((SAFE + 68u - gap) >> 2) + 1u) : 0u;
      u32 sh = 2 * (((rank - v - 1) & 7u) + 7 - ((lane - v - 1) & 7u));
      sh -= sh >= 14 ? 14 : 0;
      const u32 win = ((1u << held) - 1u) * 0x4001u;
      u32 cand = (win >> (sh + 7 >= 14 ? sh - 7 : sh + 7)) & 127u;
      cand = (cand | (cand << 12)) & 0x000F000Fu;
      cand = (cand | (cand << 6)) & 0x03030303u;
      cand = (cand | (cand << 3)) & 0x11111111u;
      cand = (cand * 15u) & ~__builtin_amdgcn_readlane(cut, v);
      const u32 hv = top8((lane != rank && lane != v && cut == 0 && ext < EXT_MAX && cand) ? (eff << 3) | lane : 0u);
      const u32 h = hv & 7u;
      if ((hv >> 3) < lo + LAG_MIN) break;
      u32 ch = __builtin_amdgcn_readlane(cand, h);
      const u32 many = min(min(((hv >> 3) - lo - LAG_MIN) / 6 + 1, EXT_MAX - __builtin_amdgcn_readlane(ext, h)), (u32)__builtin_popcount(ch));
      for (u32 j = 0; j < many; ++j, ch &= ch - 1) {
        const u32 u = __builtin_ctz(ch);
        if (lane == h) {
          const u64 x = (u64)(v | (u << 3)) << (8 * (ext & 7));
          if (ext < 8) e0 |= x; else if (ext < 16) e1 |= x; else e2 |= x;
          const u32 at = 7 + (u >> 2) + sh;
          far = max(far, gap + 4 * (at >= 14 ? at - 14 : at));
          ++ext;
          --eff;
        }
        if (lane == v) { cut |= 1u << u; ++eff; }
      }
    }
#pragma unroll
    for (int part = 0; part < 4; ++part) {
      const int lnk = part * 2 + (lane >> 5), k = lane & 31;
      const u64 s0 = __shfl(e0, lnk), s1 = __shfl(e1, lnk), s2 = __shfl(e2, lnk), es = k < 8 ? s0 : k < 16 ? s1 : s2;
      const u32 cs = __shfl(cut, lnk), xs = __shfl(ext, lnk), fs = __shfl(far, lnk);
      const u32 w = k < 29 ? ((u32)k < xs ? 0x100u | ((u32)(es >> (8 * (k & 7))) & 255u) : 0u)
                  : k == 29 ? (fs > SAFE ? ((fs - SAFE) * 13u) >> 4 : 0u) : (cs >> (k == 31 ? 14 : 0)) & 0x3FFFu;
      st_agt(me + F_PLAN + lnk * 128 + k * 4, (call << 16) | w);
    }
  }
}

}

extern "C" void kernel_launch(int rank, int nranks, ncclComm_t comm,
                              hipStream_t stream,
                              void* const* d_in, const int* in_sizes, int n_in,
                              void* d_out, int out_size,
                              const int* in_dtypes, int out_dtype,
                              void* const* d_out_all,
                              void* const* d_ws_all, size_t ws_bytes) {
  if (nranks != NR || (unsigned)rank >= (unsigned)NR || n_in < 4 || ws_bytes < (size_t)WS_NEED + MiB ||
      (size_t)out_size * 4 != OUT_BYTES || in_sizes[0] != 4096 * KSL || in_sizes[1] != KSL * NCOL) {
    fprintf(stderr, "kernel_launch: unexpected shapes\n");
    return;
  }
  Params p;
  p.x = (const signed char*)d_in[0];
  p.w = (const signed char*)d_in[1];
  p.sx = (const float*)d_in[2];
  p.sw = (const float*)d_in[3];
  p.out = (float*)d_out;
  for (int i = 0; i < NR; ++i) p.ws[i] = (char*)d_ws_all[i];
  p.me = p.ws[rank];
  p.ws_bytes = WS_NEED + MiB;
  p.rank = rank;
  gemm_rs_kernel<<<NWG, NTHR, 0, stream>>>(p);
  HIP_CHECK(hipGetLastError());
}
```
